```python
import numpy as np

kernel_name = "gpu_a2a_gemm_m4096_k4096_n16384_e4m3e5m2_dequant"


ABSMAX_RTOL = 0.05

COMPUTE_DTYPE = "float32"
EPILOGUE = "dequant"
IN_QUANT = None
OUT_QUANT = None
IN_FP8 = "e4m3e5m2"
IN_DTYPES = {"x": "e4m3", "w_mat": "e5m2",
             "scale_x": "float32", "scale_w": "float32"}
WORLD_SIZE = 8

def _snap_fp8(x, mant_bits, max_abs, min_normal):
    a = np.abs(x.astype(np.float64))
    _m, _e = np.frexp(a)
    q = np.where(a >= min_normal, np.ldexp(1.0, _e - 1 - mant_bits),
                 min_normal * 2.0 ** -mant_bits)
    snapped = np.minimum(np.round(a / q) * q, max_abs)
    return (np.sign(x) * snapped).astype(np.float32)

def setup_inputs():

    x = _snap_fp8(np.random.randn(4096, 4096).astype(np.float32), 3, 448.0, 2.0 ** -6)
    w_mat = _snap_fp8(np.random.randn(4096, 16384).astype(np.float32), 2, 57344.0, 2.0 ** -14)
    scale_x = (0.02 * (1.0 + np.random.rand(1))).astype(np.float32)
    scale_w = (0.02 * (1.0 + np.random.rand(1))).astype(np.float32)
    return {"x": x, "w_mat": w_mat, "scale_x": scale_x, "scale_w": scale_w}

def reference(x, w_mat, scale_x, scale_w):
    acc = (x @ w_mat).astype(np.float32)
    y = (acc * (scale_x * scale_w)).astype(np.float32)
    return y
```

```cpp
#include <hip/hip_runtime.h>
#include <rccl/rccl.h>
#include <cstdio>
#include <cstddef>

#define HIP_CHECK(expr) do { hipError_t _e = (expr); if (_e != hipSuccess) { \
    fprintf(stderr, "HIP error %s at %s:%d\n", hipGetErrorString(_e), __FILE__, __LINE__); \
    return; } } while (0)

namespace {
constexpr int NW = 4;
constexpr int NRANK = 8;
constexpr unsigned MROWS = 512, KDIM = 4096, NDIM = 16384, KSH = 512;
constexpr unsigned WS_FLAG = 0;
constexpr unsigned WS_TMO = 20480;
constexpr unsigned PHS = 2;
constexpr unsigned NREP = 8;
constexpr unsigned WS_REP = 24576;
constexpr unsigned WS_CTR = 32768;
constexpr unsigned WS_STG = 65536;
constexpr unsigned STG_PAR = 28u * 65536u;
constexpr unsigned WS_DBG = WS_STG + 2u * STG_PAR;
constexpr unsigned WS_USED = WS_DBG + 256u * 8u * 512u;

typedef unsigned u32x4 __attribute__((ext_vector_type(4)));
struct Srd { unsigned w[4]; };
struct Params {
  Srd w, x, out;
  const float* sx; const float* sw;
  Srd wsr[NRANK];
  unsigned rank;
};

static_assert(offsetof(Params, x) == 16 && offsetof(Params, out) == 32 && offsetof(Params, sx) == 48 && offsetof(Params, sw) == 56 &&
              offsetof(Params, wsr) == 64 && offsetof(Params, rank) == 192, "kernel-argument offsets are used by the asm below");

__device__ __forceinline__ unsigned rfl(unsigned v) { return __builtin_amdgcn_readfirstlane(v); }
__device__ __forceinline__ unsigned rowof(unsigned G, unsigned r) { return 64 * (r >> 3) + 8 * G + (r & 7); }
inline Srd make_srd(const void* p, unsigned bytes) {
  const unsigned long long a = (unsigned long long)p;
  Srd s; s.w[0] = (unsigned)a; s.w[1] = (unsigned)(a >> 32) & 0xffffu; s.w[2] = bytes; s.w[3] = 0x00020000u; return s;
}
}

__global__ __launch_bounds__(256, 1) void a2a_gemm_kernel(Params P) {
  __shared__ __attribute__((aligned(1024))) unsigned char smem[163840];
  unsigned long long st0l = 0;

  const auto karg = __builtin_amdgcn_kernarg_segment_ptr();
  u32x4 wsrd, xsrd, osrd, asrd, psrd; unsigned long long psx, psw, pfl; unsigned rank;
  asm volatile("s_load_dwordx4 %0, %8, 0x0\n\ts_load_dwordx4 %1, %8, 0x10\n\ts_load_dwordx4 %3, %8, 0x40\n\ts_load_dword %7, %8, 0xc0\n\t"
               "s_load_dwordx4 %2, %8, 0x20\n\ts_load_dwordx2 %4, %8, 0x30\n\ts_load_dwordx2 %5, %8, 0x38\n\ts_load_dwordx2 %6, %8, 0x40"
               : "=&s"(wsrd), "=&s"(xsrd), "=&s"(osrd), "=&s"(asrd), "=&s"(psx), "=&s"(psw), "=&s"(pfl), "=&s"(rank) : "s"(karg) : "memory");
  __builtin_amdgcn_sched_barrier(0);
  const unsigned tid = threadIdx.x;
  unsigned l = tid & 63, w = rfl(tid >> 6), b = blockIdx.x;
  const unsigned vct = (((b >> 3) * 4 + w) * 2 + (l & 1)) * 128;
  const unsigned sm0 = w * 4096, sctr = WS_CTR + b * 64;
  const unsigned srot = ((b >> 4) & 3u) * 128;
  constexpr unsigned ts = 3;
  const unsigned tq = b >> 4;
  const unsigned tn = tq - (tq > ts ? 1u : 0u) - (tq > ts + 8 ? 1u : 0u);
  const unsigned kk = tn * 2 + ((b >> 3) & 1), kq = kk / 7;
  const bool grp0 = (b & 7) == 0;
  const unsigned dd = ((tq & 7u) == ts ? 0u : (grp0 ? 1 + (kk - 7 * kq) : (b & 7))) & 7u;
  asm volatile("s_load_dwordx4 %0, %1, %2 offset:0x40" : "=&s"(psrd) : "s"(karg), "s"(dd * 16) : "memory");
  const unsigned rest = grp0 ? 2 * ts + (kq & 1) + (kq & 2) * 8 : (b >> 3), pG = rest & 7, pR = rest >> 3, pj = pR * 4 + w;
  const unsigned pe = 7 * pj + (dd - 1), pt = 4 + (pe >> 2), sp = pe & 3;
  const bool doPush = dd != 0;
  const unsigned vpd = (doPush ? WS_STG + (pt - 4) * 65536u + pG * 8192 + (sp >> 1) * 1024 + (sp & 1) * 512 : 0x40000000u) + (l >> 5) * 2048 + (l & 31) * 16;
  const unsigned vpx = rowof(pG, w * 16 + (l >> 3)) * KSH + (l & 7) * 16;
  const unsigned vpr = 98304u + ((l >> 5) * 16 + (l & 15)) * 128 + w * 32 + ((l >> 4) & 1) * 16;
  const unsigned spf = doPush ? WS_FLAG + pG * 2048 + pt * 64 + sp * 4 : 0x80000000u;
  const unsigned vrep = l == 0 ? 0u : (doPush && pt < 6 ? WS_REP + (pG * NREP + (l - 1)) * 64 + pe * 4 - spf : 0x40000000u);
  const unsigned sps = doPush ? (pj >> 1) : 0xffu;
  {
    const unsigned n0 = ((b & 7) * 16 + (b >> 4)) * 128, G = ((b >> 3) & 1) * 4 + w;
    const unsigned dq = l >> 3, dc = l & 7;
    const unsigned vdma = dq * NDIM + ((dc ^ (dq >> 1)) << 4) + n0;
    const unsigned vxa = rowof(G, dq) * KSH + ((dc ^ (dq >> 1)) << 4);
    const unsigned vtx = ((b >> 3) * 4 + w) * 2048 + (l & 15) * 128;
    const unsigned sw0 = (srot + w * 32) * NDIM;
    asm volatile(
      "v_mov_b32_e32 v84, %[vdma]\n"
      "v_add_u32_e32 v85, 130048, %[vdma]\n"
      "v_xor_b32_e32 v86, 64, %[vdma]\n"
      "v_add_u32_e32 v86, 260096, v86\n"
      "v_xor_b32_e32 v87, 64, %[vdma]\n"
      "v_add_u32_e32 v87, 390144, v87\n"
      "s_mov_b32 m0, %[sm0]\n"
      "s_waitcnt lgkmcnt(0)\n"
      "buffer_load_dword v62, off, %[asrd], %[sctr] sc1\n"
      "s_sub_u32 s78, %[rank], %[sdd]\n"
      "s_and_b32 s78, s78, 7\n"
      "s_lshl_b32 s78, s78, 18\n"
      "s_lshr_b32 s79, %[rank], 1\n"
      "s_add_u32 s79, s79, %[spr]\n"
      "s_and_b32 s79, s79, 3\n"
      "s_lshl_b32 s79, s79, 7\n"
      "s_add_u32 s78, s78, s79\n"
      "s_add_u32 s79, s78, 31744\n"
      "s_lshr_b32 m0, %[sm0], 1\n"
      "s_add_u32 m0, m0, 98304\n"
      "s_and_b32 s80, %[rank], 1\n"
      "buffer_load_dwordx4 %[vpx], %[xsrd], s78 offen lds\n"
      "buffer_load_dwordx4 %[vpx], %[xsrd], s79 offen offset:1024 lds\n"
      "s_lshl_b32 s80, s80, 6\n"
      "v_xor_b32_e32 v58, s80, %[vpr]\n"
      "s_mov_b32 m0, %[sm0]\n"
      "s_cmp_gt_u32 %[sps], 1\n"
      "s_cbranch_scc1 Lc0240_%=\n"
      "s_waitcnt vmcnt(0)\n"
      "s_barrier\n"
      "ds_read_b128 v[64:67], v58\n"
      "ds_read_b128 v[68:71], v58 offset:4096\n"
      "v_add_u32_e32 v62, 1, v62\n"
      "s_nop 0\n"
      "v_readfirstlane_b32 s68, v62\n"
      "s_and_b32 s81, s68, 1\n"
      "s_mul_i32 s81, s81, 1835008\n"
      "s_add_u32 s82, s81, 4096\n"
      "v_add_u32_e32 v62, -1, v62\n"
      "s_waitcnt lgkmcnt(0)\n"
      "s_cmp_lg_u32 %[sps], 0\n"
      "s_cbranch_scc1 Lc0240_%=\n"
      "buffer_store_dwordx4 v[64:67], %[vpd], %[psrd], s81 offen sc0 sc1\n"
      "buffer_store_dwordx4 v[68:71], %[vpd], %[psrd], s82 offen sc0 sc1\n"
      "Lc0240_%=:\n"
      "s_lshl_b32 s71, %[rank], 18\n"
      "s_mov_b64 exec, 0xffff\n"
      "buffer_load_dword v57, %[vtx], %[xsrd], s71 offen\n"
      "s_mov_b64 exec, -1\n"
      "s_lshl_b32 s66, %[rank], 23\n"
      "s_add_u32 s66, s66, %[sw0]\n"
      "buffer_load_dwordx4 v84, %[wsrd], s66 offen offset:0  lds\n"
      "buffer_load_dwordx4 v85, %[wsrd], s66 offen offset:1024  lds\n"
      "buffer_load_dwordx4 v86, %[wsrd], s66 offen offset:2048  lds\n"
      "buffer_load_dwordx4 v87, %[wsrd], s66 offen offset:3072  lds\n"
      "s_lshl_b32 s71, %[rank], 18\n"
      "v_add_u32_e32 v80, s71, %[vxa]\n"
      "v_xor_b32_e32 v81, 64, v80\n"
      "v_add_u32_e32 v81, 31744, v81\n"
      "v_add_u32_e32 v82, 63488, v80\n"
      "v_xor_b32_e32 v83, 64, v80\n"
      "v_add_u32_e32 v83, 95232, v83\n"
      "s_lshl_b32 m0, %[sm0], 1\n"
      "s_add_u32 m0, m0, 131072\n"
      "s_add_u32 s71, %[srot], 0\n"
      "s_and_b32 s71, s71, 0x180\n"
      "buffer_load_dwordx4 v80, %[xsrd], s71 offen offset:0 lds\n"
      "buffer_load_dwordx4 v81, %[xsrd], s71 offen offset:1024 lds\n"
      "buffer_load_dwordx4 v82, %[xsrd], s71 offen offset:2048 lds\n"
      "buffer_load_dwordx4 v83, %[xsrd], s71 offen offset:3072 lds\n"
      "s_lshl_b32 m0, %[sm0], 1\n"
      "s_add_u32 m0, m0, 135168\n"
      "s_add_u32 s71, %[srot], 0\n"
      "s_and_b32 s71, s71, 0x180\n"
      "s_add_u32 s71, s71, 131072\n"
      "buffer_load_dwordx4 v80, %[xsrd], s71 offen offset:0 lds\n"
      "buffer_load_dwordx4 v81, %[xsrd], s71 offen offset:1024 lds\n"
      "buffer_load_dwordx4 v82, %[xsrd], s71 offen offset:2048 lds\n"
      "buffer_load_dwordx4 v83, %[xsrd], s71 offen offset:3072 lds\n"
      "s_cmp_lg_u32 %[sps], 0\n"
      "s_cbranch_scc1 Lq0242_%=\n"
      "s_waitcnt vmcnt(13)\n"
      "v_add_u32_e32 v63, 1, v62\n"
      "s_mov_b64 exec, 511\n"
      "buffer_store_dword v63, %[vrep], %[psrd], %[spf] offen sc0 sc1\n"
      "s_mov_b64 exec, -1\n"
      "Lq0242_%=:\n"
      : "+v"(l)
      : [wsrd] "s"(wsrd), [xsrd] "s"(xsrd), [asrd] "s"(asrd), [psrd] "s"(psrd), [rank] "s"(rank), [sm0] "s"(sm0), [sps] "s"(sps),
        [sctr] "s"(sctr), [srot] "s"(srot), [sdd] "s"(dd), [spr] "s"(pR), [sw0] "s"(sw0), [st0] "s"((unsigned)st0l),
        [vdma] "v"(vdma), [vxa] "v"(vxa), [vpx] "v"(vpx), [vpr] "v"(vpr), [vtx] "v"(vtx), [vrep] "v"(vrep), [spf] "s"(spf), [vpd] "v"(vpd), [vct] "v"(vct)
      : "v54", "v55", "v56", "v57", "v58", "v59", "v60", "v61", "v62", "v63", "v64", "v65", "v66", "v67", "v68", "v69", "v70", "v71", "v72", "v73", "v74", "v75", "v76", "v77", "v78", "v79", "v80", "v81", "v82", "v83", "v84", "v85", "v86", "v87", "v88", "v89", "v90", "v91", "v92", "v93", "v94", "v95", "v96", "v97", "v98", "v99", "v100", "v101", "v102", "v103", "v104", "v105", "v106", "v107", "v108", "v109", "v110", "v111", "v112", "v113", "v114", "v115", "v116", "v117", "v118", "v119", "v120", "v121", "v122", "v123", "v124", "v125", "v126", "v127", "v128", "v129", "v130", "v131", "v132", "v133", "v134", "v135", "v136", "v137", "v138", "v139", "v140", "v141", "v142", "v143", "v144", "v145", "v146", "v147", "v148", "v149", "v150", "v151", "v152", "v153", "v154", "v155", "v156", "v157", "v158", "v159", "v160", "v161", "v162", "v163", "v164", "v165", "v166", "v167", "v168", "v169", "v170", "v171", "v172", "v173", "v174", "v175", "v176", "v177", "v178", "v179", "v180", "v181", "v182", "v183", "v184", "v185", "v186", "v187", "v188", "v189", "v190", "v191", "v192", "v193", "v194", "v195", "v196", "v197", "v198", "v199", "v200", "v201", "v202", "v203", "v204", "v205", "v206", "v207", "v208", "v209", "v210", "v211", "v212", "v213", "v214", "v215", "v216", "v217", "v218", "v219", "v220", "v221", "v222", "v223", "v224", "v225", "v226", "v227", "v228", "v229", "v230", "v231", "v232", "v233", "v234", "v235", "v236", "v237", "v238", "v239", "v240", "v241", "v242", "v243", "v244", "v245", "v246", "v247", "v248", "v249", "v250", "v251", "v252", "v253", "v254", "v255", "a0", "a1", "a2", "a3", "a4", "a5", "a6", "a7", "a8", "a9", "a10", "a11", "a12", "a13", "a14", "a15", "a16", "a17", "a18", "a19", "a20", "a21", "a22", "a23", "a24", "a25", "a26", "a27", "a28", "a29", "a30", "a31", "a32", "a33", "a34", "a35", "a36", "a37", "a38", "a39", "a40", "a41", "a42", "a43", "a44", "a45", "a46", "a47", "a48", "a49", "a50", "a51", "a52", "a53", "a54", "a55", "a56", "a57", "a58", "a59", "a60", "a61", "a62", "a63", "a64", "a65", "a66", "a67", "a68", "a69", "a70", "a71", "a72", "a73", "a74", "a75", "a76", "a77", "a78", "a79", "a80", "a81", "a82", "a83", "a84", "a85", "a86", "a87", "a88", "a89", "a90", "a91", "a92", "a93", "a94", "a95", "a96", "a97", "a98", "a99", "a100", "a101", "a102", "a103", "a104", "a105", "a106", "a107", "a108", "a109", "a110", "a111", "a112", "a113", "a114", "a115", "a116", "a117", "a118", "a119", "a120", "a121", "a122", "a123", "a124", "a125", "a126", "a127", "a128", "a129", "a130", "a131", "a132", "a133", "a134", "a135", "a136", "a137", "a138", "a139", "a140", "a141", "a142", "a143", "a144", "a145", "a146", "a147", "a148", "a149", "a150", "a151", "a152", "a153", "a154", "a155", "a156", "a157", "a158", "a159", "a160", "a161", "a162", "a163", "a164", "a165", "a166", "a167", "a168", "a169", "a170", "a171", "a172", "a173", "a174", "a175", "a176", "a177", "a178", "a179", "a180", "a181", "a182", "a183", "a184", "a185", "a186", "a187", "a188", "a189", "a190", "a191", "s56", "s57", "s58", "s59", "s60", "s61", "s62", "s63", "s64", "s65", "s66", "s67", "s68", "s69", "s70", "s71", "s72", "s73", "s74", "s75", "s76", "s77", "s78", "s79", "s80", "s81", "s82", "s83", "s84", "s85", "s86", "s87", "s88", "s89", "s90", "s91", "vcc", "scc", "memory");
    asm volatile("" : "+s"(w), "+s"(b));
  }
  const unsigned xcd = b & 7, idx = b >> 3, mt = idx & 1, nt = xcd * 16 + (idx >> 1);
  const unsigned n0 = nt * 128;
  const unsigned G = mt * 4 + w;
  const unsigned g = l >> 4, q = (l & 15) >> 1, pp = l & 1;
  const unsigned vrd = (16 * g + q) * 128 + (((q >> 1) | ((g & 1) << 2)) << 4) + 8 * pp + (unsigned)(size_t)smem;
  unsigned tw;
  if (l < 2) tw = l;
  else if (l < 9) tw = ((mt ? 0x6974523u : 0x7865432u) >> (4 * (l - 2))) & 15u;
  else tw = (l - 5) + ((((l - 5) ^ mt) & 1u) ? 4u : 6u);
  if (l == 31) tw = 31 - mt;
  const unsigned e = 4 * (tw - 4) + w, ej = e / 7, ed = e - 7 * ej;
  const unsigned vte = tw < 4 ? (((tw * 4 + (srot >> 5) + w) & 15u) << 8) : ((1 + ed) | (PHS << 4) | ((ej & 15u) << 8));
  const unsigned vtl = (tw < 32 ? 0u : 0x80000000u) | ((tw & 7u) * 16384u + sm0);
  const unsigned vta = (l >= 4 && l < 32) ? WS_STG + (l - 4) * 65536u : 0x80000000u;
  const unsigned va0 = G * 8192 + l * 16;
  const unsigned var = 65536u + w * 8192 + (l & 15) * 128 + ((g ^ ((l & 15) >> 1)) << 4);
  const unsigned vfl = l < 28 ? WS_FLAG + G * 2048 + (l + 4) * 64 : 0x80000000u;
  const unsigned sfo = WS_FLAG + G * 2048 + 6 * 64, sfl = WS_FLAG + G * 2048 + 30 * 64;
  const unsigned vout = (rowof(G, l >> 3) * NDIM + n0) * 4 + (l & 7) * 16;
  const unsigned vew = 131072u + w * 8192 + (l & 15) * 128 + ((g ^ (l & 7)) << 4);
  const unsigned ver = 131072u + w * 8192 + (l >> 3) * 128 + (((l & 7) ^ (l >> 3)) << 4);
  const unsigned srep = WS_REP + G * NREP * 64;
  const unsigned stmo = WS_TMO, sdbg = WS_DBG + (b * 8 + w) * 512;
  asm volatile(
      "s_nop 1\n"
      "v_mov_b32_e32 v72, 0\n"
      "v_xor_b32_e32 v88, 0, %[vrd]\n"
      "v_xor_b32_e32 v89, 16, %[vrd]\n"
      "v_xor_b32_e32 v90, 32, %[vrd]\n"
      "v_xor_b32_e32 v91, 48, %[vrd]\n"
      "v_xor_b32_e32 v92, 64, %[vrd]\n"
      "v_xor_b32_e32 v93, 80, %[vrd]\n"
      "v_xor_b32_e32 v94, 96, %[vrd]\n"
      "v_xor_b32_e32 v95, 112, %[vrd]\n"
      "v_add_u32_e32 v59, 4096, %[va0]\n"
      "v_mov_b32_e32 v76, %[var]\n"
      "v_xor_b32_e32 v77, 64, %[var]\n"
      "v_add_u32_e32 v78, 0x10000, v76\n"
      "v_add_u32_e32 v79, 0x10000, v77\n"
      "s_mov_b32 s69, 0\n"
      "s_mov_b32 s83, 0\n"
      "s_mov_b32 s70, %[sfo]\n"
      "s_mov_b32 s67, 0xf0000000\n"
      "v_and_b32_e32 v54, 7, %[vte]\n"
      "v_add_u32_e32 v54, %[rank], v54\n"
      "v_and_b32_e32 v54, 7, v54\n"
      "v_bfe_u32 v73, %[vte], 4, 4\n"
      "v_mul_u32_u24_e32 v73, v73, v54\n"
      "v_lshrrev_b32_e32 v74, 8, %[vte]\n"
      "v_and_b32_e32 v75, 12, v73\n"
      "v_add_u32_e32 v75, v75, v74\n"
      "v_and_b32_e32 v75, 12, v75\n"
      "v_add_u32_e32 v73, v73, v74\n"
      "v_and_or_b32 v73, v73, 3, v75\n"
      "v_lshl_add_u32 v54, v54, 4, v73\n"
      "v_lshl_or_b32 v54, v54, 19, %[vtl]\n"
      "s_nop 0\n"
      "s_waitcnt vmcnt(15)\n"
      "v_add_u32_e32 v62, 1, v62\n"
      "s_nop 0\n"
      "v_readfirstlane_b32 s68, v62\n"
      "s_and_b32 s81, s68, 1\n"
      "s_mul_i32 s81, s81, 1835008\n"
      "s_add_u32 s82, s81, 4096\n"
      "v_add_u32_e32 v63, s81, %[vta]\n"
      "s_cmp_eq_u32 %[sps], 1\n"
      "s_cbranch_scc0 Lps243_%=\n"
      "buffer_store_dwordx4 v[64:67], %[vpd], %[psrd], s81 offen sc0 sc1\n"
      "buffer_store_dwordx4 v[68:71], %[vpd], %[psrd], s82 offen sc0 sc1\n"
      "Lps243_%=:\n"
      "v_readlane_b32 s90, v54, 1\n"
      "s_and_b32 s66, s90, 0xfffe0000\n"
      "s_and_b32 m0, s90, 0x1ffff\n"
      "s_nop 0\n"
      "buffer_load_dwordx4 v84, %[wsrd], s66 offen offset:0  lds\n"
      "buffer_load_dwordx4 v85, %[wsrd], s66 offen offset:1024  lds\n"
      "buffer_load_dwordx4 v86, %[wsrd], s66 offen offset:2048  lds\n"
      "buffer_load_dwordx4 v87, %[wsrd], s66 offen offset:3072  lds\n"
      "s_lshl_b32 m0, %[sm0], 1\n"
      "s_add_u32 m0, m0, 65536\n"
      "s_add_u32 s71, %[srot], 128\n"
      "s_and_b32 s71, s71, 0x180\n"
      "buffer_load_dwordx4 v80, %[xsrd], s71 offen offset:0 lds\n"
      "buffer_load_dwordx4 v81, %[xsrd], s71 offen offset:1024 lds\n"
      "buffer_load_dwordx4 v82, %[xsrd], s71 offen offset:2048 lds\n"
      "buffer_load_dwordx4 v83, %[xsrd], s71 offen offset:3072 lds\n"
      "s_lshl_b32 m0, %[sm0], 1\n"
      "s_add_u32 m0, m0, 69632\n"
      "s_add_u32 s71, %[srot], 128\n"
      "s_and_b32 s71, s71, 0x180\n"
      "s_add_u32 s71, s71, 131072\n"
      "buffer_load_dwordx4 v80, %[xsrd], s71 offen offset:0 lds\n"
      "buffer_load_dwordx4 v81, %[xsrd], s71 offen offset:1024 lds\n"
      "buffer_load_dwordx4 v82, %[xsrd], s71 offen offset:2048 lds\n"
      "buffer_load_dwordx4 v83, %[xsrd], s71 offen offset:3072 lds\n"
      "v_readlane_b32 s90, v54, 2\n"
      "s_and_b32 s66, s90, 0xfffe0000\n"
      "s_and_b32 m0, s90, 0x1ffff\n"
      "s_nop 0\n"
      "buffer_load_dwordx4 v84, %[wsrd], s66 offen offset:0  lds\n"
      "buffer_load_dwordx4 v85, %[wsrd], s66 offen offset:1024  lds\n"
      "buffer_load_dwordx4 v86, %[wsrd], s66 offen offset:2048  lds\n"
      "buffer_load_dwordx4 v87, %[wsrd], s66 offen offset:3072  lds\n"
      "s_cmp_eq_u32 %[sps], 1\n"
      "s_cbranch_scc0 Lpb244_%=\n"
      "s_waitcnt vmcnt(16)\n"
      "s_mov_b64 exec, 1\n"
      "buffer_store_dword v62, off, %[psrd], %[spf] sc0 sc1\n"
      "s_mov_b64 exec, -1\n"
      "Lpb244_%=:\n"
      "global_load_dword v61, v72, %[psx]\n"
      "global_load_dword v60, v72, %[psw]\n"
      "s_waitcnt vmcnt(18)\n"
      "ds_read_b128 a[0:3], v78 offset:0\n"
      "ds_read_b128 a[4:7], v79 offset:0\n"
      "ds_read_b128 a[8:11], v78 offset:2048\n"
      "ds_read_b128 a[12:15], v79 offset:2048\n"
      "ds_read_b128 a[16:19], v78 offset:4096\n"
      "ds_read_b128 a[20:23], v79 offset:4096\n"
      "ds_read_b128 a[24:27], v78 offset:6144\n"
      "ds_read_b128 a[28:31], v79 offset:6144\n"
      "s_barrier\n"
      "s_cmp_le_u32 %[sps], 1\n"
      "s_cbranch_scc1 Lc2245_%=\n"
      "ds_read_b128 v[64:67], v58\n"
      "ds_read_b128 v[68:71], v58 offset:4096\n"
      "Lc2245_%=:\n"
      "ds_read_b64_tr_b8 v[96:97], v88 offset:0\n"
      "ds_read_b64_tr_b8 v[98:99], v88 offset:1024\n"
      "ds_read_b64_tr_b8 v[100:101], v88 offset:8192\n"
      "ds_read_b64_tr_b8 v[102:103], v88 offset:9216\n"
      "ds_read_b64_tr_b8 v[104:105], v89 offset:0\n"
      "ds_read_b64_tr_b8 v[106:107], v89 offset:1024\n"
      "ds_read_b64_tr_b8 v[108:109], v89 offset:8192\n"
      "ds_read_b64_tr_b8 v[110:111], v89 offset:9216\n"
      "Ls0_%=:\n"
      "s_waitcnt lgkmcnt(4)\n"
      "v_mfma_f32_16x16x128_f8f6f4 v[128:131], v[96:103], a[0:7], 0 cbsz:1\n"
      "ds_read_b64_tr_b8 v[112:113], v90 offset:0\n"
      "s_getpc_b64 s[74:75]\n"
      "Lct246_%=:\n"
      "s_add_u32 s74, s74, Ls1_%=-Lct246_%=\n"
      "s_addc_u32 s75, s75, 0\n"
      "v_min_u32_e32 v58, Lend_%=-Ls1_%=-4, %[vct]\n"
      "s_mov_b64 exec, 3\n"
      "global_load_dword v57, v58, s[74:75]\n"
      "s_mov_b64 exec, -1\n"
      "v_mfma_f32_16x16x128_f8f6f4 v[132:135], v[96:103], a[8:15], 0 cbsz:1\n"
      "ds_read_b64_tr_b8 v[114:115], v90 offset:1024\n"
      "v_readlane_b32 s90, v54, 3\n"
      "s_and_b32 s66, s90, 0xfffe0000\n"
      "s_and_b32 m0, s90, 0x1ffff\n"
      "s_nop 0\n"
      "buffer_load_dwordx4 v84, %[wsrd], s66 offen offset:0  lds\n"
      "v_mfma_f32_16x16x128_f8f6f4 v[136:139], v[96:103], a[16:23], 0 cbsz:1\n"
      "ds_read_b64_tr_b8 v[116:117], v90 offset:8192\n"
      "v_mfma_f32_16x16x128_f8f6f4 v[140:143], v[96:103], a[24:31], 0 cbsz:1\n"
      "ds_read_b64_tr_b8 v[118:119], v90 offset:9216\n"
      "buffer_load_dwordx4 v85, %[wsrd], s66 offen offset:1024  lds\n"
      "s_waitcnt lgkmcnt(4)\n"
      "v_mfma_f32_16x16x128_f8f6f4 v[156:159], v[104:111], a[24:31], 0 cbsz:1\n"
      "ds_read_b64_tr_b8 v[120:121], v91 offset:0\n"
      "s_cmp_eq_u32 %[sps], 2\n"
      "s_cbranch_scc0 Lps247_%=\n"
      "buffer_store_dwordx4 v[64:67], %[vpd], %[psrd], s81 offen sc0 sc1\n"
      "buffer_store_dwordx4 v[68:71], %[vpd], %[psrd], s82 offen sc0 sc1\n"
      "Lps247_%=:\n"
      "v_mfma_f32_16x16x128_f8f6f4 v[152:155], v[104:111], a[16:23], 0 cbsz:1\n"
      "ds_read_b64_tr_b8 v[122:123], v91 offset:1024\n"
      "buffer_load_dwordx4 v86, %[wsrd], s66 offen offset:2048  lds\n"
      "v_mfma_f32_16x16x128_f8f6f4 v[148:151], v[104:111], a[8:15], 0 cbsz:1\n"
      "ds_read_b64_tr_b8 v[124:125], v91 offset:8192\n"
      "v_mfma_f32_16x16x128_f8f6f4 v[144:147], v[104:111], a[0:7], 0 cbsz:1\n"
      "ds_read_b64_tr_b8 v[126:127], v91 offset:9216\n"
      "buffer_load_dwordx4 v87, %[wsrd], s66 offen offset:3072  lds\n"
      "s_waitcnt lgkmcnt(4)\n"
      "v_mfma_f32_16x16x128_f8f6f4 v[160:163], v[112:119], a[0:7], 0 cbsz:1\n"
      "ds_read_b64_tr_b8 v[96:97], v92 offset:0\n"
      "s_barrier\n"
      "v_mfma_f32_16x16x128_f8f6f4 v[164:167], v[112:119], a[8:15], 0 cbsz:1\n"
      "ds_read_b64_tr_b8 v[98:99], v92 offset:1024\n"
      "s_lshl_b32 m0, %[sm0], 1\n"
      "s_add_u32 m0, m0, 98304\n"
      "s_add_u32 s71, %[srot], 256\n"
      "s_and_b32 s71, s71, 0x180\n"
      "buffer_load_dwordx4 v80, %[xsrd], s71 offen offset:0 lds\n"
      "v_mfma_f32_16x16x128_f8f6f4 v[168:171], v[112:119], a[16:23], 0 cbsz:1\n"
      "ds_read_b64_tr_b8 v[100:101], v92 offset:8192\n"
      "v_mfma_f32_16x16x128_f8f6f4 v[172:175], v[112:119], a[24:31], 0 cbsz:1\n"
      "ds_read_b64_tr_b8 v[102:103], v92 offset:9216\n"
      "buffer_load_dwordx4 v81, %[xsrd], s71 offen offset:1024 lds\n"
      "s_waitcnt lgkmcnt(4)\n"
      "v_mfma_f32_16x16x128_f8f6f4 v[188:191], v[120:127], a[24:31], 0 cbsz:1\n"
      "ds_read_b64_tr_b8 v[104:105], v93 offset:0\n"
      "v_mfma_f32_16x16x128_f8f6f4 v[184:187], v[120:127], a[16:23], 0 cbsz:1\n"
      "ds_read_b64_tr_b8 v[106:107], v93 offset:1024\n"
      "buffer_load_dwordx4 v82, %[xsrd], s71 offen offset:2048 lds\n"
      "v_mfma_f32_16x16x128_f8f6f4 v[180:183], v[120:127], a[8:15], 0 cbsz:1\n"
      "ds_read_b64_tr_b8 v[108:109], v93 offset:8192\n"
      "v_mfma_f32_16x16x128_f8f6f4 v[176:179], v[120:127], a[0:7], 0 cbsz:1\n"
      "ds_read_b64_tr_b8 v[110:111], v93 offset:9216\n"
      "buffer_load_dwordx4 v83, %[xsrd], s71 offen offset:3072 lds\n"
      "s_waitcnt lgkmcnt(4)\n"
      "v_mfma_f32_16x16x128_f8f6f4 v[192:195], v[96:103], a[0:7], 0 cbsz:1\n"
      "ds_read_b64_tr_b8 v[112:113], v94 offset:0\n"
      "v_mfma_f32_16x16x128_f8f6f4 v[196:199], v[96:103], a[8:15], 0 cbsz:1\n"
      "ds_read_b64_tr_b8 v[114:115], v94 offset:1024\n"
      "s_lshl_b32 m0, %[sm0], 1\n"
      "s_add_u32 m0, m0, 102400\n"
      "s_add_u32 s71, %[srot], 256\n"
      "s_and_b32 s71, s71, 0x180\n"
      "s_add_u32 s71, s71, 131072\n"
      "buffer_load_dwordx4 v80, %[xsrd], s71 offen offset:0 lds\n"
      "v_mfma_f32_16x16x128_f8f6f4 v[200:203], v[96:103], a[16:23], 0 cbsz:1\n"
      "ds_read_b64_tr_b8 v[116:117], v94 offset:8192\n"
      "v_mfma_f32_16x16x128_f8f6f4 v[204:207], v[96:103], a[24:31], 0 cbsz:1\n"
      "ds_read_b64_tr_b8 v[118:119], v94 offset:9216\n"
      "buffer_load_dwordx4 v81, %[xsrd], s71 offen offset:1024 lds\n"
      "s_waitcnt lgkmcnt(4)\n"
      "v_mfma_f32_16x16x128_f8f6f4 v[220:223], v[104:111], a[24:31], 0 cbsz:1\n"
      "ds_read_b64_tr_b8 v[120:121], v95 offset:0\n"
      "v_mfma_f32_16x16x128_f8f6f4 v[216:219], v[104:111], a[16:23], 0 cbsz:1\n"
      "ds_read_b64_tr_b8 v[122:123], v95 offset:1024\n"
      "buffer_load_dwordx4 v82, %[xsrd], s71 offen offset:2048 lds\n"
      "v_mfma_f32_16x16x128_f8f6f4 v[212:215], v[104:111], a[8:15], 0 cbsz:1\n"
      "ds_read_b64_tr_b8 v[124:125], v95 offset:8192\n"
      "s_waitcnt vmcnt(18)\n"
      "s_barrier\n"
      "ds_read_b128 a[32:35], v76 offset:0\n"
      "ds_read_b128 a[36:39], v77 offset:0\n"
      "v_mfma_f32_16x16x128_f8f6f4 v[208:211], v[104:111], a[0:7], 0 cbsz:1\n"
      "ds_read_b64_tr_b8 v[126:127], v95 offset:9216\n"
      "buffer_load_dwordx4 v83, %[xsrd], s71 offen offset:3072 lds\n"
      "ds_read_b128 a[40:43], v76 offset:2048\n"
      "ds_read_b128 a[44:47], v77 offset:2048\n"
      "s_waitcnt lgkmcnt(4)\n"
      "v_mfma_f32_16x16x128_f8f6f4 v[224:227], v[112:119], a[0:7], 0 cbsz:1\n"
      "ds_read_b64_tr_b8 v[96:97], v88 offset:16384\n"
      "ds_read_b128 a[48:51], v76 offset:4096\n"
      "ds_read_b128 a[52:55], v77 offset:4096\n"
      "v_mfma_f32_16x16x128_f8f6f4 v[228:231], v[112:119], a[8:15], 0 cbsz:1\n"
      "ds_read_b64_tr_b8 v[98:99], v88 offset:17408\n"
      "ds_read_b128 a[56:59], v76 offset:6144\n"
      "ds_read_b128 a[60:63], v77 offset:6144\n"
      "v_mfma_f32_16x16x128_f8f6f4 v[232:235], v[112:119], a[16:23], 0 cbsz:1\n"
      "ds_read_b64_tr_b8 v[100:101], v88 offset:24576\n"
      "v_mfma_f32_16x16x128_f8f6f4 v[236:239], v[112:119], a[24:31], 0 cbsz:1\n"
      "ds_read_b64_tr_b8 v[102:103], v88 offset:25600\n"
      "s_waitcnt lgkmcnt(4)\n"
      "v_mfma_f32_16x16x128_f8f6f4 v[252:255], v[120:127], a[24:31], 0 cbsz:1\n"
      "ds_read_b64_tr_b8 v[104:105], v89 offset:16384\n"
      "s_cmp_eq_u32 %[sps], 3\n"
      "s_cbranch_scc0 Lps248_%=\n"
      "buffer_store_dwordx4 v[64:67], %[vpd], %[psrd], s81 offen sc0 sc1\n"
      "buffer_store_dwordx4 v[68:71], %[vpd], %[psrd], s82 offen sc0 sc1\n"
      "Lps248_%=:\n"
      "v_mfma_f32_16x16x128_f8f6f4 v[248:251], v[120:127], a[16:23], 0 cbsz:1\n"
      "ds_read_b64_tr_b8 v[106:107], v89 offset:17408\n"
      "v_mfma_f32_16x16x128_f8f6f4 v[244:247], v[120:127], a[8:15], 0 cbsz:1\n"
      "ds_read_b64_tr_b8 v[108:109], v89 offset:24576\n"
      "v_mfma_f32_16x16x128_f8f6f4 v[240:243], v[120:127], a[0:7], 0 cbsz:1\n"
      "ds_read_b64_tr_b8 v[110:111], v89 offset:25600\n"
      "Ls1_%=:\n"
      "s_cmp_eq_u32 %[smt], 0\n"
      "s_cbranch_scc1 Ltw249_%=\n"
      "s_waitcnt vmcnt(8)\n"
      "Ltw249_%=:\n"
      "s_waitcnt vmcnt(15)\n"
      "s_barrier\n"
      "s_waitcnt lgkmcnt(4)\n"
      "v_mfma_f32_16x16x128_f8f6f4 v[128:131], v[96:103], a[32:39], v[128:131] cbsz:1\n"
      "ds_read_b64_tr_b8 v[112:113], v90 offset:16384\n"
      "v_mfma_f32_16x16x128_f8f6f4 v[132:135], v[96:103], a[40:47], v[132:135] cbsz:1\n"
      "ds_read_b64_tr_b8 v[114:115], v90 offset:17408\n"
      "v_readlane_b32 s90, v54, 4\n"
      "s_and_b32 s66, s90, 0xfffe0000\n"
      "s_and_b32 m0, s90, 0x1ffff\n"
      "s_nop 0\n"
      "buffer_load_dwordx4 v84, %[wsrd], s66 offen offset:0  lds\n"
      "v_mfma_f32_16x16x128_f8f6f4 v[136:139], v[96:103], a[48:55], v[136:139] cbsz:1\n"
      "ds_read_b64_tr_b8 v[116:117], v90 offset:24576\n"
      "v_mfma_f32_16x16x128_f8f6f4 v[140:143], v[96:103], a[56:63], v[140:143] cbsz:1\n"
      "ds_read_b64_tr_b8 v[118:119], v90 offset:25600\n"
      "buffer_load_dwordx4 v85, %[wsrd], s66 offen offset:1024  lds\n"
      "s_waitcnt lgkmcnt(4)\n"
      "v_mfma_f32_16x16x128_f8f6f4 v[156:159], v[104:111], a[56:63], v[156:159] cbsz:1\n"
      "ds_read_b64_tr_b8 v[120:121], v91 offset:16384\n"
      "v_mfma_f32_16x16x128_f8f6f4 v[152:155], v[104:111], a[48:55], v[152:155] cbsz:1\n"
      "ds_read_b64_tr_b8 v[122:123], v91 offset:17408\n"
      "buffer_load_dwordx4 v86, %[wsrd], s66 offen offset:2048  lds\n"
      "v_mfma_f32_16x16x128_f8f6f4 v[148:151], v[104:111], a[40:47], v[148:151] cbsz:1\n"
      "ds_read_b64_tr_b8 v[124:125], v91 offset:24576\n"
      "v_mfma_f32_16x16x128_f8f6f4 v[144:147], v[104:111], a[32:39], v[144:147] cbsz:1\n"
      "ds_read_b64_tr_b8 v[126:127], v91 offset:25600\n"
      "buffer_load_dwordx4 v87, %[wsrd], s66 offen offset:3072  lds\n"
      "s_waitcnt lgkmcnt(4)\n"
      "v_mfma_f32_16x16x128_f8f6f4 v[160:163], v[112:119], a[32:39], v[160:163] cbsz:1\n"
      "ds_read_b64_tr_b8 v[96:97], v92 offset:16384\n"
      "v_mfma_f32_16x16x128_f8f6f4 v[164:167], v[112:119], a[40:47], v[164:167] cbsz:1\n"
      "ds_read_b64_tr_b8 v[98:99], v92 offset:17408\n"
      "s_lshl_b32 m0, %[sm0], 1\n"
      "s_add_u32 m0, m0, 131072\n"
      "s_add_u32 s71, %[srot], 384\n"
      "s_and_b32 s71, s71, 0x180\n"
      "buffer_load_dwordx4 v80, %[xsrd], s71 offen offset:0 lds\n"
      "v_mfma_f32_16x16x128_f8f6f4 v[168:171], v[112:119], a[48:55], v[168:171] cbsz:1\n"
      "ds_read_b64_tr_b8 v[100:101], v92 offset:24576\n"
      "v_mfma_f32_16x16x128_f8f6f4 v[172:175], v[112:119], a[56:63], v[172:175] cbsz:1\n"
      "ds_read_b64_tr_b8 v[102:103], v92 offset:25600\n"
      "buffer_load_dwordx4 v81, %[xsrd], s71 offen offset:1024 lds\n"
      "s_waitcnt lgkmcnt(4)\n"
      "v_mfma_f32_16x16x128_f8f6f4 v[188:191], v[120:127], a[56:63], v[188:191] cbsz:1\n"
      "ds_read_b64_tr_b8 v[104:105], v93 offset:16384\n"
      "s_cmp_eq_u32 %[sps], 2\n"
      "s_cbranch_scc0 Lpb250_%=\n"
      "s_waitcnt vmcnt(16)\n"
      "s_mov_b64 exec, 1\n"
      "buffer_store_dword v62, off, %[psrd], %[spf] sc0 sc1\n"
      "s_mov_b64 exec, -1\n"
      "Lpb250_%=:\n"
      "v_mfma_f32_16x16x128_f8f6f4 v[184:187], v[120:127], a[48:55], v[184:187] cbsz:1\n"
      "ds_read_b64_tr_b8 v[106:107], v93 offset:17408\n"
      "buffer_load_dwordx4 v82, %[xsrd], s71 offen offset:2048 lds\n"
      "v_mfma_f32_16x16x128_f8f6f4 v[180:183], v[120:127], a[40:47], v[180:183] cbsz:1\n"
      "ds_read_b64_tr_b8 v[108:109], v93 offset:24576\n"
      "v_mfma_f32_16x16x128_f8f6f4 v[176:179], v[120:127], a[32:39], v[176:179] cbsz:1\n"
      "ds_read_b64_tr_b8 v[110:111], v93 offset:25600\n"
      "buffer_load_dwordx4 v83, %[xsrd], s71 offen offset:3072 lds\n"
      "s_waitcnt lgkmcnt(4)\n"
      "v_mfma_f32_16x16x128_f8f6f4 v[192:195], v[96:103], a[32:39], v[192:195] cbsz:1\n"
      "ds_read_b64_tr_b8 v[112:113], v94 offset:16384\n"
      "v_mfma_f32_16x16x128_f8f6f4 v[196:199], v[96:103], a[40:47], v[196:199] cbsz:1\n"
      "ds_read_b64_tr_b8 v[114:115], v94 offset:17408\n"
      "s_lshl_b32 m0, %[sm0], 1\n"
      "s_add_u32 m0, m0, 135168\n"
      "s_add_u32 s71, %[srot], 384\n"
      "s_and_b32 s71, s71, 0x180\n"
      "s_add_u32 s71, s71, 131072\n"
      "buffer_load_dwordx4 v80, %[xsrd], s71 offen offset:0 lds\n"
      "v_mfma_f32_16x16x128_f8f6f4 v[200:203], v[96:103], a[48:55], v[200:203] cbsz:1\n"
      "ds_read_b64_tr_b8 v[116:117], v94 offset:24576\n"
      "v_mfma_f32_16x16x128_f8f6f4 v[204:207], v[96:103], a[56:63], v[204:207] cbsz:1\n"
      "ds_read_b64_tr_b8 v[118:119], v94 offset:25600\n"
      "buffer_load_dwordx4 v81, %[xsrd], s71 offen offset:1024 lds\n"
      "s_waitcnt lgkmcnt(4)\n"
      "v_mfma_f32_16x16x128_f8f6f4 v[220:223], v[104:111], a[56:63], v[220:223] cbsz:1\n"
      "ds_read_b64_tr_b8 v[120:121], v95 offset:16384\n"
      "v_mfma_f32_16x16x128_f8f6f4 v[216:219], v[104:111], a[48:55], v[216:219] cbsz:1\n"
      "ds_read_b64_tr_b8 v[122:123], v95 offset:17408\n"
      "buffer_load_dwordx4 v82, %[xsrd], s71 offen offset:2048 lds\n"
      "v_mfma_f32_16x16x128_f8f6f4 v[212:215], v[104:111], a[40:47], v[212:215] cbsz:1\n"
      "ds_read_b64_tr_b8 v[124:125], v95 offset:24576\n"
      "s_waitcnt vmcnt(11)\n"
      "ds_read_b128 a[64:67], v76 offset:32768\n"
      "ds_read_b128 a[68:71], v77 offset:32768\n"
      "v_mfma_f32_16x16x128_f8f6f4 v[208:211], v[104:111], a[32:39], v[208:211] cbsz:1\n"
      "ds_read_b64_tr_b8 v[126:127], v95 offset:25600\n"
      "buffer_load_dwordx4 v83, %[xsrd], s71 offen offset:3072 lds\n"
      "ds_read_b128 a[72:75], v76 offset:34816\n"
      "ds_read_b128 a[76:79], v77 offset:34816\n"
      "s_waitcnt lgkmcnt(4)\n"
      "v_mfma_f32_16x16x128_f8f6f4 v[224:227], v[112:119], a[32:39], v[224:227] cbsz:1\n"
      "ds_read_b64_tr_b8 v[96:97], v88 offset:32768\n"
      "ds_read_b128 a[80:83], v76 offset:36864\n"
      "ds_read_b128 a[84:87], v77 offset:36864\n"
      "v_mfma_f32_16x16x128_f8f6f4 v[228:231], v[112:119], a[40:47], v[228:231] cbsz:1\n"
      "ds_read_b64_tr_b8 v[98:99], v88 offset:33792\n"
      "ds_read_b128 a[88:91], v76 offset:38912\n"
      "ds_read_b128 a[92:95], v77 offset:38912\n"
      "v_mfma_f32_16x16x128_f8f6f4 v[232:235], v[112:119], a[48:55], v[232:235] cbsz:1\n"
      "ds_read_b64_tr_b8 v[100:101], v88 offset:40960\n"
      "v_mfma_f32_16x16x128_f8f6f4 v[236:239], v[112:119], a[56:63], v[236:239] cbsz:1\n"
      "ds_read_b64_tr_b8 v[102:103], v88 offset:41984\n"
      "s_waitcnt lgkmcnt(4)\n"
      "v_mfma_f32_16x16x128_f8f6f4 v[252:255], v[120:127], a[56:63], v[252:255] cbsz:1\n"
      "ds_read_b64_tr_b8 v[104:105], v89 offset:32768\n"
      "s_cmp_eq_u32 %[sps], 4\n"
      "s_cbranch_scc0 Lps251_%=\n"
      "buffer_store_dwordx4 v[64:67], %[vpd], %[psrd], s81 offen sc0 sc1\n"
      "buffer_store_dwordx4 v[68:71], %[vpd], %[psrd], s82 offen sc0 sc1\n"
      "Lps251_%=:\n"
      "v_mfma_f32_16x16x128_f8f6f4 v[248:251], v[120:127], a[48:55], v[248:251] cbsz:1\n"
      "ds_read_b64_tr_b8 v[106:107], v89 offset:33792\n"
      "v_mfma_f32_16x16x128_f8f6f4 v[244:247], v[120:127], a[40:47], v[244:247] cbsz:1\n"
      "ds_read_b64_tr_b8 v[108:109], v89 offset:40960\n"
      "v_mfma_f32_16x16x128_f8f6f4 v[240:243], v[120:127], a[32:39], v[240:243] cbsz:1\n"
      "ds_read_b64_tr_b8 v[110:111], v89 offset:41984\n"
      "Ls2_%=:\n"
      "s_cmp_eq_u32 %[smt], 1\n"
      "s_cbranch_scc1 Ltw252_%=\n"
      "s_waitcnt vmcnt(20)\n"
      "Ltw252_%=:\n"
      "s_waitcnt vmcnt(27)\n"
      "s_barrier\n"
      "s_waitcnt lgkmcnt(4)\n"
      "v_mfma_f32_16x16x128_f8f6f4 v[128:131], v[96:103], a[64:71], v[128:131] cbsz:1\n"
      "ds_read_b64_tr_b8 v[112:113], v90 offset:32768\n"
      "v_readlane_b32 s90, v54, 6\n"
      "s_and_b32 s66, s90, 0xfffe0000\n"
      "s_and_b32 m0, s90, 0x1ffff\n"
      "s_nop 0\n"
      "buffer_load_dwordx4 v84, %[wsrd], s66 offen offset:0  lds\n"
      "v_mfma_f32_16x16x128_f8f6f4 v[132:135], v[96:103], a[72:79], v[132:135] cbsz:1\n"
      "ds_read_b64_tr_b8 v[114:115], v90 offset:33792\n"
      "buffer_load_dwordx4 v85, %[wsrd], s66 offen offset:1024  lds\n"
      "v_mfma_f32_16x16x128_f8f6f4 v[136:139], v[96:103], a[80:87], v[136:139] cbsz:1\n"
      "ds_read_b64_tr_b8 v[116:117], v90 offset:40960\n"
      "buffer_load_dwordx4 v86, %[wsrd], s66 offen offset:2048  lds\n"
      "v_mfma_f32_16x16x128_f8f6f4 v[140:143], v[96:103], a[88:95], v[140:143] cbsz:1\n"
      "ds_read_b64_tr_b8 v[118:119], v90 offset:41984\n"
      "buffer_load_dwordx4 v87, %[wsrd], s66 offen offset:3072  lds\n"
      "s_waitcnt lgkmcnt(4)\n"
      "v_mfma_f32_16x16x128_f8f6f4 v[156:159], v[104:111], a[88:95], v[156:159] cbsz:1\n"
      "ds_read_b64_tr_b8 v[120:121], v91 offset:32768\n"
      "s_mov_b32 s91, %[srep]\n"
      "s_load_dwordx8 s[56:63], %[pfl], s91\n"
      "v_mfma_f32_16x16x128_f8f6f4 v[152:155], v[104:111], a[80:87], v[152:155] cbsz:1\n"
      "ds_read_b64_tr_b8 v[122:123], v91 offset:33792\n"
      "v_readlane_b32 s90, v54, 5\n"
      "s_and_b32 s66, s90, 0xfffe0000\n"
      "s_and_b32 m0, s90, 0x1ffff\n"
      "s_nop 0\n"
      "buffer_load_dwordx4 v84, %[wsrd], s66 offen offset:0  lds\n"
      "v_mfma_f32_16x16x128_f8f6f4 v[148:151], v[104:111], a[72:79], v[148:151] cbsz:1\n"
      "ds_read_b64_tr_b8 v[124:125], v91 offset:40960\n"
      "v_mfma_f32_16x16x128_f8f6f4 v[144:147], v[104:111], a[64:71], v[144:147] cbsz:1\n"
      "ds_read_b64_tr_b8 v[126:127], v91 offset:41984\n"
      "buffer_load_dwordx4 v85, %[wsrd], s66 offen offset:1024  lds\n"
      "s_waitcnt lgkmcnt(4)\n"
      "v_mfma_f32_16x16x128_f8f6f4 v[160:163], v[112:119], a[64:71], v[160:163] cbsz:1\n"
      "ds_read_b64_tr_b8 v[96:97], v92 offset:32768\n"
      "s_cmp_eq_u32 %[sps], 3\n"
      "s_cbranch_scc0 Lpb253_%=\n"
      "s_waitcnt vmcnt(18)\n"
      "s_mov_b64 exec, 1\n"
      "buffer_store_dword v62, off, %[psrd], %[spf] sc0 sc1\n"
      "s_mov_b64 exec, -1\n"
      "Lpb253_%=:\n"
      "v_mfma_f32_16x16x128_f8f6f4 v[164:167], v[112:119], a[72:79], v[164:167] cbsz:1\n"
      "ds_read_b64_tr_b8 v[98:99], v92 offset:33792\n"
      "buffer_load_dwordx4 v86, %[wsrd], s66 offen offset:2048  lds\n"
      "v_mfma_f32_16x16x128_f8f6f4 v[168:171], v[112:119], a[80:87], v[168:171] cbsz:1\n"
      "ds_read_b64_tr_b8 v[100:101], v92 offset:40960\n"
      "v_mfma_f32_16x16x128_f8f6f4 v[172:175], v[112:119], a[88:95], v[172:175] cbsz:1\n"
      "ds_read_b64_tr_b8 v[102:103], v92 offset:41984\n"
      "buffer_load_dwordx4 v87, %[wsrd], s66 offen offset:3072  lds\n"
      "s_waitcnt lgkmcnt(4)\n"
      "v_mfma_f32_16x16x128_f8f6f4 v[188:191], v[120:127], a[88:95], v[188:191] cbsz:1\n"
      "ds_read_b64_tr_b8 v[104:105], v93 offset:32768\n"
      "v_mfma_f32_16x16x128_f8f6f4 v[184:187], v[120:127], a[80:87], v[184:187] cbsz:1\n"
      "ds_read_b64_tr_b8 v[106:107], v93 offset:33792\n"
      "v_mfma_f32_16x16x128_f8f6f4 v[180:183], v[120:127], a[72:79], v[180:183] cbsz:1\n"
      "ds_read_b64_tr_b8 v[108:109], v93 offset:40960\n"
      "s_waitcnt lgkmcnt(0)\n"
      "s_min_u32 s56, s56, s57\n"
      "s_min_u32 s58, s58, s59\n"
      "s_min_u32 s60, s60, s61\n"
      "s_min_u32 s62, s62, s63\n"
      "s_min_u32 s56, s56, s58\n"
      "s_min_u32 s60, s60, s62\n"
      "s_min_u32 s56, s56, s60\n"
      "s_cmp_ge_u32 s56, s68\n"
      "s_cbranch_scc0 Llate_%=\n"
      "s_mov_b32 s89, 12\n"
      "v_readlane_b32 s65, v63, 4\n"
      "s_nop 1\n"
      "v_mfma_f32_16x16x128_f8f6f4 v[176:179], v[120:127], a[64:71], v[176:179] cbsz:1\n"
      "ds_read_b64_tr_b8 v[110:111], v93 offset:41984\n"
      "s_waitcnt lgkmcnt(4)\n"
      "v_mfma_f32_16x16x128_f8f6f4 v[192:195], v[96:103], a[64:71], v[192:195] cbsz:1\n"
      "ds_read_b64_tr_b8 v[112:113], v94 offset:32768\n"
      "buffer_load_dwordx4 a[0:3], %[va0], %[asrd], s65 offen offset:0 sc0 sc1\n"
      "v_mfma_f32_16x16x128_f8f6f4 v[196:199], v[96:103], a[72:79], v[196:199] cbsz:1\n"
      "ds_read_b64_tr_b8 v[114:115], v94 offset:33792\n"
      "v_mfma_f32_16x16x128_f8f6f4 v[200:203], v[96:103], a[80:87], v[200:203] cbsz:1\n"
      "ds_read_b64_tr_b8 v[116:117], v94 offset:40960\n"
      "buffer_load_dwordx4 a[4:7], %[va0], %[asrd], s65 offen offset:1024 sc0 sc1\n"
      "v_mfma_f32_16x16x128_f8f6f4 v[204:207], v[96:103], a[88:95], v[204:207] cbsz:1\n"
      "ds_read_b64_tr_b8 v[118:119], v94 offset:41984\n"
      "s_waitcnt lgkmcnt(4)\n"
      "v_mfma_f32_16x16x128_f8f6f4 v[220:223], v[104:111], a[88:95], v[220:223] cbsz:1\n"
      "ds_read_b64_tr_b8 v[120:121], v95 offset:32768\n"
      "buffer_load_dwordx4 a[8:11], %[va0], %[asrd], s65 offen offset:2048 sc0 sc1\n"
      "v_mfma_f32_16x16x128_f8f6f4 v[216:219], v[104:111], a[80:87], v[216:219] cbsz:1\n"
      "ds_read_b64_tr_b8 v[122:123], v95 offset:33792\n"
      "v_mfma_f32_16x16x128_f8f6f4 v[212:215], v[104:111], a[72:79], v[212:215] cbsz:1\n"
      "ds_read_b64_tr_b8 v[124:125], v95 offset:40960\n"
      "buffer_load_dwordx4 a[12:15], %[va0], %[asrd], s65 offen offset:3072 sc0 sc1\n"
      "s_waitcnt vmcnt(12)\n"
      "ds_read_b128 a[96:99], v78 offset:0\n"
      "ds_read_b128 a[100:103], v79 offset:0\n"
      "v_mfma_f32_16x16x128_f8f6f4 v[208:211], v[104:111], a[64:71], v[208:211] cbsz:1\n"
      "ds_read_b64_tr_b8 v[126:127], v95 offset:41984\n"
      "ds_read_b128 a[104:107], v78 offset:2048\n"
      "ds_read_b128 a[108:111], v79 offset:2048\n"
      "s_waitcnt lgkmcnt(4)\n"
      "v_mfma_f32_16x16x128_f8f6f4 v[224:227], v[112:119], a[64:71], v[224:227] cbsz:1\n"
      "ds_read_b64_tr_b8 v[96:97], v88 offset:49152\n"
      "buffer_load_dwordx4 a[16:19], v59, %[asrd], s65 offen offset:0 sc0 sc1\n"
      "ds_read_b128 a[112:115], v78 offset:4096\n"
      "ds_read_b128 a[116:119], v79 offset:4096\n"
      "v_mfma_f32_16x16x128_f8f6f4 v[228:231], v[112:119], a[72:79], v[228:231] cbsz:1\n"
      "ds_read_b64_tr_b8 v[98:99], v88 offset:50176\n"
      "ds_read_b128 a[120:123], v78 offset:6144\n"
      "ds_read_b128 a[124:127], v79 offset:6144\n"
      "v_mfma_f32_16x16x128_f8f6f4 v[232:235], v[112:119], a[80:87], v[232:235] cbsz:1\n"
      "ds_read_b64_tr_b8 v[100:101], v88 offset:57344\n"
      "buffer_load_dwordx4 a[20:23], v59, %[asrd], s65 offen offset:1024 sc0 sc1\n"
      "v_mfma_f32_16x16x128_f8f6f4 v[236:239], v[112:119], a[88:95], v[236:239] cbsz:1\n"
      "ds_read_b64_tr_b8 v[102:103], v88 offset:58368\n"
      "v_xor_b32_e32 v88, 0x10000, v88\n"
      "s_waitcnt lgkmcnt(4)\n"
      "v_mfma_f32_16x16x128_f8f6f4 v[252:255], v[120:127], a[88:95], v[252:255] cbsz:1\n"
      "ds_read_b64_tr_b8 v[104:105], v89 offset:49152\n"
      "buffer_load_dwordx4 a[24:27], v59, %[asrd], s65 offen offset:2048 sc0 sc1\n"
      "v_mfma_f32_16x16x128_f8f6f4 v[248:251], v[120:127], a[80:87], v[248:251] cbsz:1\n"
      "ds_read_b64_tr_b8 v[106:107], v89 offset:50176\n"
      "v_mfma_f32_16x16x128_f8f6f4 v[244:247], v[120:127], a[72:79], v[244:247] cbsz:1\n"
      "ds_read_b64_tr_b8 v[108:109], v89 offset:57344\n"
      "buffer_load_dwordx4 a[28:31], v59, %[asrd], s65 offen offset:3072 sc0 sc1\n"
      "v_mfma_f32_16x16x128_f8f6f4 v[240:243], v[120:127], a[64:71], v[240:243] cbsz:1\n"
      "ds_read_b64_tr_b8 v[110:111], v89 offset:58368\n"
      "v_xor_b32_e32 v89, 0x10000, v89\n"
      "Ls3_%=:\n"
      "s_cmp_eq_u32 %[smt], 0\n"
      "s_cbranch_scc1 Ltw254_%=\n"
      "s_waitcnt vmcnt(8)\n"
      "Ltw254_%=:\n"
      "s_waitcnt vmcnt(24)\n"
      "s_barrier\n"
      "s_waitcnt lgkmcnt(4)\n"
      "v_mfma_f32_16x16x128_f8f6f4 v[128:131], v[96:103], a[96:103], v[128:131] cbsz:1\n"
      "ds_read_b64_tr_b8 v[112:113], v90 offset:49152\n"
      "v_mfma_f32_16x16x128_f8f6f4 v[132:135], v[96:103], a[104:111], v[132:135] cbsz:1\n"
      "ds_read_b64_tr_b8 v[114:115], v90 offset:50176\n"
      "v_readlane_b32 s90, v54, 7\n"
      "s_and_b32 s66, s90, 0xfffe0000\n"
      "s_and_b32 m0, s90, 0x1ffff\n"
      "s_nop 0\n"
      "buffer_load_dwordx4 v84, %[wsrd], s66 offen offset:0  lds\n"
      "v_mfma_f32_16x16x128_f8f6f4 v[136:139], v[96:103], a[112:119], v[136:139] cbsz:1\n"
      "ds_read_b64_tr_b8 v[116:117], v90 offset:57344\n"
      "v_mfma_f32_16x16x128_f8f6f4 v[140:143], v[96:103], a[120:127], v[140:143] cbsz:1\n"
      "ds_read_b64_tr_b8 v[118:119], v90 offset:58368\n"
      "v_xor_b32_e32 v90, 0x10000, v90\n"
      "buffer_load_dwordx4 v85, %[wsrd], s66 offen offset:1024  lds\n"
      "s_waitcnt lgkmcnt(4)\n"
      "v_mfma_f32_16x16x128_f8f6f4 v[156:159], v[104:111], a[120:127], v[156:159] cbsz:1\n"
      "ds_read_b64_tr_b8 v[120:121], v91 offset:49152\n"
      "v_mfma_f32_16x16x128_f8f6f4 v[152:155], v[104:111], a[112:119], v[152:155] cbsz:1\n"
      "ds_read_b64_tr_b8 v[122:123], v91 offset:50176\n"
      "buffer_load_dwordx4 v86, %[wsrd], s66 offen offset:2048  lds\n"
      "v_mfma_f32_16x16x128_f8f6f4 v[148:151], v[104:111], a[104:111], v[148:151] cbsz:1\n"
      "ds_read_b64_tr_b8 v[124:125], v91 offset:57344\n"
      "v_mfma_f32_16x16x128_f8f6f4 v[144:147], v[104:111], a[96:103], v[144:147] cbsz:1\n"
      "ds_read_b64_tr_b8 v[126:127], v91 offset:58368\n"
      "v_xor_b32_e32 v91, 0x10000, v91\n"
      "buffer_load_dwordx4 v87, %[wsrd], s66 offen offset:3072  lds\n"
      "v_readlane_b32 s65, v63, 5\n"
      "s_nop 1\n"
      "s_waitcnt lgkmcnt(4)\n"
      "v_mfma_f32_16x16x128_f8f6f4 v[160:163], v[112:119], a[96:103], v[160:163] cbsz:1\n"
      "ds_read_b64_tr_b8 v[96:97], v92 offset:49152\n"
      "v_mfma_f32_16x16x128_f8f6f4 v[164:167], v[112:119], a[104:111], v[164:167] cbsz:1\n"
      "ds_read_b64_tr_b8 v[98:99], v92 offset:50176\n"
      "buffer_load_dwordx4 a[32:35], %[va0], %[asrd], s65 offen offset:0 sc0 sc1\n"
      "v_mfma_f32_16x16x128_f8f6f4 v[168:171], v[112:119], a[112:119], v[168:171] cbsz:1\n"
      "ds_read_b64_tr_b8 v[100:101], v92 offset:57344\n"
      "v_mfma_f32_16x16x128_f8f6f4 v[172:175], v[112:119], a[120:127], v[172:175] cbsz:1\n"
      "ds_read_b64_tr_b8 v[102:103], v92 offset:58368\n"
      "v_xor_b32_e32 v92, 0x10000, v92\n"
      "buffer_load_dwordx4 a[36:39], %[va0], %[asrd], s65 offen offset:1024 sc0 sc1\n"
      "s_waitcnt lgkmcnt(4)\n"
      "v_mfma_f32_16x16x128_f8f6f4 v[188:191], v[120:127], a[120:127], v[188:191] cbsz:1\n"
      "ds_read_b64_tr_b8 v[104:105], v93 offset:49152\n"
      "s_cmp_eq_u32 %[sps], 4\n"
      "s_cbranch_scc0 Lpb255_%=\n"
      "s_waitcnt vmcnt(22)\n"
      "s_mov_b64 exec, 1\n"
      "buffer_store_dword v62, off, %[psrd], %[spf] sc0 sc1\n"
      "s_mov_b64 exec, -1\n"
      "Lpb255_%=:\n"
      "v_mfma_f32_16x16x128_f8f6f4 v[184:187], v[120:127], a[112:119], v[184:187] cbsz:1\n"
      "ds_read_b64_tr_b8 v[106:107], v93 offset:50176\n"
      "buffer_load_dwordx4 a[40:43], %[va0], %[asrd], s65 offen offset:2048 sc0 sc1\n"
      "v_mfma_f32_16x16x128_f8f6f4 v[180:183], v[120:127], a[104:111], v[180:183] cbsz:1\n"
      "ds_read_b64_tr_b8 v[108:109], v93 offset:57344\n"
      "v_mfma_f32_16x16x128_f8f6f4 v[176:179], v[120:127], a[96:103], v[176:179] cbsz:1\n"
      "ds_read_b64_tr_b8 v[110:111], v93 offset:58368\n"
      "v_xor_b32_e32 v93, 0x10000, v93\n"
      "buffer_load_dwordx4 a[44:47], %[va0], %[asrd], s65 offen offset:3072 sc0 sc1\n"
      "s_waitcnt lgkmcnt(4)\n"
      "v_mfma_f32_16x16x128_f8f6f4 v[192:195], v[96:103], a[96:103], v[192:195] cbsz:1\n"
      "ds_read_b64_tr_b8 v[112:113], v94 offset:49152\n"
      "v_mfma_f32_16x16x128_f8f6f4 v[196:199], v[96:103], a[104:111], v[196:199] cbsz:1\n"
      "ds_read_b64_tr_b8 v[114:115], v94 offset:50176\n"
      "buffer_load_dwordx4 a[48:51], v59, %[asrd], s65 offen offset:0 sc0 sc1\n"
      "v_mfma_f32_16x16x128_f8f6f4 v[200:203], v[96:103], a[112:119], v[200:203] cbsz:1\n"
      "ds_read_b64_tr_b8 v[116:117], v94 offset:57344\n"
      "v_mfma_f32_16x16x128_f8f6f4 v[204:207], v[96:103], a[120:127], v[204:207] cbsz:1\n"
      "ds_read_b64_tr_b8 v[118:119], v94 offset:58368\n"
      "v_xor_b32_e32 v94, 0x10000, v94\n"
      "buffer_load_dwordx4 a[52:55], v59, %[asrd], s65 offen offset:1024 sc0 sc1\n"
      "s_waitcnt lgkmcnt(4)\n"
      "v_mfma_f32_16x16x128_f8f6f4 v[220:223], v[104:111], a[120:127], v[220:223] cbsz:1\n"
      "ds_read_b64_tr_b8 v[120:121], v95 offset:49152\n"
      "v_mfma_f32_16x16x128_f8f6f4 v[216:219], v[104:111], a[112:119], v[216:219] cbsz:1\n"
      "ds_read_b64_tr_b8 v[122:123], v95 offset:50176\n"
      "buffer_load_dwordx4 a[56:59], v59, %[asrd], s65 offen offset:2048 sc0 sc1\n"
      "v_mfma_f32_16x16x128_f8f6f4 v[212:215], v[104:111], a[104:111], v[212:215] cbsz:1\n"
      "ds_read_b64_tr_b8 v[124:125], v95 offset:57344\n"
      "v_mfma_f32_16x16x128_f8f6f4 v[208:211], v[104:111], a[96:103], v[208:211] cbsz:1\n"
      "ds_read_b64_tr_b8 v[126:127], v95 offset:58368\n"
      "v_xor_b32_e32 v95, 0x10000, v95\n"
      "buffer_load_dwordx4 a[60:63], v59, %[asrd], s65 offen offset:3072 sc0 sc1\n"
      "s_waitcnt lgkmcnt(4)\n"
      "v_mfma_f32_16x16x128_f8f6f4 v[224:227], v[112:119], a[96:103], v[224:227] cbsz:1\n"
      "ds_read_b64_tr_b8 v[96:97], v88 offset:0\n"
      "v_mfma_f32_16x16x128_f8f6f4 v[228:231], v[112:119], a[104:111], v[228:231] cbsz:1\n"
      "ds_read_b64_tr_b8 v[98:99], v88 offset:1024\n"
      "v_readlane_b32 s90, v54, 8\n"
      "s_and_b32 s66, s90, 0xfffe0000\n"
      "s_and_b32 m0, s90, 0x1ffff\n"
      "s_nop 0\n"
      "buffer_load_dwordx4 v84, %[wsrd], s66 offen offset:0  lds\n"
      "v_mfma_f32_16x16x128_f8f6f4 v[232:235], v[112:119], a[112:119], v[232:235] cbsz:1\n"
      "ds_read_b64_tr_b8 v[100:101], v88 offset:8192\n"
      "v_mfma_f32_16x16x128_f8f6f4 v[236:239], v[112:119], a[120:127], v[236:239] cbsz:1\n"
      "ds_read_b64_tr_b8 v[102:103], v88 offset:9216\n"
      "buffer_load_dwordx4 v85, %[wsrd], s66 offen offset:1024  lds\n"
      "s_waitcnt lgkmcnt(4)\n"
      "v_mfma_f32_16x16x128_f8f6f4 v[252:255], v[120:127], a[120:127], v[252:255] cbsz:1\n"
      "ds_read_b64_tr_b8 v[104:105], v89 offset:0\n"
      "v_mfma_f32_16x16x128_f8f6f4 v[248:251], v[120:127], a[112:119], v[248:251] cbsz:1\n"
      "ds_read_b64_tr_b8 v[106:107], v89 offset:1024\n"
      "buffer_load_dwordx4 v86, %[wsrd], s66 offen offset:2048  lds\n"
      "v_mfma_f32_16x16x128_f8f6f4 v[244:247], v[120:127], a[104:111], v[244:247] cbsz:1\n"
      "ds_read_b64_tr_b8 v[108:109], v89 offset:8192\n"
      "s_load_dwordx4 s[56:59], %[pfl], s70\n"
      "s_load_dwordx4 s[60:63], %[pfl], s70 offset:64\n"
      "v_mfma_f32_16x16x128_f8f6f4 v[240:243], v[120:127], a[96:103], v[240:243] cbsz:1\n"
      "ds_read_b64_tr_b8 v[110:111], v89 offset:9216\n"
      "buffer_load_dwordx4 v87, %[wsrd], s66 offen offset:3072  lds\n"
      "Ls4_%=:\n"
      "s_waitcnt vmcnt(16)\n"
      "s_barrier\n"
      "s_waitcnt lgkmcnt(4)\n"
      "v_mfma_f32_16x16x128_f8f6f4 v[128:131], v[96:103], a[0:7], v[128:131] cbsz:1\n"
      "ds_read_b64_tr_b8 v[112:113], v90 offset:0\n"
      "v_mfma_f32_16x16x128_f8f6f4 v[132:135], v[96:103], a[8:15], v[132:135] cbsz:1\n"
      "ds_read_b64_tr_b8 v[114:115], v90 offset:1024\n"
      "v_readlane_b32 s90, v54, 9\n"
      "s_and_b32 s66, s90, 0xfffe0000\n"
      "s_and_b32 m0, s90, 0x1ffff\n"
      "s_nop 0\n"
      "buffer_load_dwordx4 v84, %[wsrd], s66 offen offset:0  lds\n"
      "v_mfma_f32_16x16x128_f8f6f4 v[136:139], v[96:103], a[16:23], v[136:139] cbsz:1\n"
      "ds_read_b64_tr_b8 v[116:117], v90 offset:8192\n"
      "v_mfma_f32_16x16x128_f8f6f4 v[140:143], v[96:103], a[24:31], v[140:143] cbsz:1\n"
      "ds_read_b64_tr_b8 v[118:119], v90 offset:9216\n"
      "buffer_load_dwordx4 v85, %[wsrd], s66 offen offset:1024  lds\n"
      "s_waitcnt lgkmcnt(4)\n"
      "v_mfma_f32_16x16x128_f8f6f4 v[156:159], v[104:111], a[24:31], v[156:159] cbsz:1\n"
      "ds_read_b64_tr_b8 v[120:121], v91 offset:0\n"
      "ds_read_b64_tr_b8 v[122:123], v91 offset:1024\n"
      "v_mfma_f32_16x16x128_f8f6f4 v[152:155], v[104:111], a[16:23], v[152:155] cbsz:1\n"
      "ds_read_b64_tr_b8 v[124:125], v91 offset:8192\n"
      "ds_read_b64_tr_b8 v[126:127], v91 offset:9216\n"
      "buffer_load_dwordx4 v86, %[wsrd], s66 offen offset:2048  lds\n"
      "v_mfma_f32_16x16x128_f8f6f4 v[148:151], v[104:111], a[8:15], v[148:151] cbsz:1\n"
      "v_mfma_f32_16x16x128_f8f6f4 v[144:147], v[104:111], a[0:7], v[144:147] cbsz:1\n"
      "buffer_load_dwordx4 v87, %[wsrd], s66 offen offset:3072  lds\n"
      "s_waitcnt lgkmcnt(0)\n"
      "v_mfma_f32_16x16x128_f8f6f4 v[160:163], v[112:119], a[0:7], v[160:163] cbsz:1\n"
      "ds_read_b64_tr_b8 v[96:97], v92 offset:0\n"
      "s_min_u32 s56, s56, s57\n"
      "s_min_u32 s58, s58, s59\n"
      "s_min_u32 s60, s60, s61\n"
      "s_min_u32 s62, s62, s63\n"
      "v_mfma_f32_16x16x128_f8f6f4 v[164:167], v[112:119], a[8:15], v[164:167] cbsz:1\n"
      "ds_read_b64_tr_b8 v[98:99], v92 offset:1024\n"
      "s_min_u32 s56, s56, s58\n"
      "s_min_u32 s60, s60, s62\n"
      "s_min_u32 s56, s56, s60\n"
      "s_cmp_ge_u32 s56, s68\n"
      "s_cbranch_scc0 Lsl256_%=\n"
      "Lrt257_%=:\n"
      "v_mfma_f32_16x16x128_f8f6f4 v[168:171], v[112:119], a[16:23], v[168:171] cbsz:1\n"
      "ds_read_b64_tr_b8 v[100:101], v92 offset:8192\n"
      "s_add_u32 s70, s70, 128\n"
      "s_min_u32 s70, s70, %[sfl]\n"
      "s_lshl_b32 s89, s89, 2\n"
      "v_readlane_b32 s65, v63, 6\n"
      "s_nop 1\n"
      "v_mfma_f32_16x16x128_f8f6f4 v[172:175], v[112:119], a[24:31], v[172:175] cbsz:1\n"
      "ds_read_b64_tr_b8 v[102:103], v92 offset:9216\n"
      "s_waitcnt lgkmcnt(4)\n"
      "v_mfma_f32_16x16x128_f8f6f4 v[188:191], v[120:127], a[24:31], v[188:191] cbsz:1\n"
      "ds_read_b64_tr_b8 v[104:105], v93 offset:0\n"
      "buffer_load_dwordx4 a[64:67], %[va0], %[asrd], s65 offen offset:0 sc0 sc1\n"
      "v_mfma_f32_16x16x128_f8f6f4 v[184:187], v[120:127], a[16:23], v[184:187] cbsz:1\n"
      "ds_read_b64_tr_b8 v[106:107], v93 offset:1024\n"
      "v_mfma_f32_16x16x128_f8f6f4 v[180:183], v[120:127], a[8:15], v[180:183] cbsz:1\n"
      "ds_read_b64_tr_b8 v[108:109], v93 offset:8192\n"
      "buffer_load_dwordx4 a[68:71], %[va0], %[asrd], s65 offen offset:1024 sc0 sc1\n"
      "v_mfma_f32_16x16x128_f8f6f4 v[176:179], v[120:127], a[0:7], v[176:179] cbsz:1\n"
      "ds_read_b64_tr_b8 v[110:111], v93 offset:9216\n"
      "s_cmp_eq_u32 %[sps], 5\n"
      "s_cbranch_scc0 Lps258_%=\n"
      "buffer_store_dwordx4 v[64:67], %[vpd], %[psrd], s81 offen sc0 sc1\n"
      "buffer_store_dwordx4 v[68:71], %[vpd], %[psrd], s82 offen sc0 sc1\n"
      "Lps258_%=:\n"
      "s_waitcnt lgkmcnt(4)\n"
      "v_mfma_f32_16x16x128_f8f6f4 v[192:195], v[96:103], a[0:7], v[192:195] cbsz:1\n"
      "ds_read_b64_tr_b8 v[112:113], v94 offset:0\n"
      "buffer_load_dwordx4 a[72:75], %[va0], %[asrd], s65 offen offset:2048 sc0 sc1\n"
      "v_mfma_f32_16x16x128_f8f6f4 v[196:199], v[96:103], a[8:15], v[196:199] cbsz:1\n"
      "ds_read_b64_tr_b8 v[114:115], v94 offset:1024\n"
      "v_mfma_f32_16x16x128_f8f6f4 v[200:203], v[96:103], a[16:23], v[200:203] cbsz:1\n"
      "ds_read_b64_tr_b8 v[116:117], v94 offset:8192\n"
      "buffer_load_dwordx4 a[76:79], %[va0], %[asrd], s65 offen offset:3072 sc0 sc1\n"
      "v_mfma_f32_16x16x128_f8f6f4 v[204:207], v[96:103], a[24:31], v[204:207] cbsz:1\n"
      "ds_read_b64_tr_b8 v[118:119], v94 offset:9216\n"
      "s_waitcnt lgkmcnt(4)\n"
      "v_mfma_f32_16x16x128_f8f6f4 v[220:223], v[104:111], a[24:31], v[220:223] cbsz:1\n"
      "ds_read_b64_tr_b8 v[120:121], v95 offset:0\n"
      "buffer_load_dwordx4 a[80:83], v59, %[asrd], s65 offen offset:0 sc0 sc1\n"
      "v_mfma_f32_16x16x128_f8f6f4 v[216:219], v[104:111], a[16:23], v[216:219] cbsz:1\n"
      "ds_read_b64_tr_b8 v[122:123], v95 offset:1024\n"
      "v_mfma_f32_16x16x128_f8f6f4 v[212:215], v[104:111], a[8:15], v[212:215] cbsz:1\n"
      "ds_read_b64_tr_b8 v[124:125], v95 offset:8192\n"
      "buffer_load_dwordx4 a[84:87], v59, %[asrd], s65 offen offset:1024 sc0 sc1\n"
      "v_mfma_f32_16x16x128_f8f6f4 v[208:211], v[104:111], a[0:7], v[208:211] cbsz:1\n"
      "ds_read_b64_tr_b8 v[126:127], v95 offset:9216\n"
      "s_waitcnt lgkmcnt(4)\n"
      "v_mfma_f32_16x16x128_f8f6f4 v[224:227], v[112:119], a[0:7], v[224:227] cbsz:1\n"
      "ds_read_b64_tr_b8 v[96:97], v88 offset:16384\n"
      "buffer_load_dwordx4 a[88:91], v59, %[asrd], s65 offen offset:2048 sc0 sc1\n"
      "v_mfma_f32_16x16x128_f8f6f4 v[228:231], v[112:119], a[8:15], v[228:231] cbsz:1\n"
      "ds_read_b64_tr_b8 v[98:99], v88 offset:17408\n"
      "v_mfma_f32_16x16x128_f8f6f4 v[232:235], v[112:119], a[16:23], v[232:235] cbsz:1\n"
      "ds_read_b64_tr_b8 v[100:101], v88 offset:24576\n"
      "buffer_load_dwordx4 a[92:95], v59, %[asrd], s65 offen offset:3072 sc0 sc1\n"
      "v_mfma_f32_16x16x128_f8f6f4 v[236:239], v[112:119], a[24:31], v[236:239] cbsz:1\n"
      "ds_read_b64_tr_b8 v[102:103], v88 offset:25600\n"
      "v_readlane_b32 s65, v63, 7\n"
      "s_nop 1\n"
      "s_waitcnt lgkmcnt(4)\n"
      "v_mfma_f32_16x16x128_f8f6f4 v[252:255], v[120:127], a[24:31], v[252:255] cbsz:1\n"
      "ds_read_b64_tr_b8 v[104:105], v89 offset:16384\n"
      "v_mfma_f32_16x16x128_f8f6f4 v[248:251], v[120:127], a[16:23], v[248:251] cbsz:1\n"
      "ds_read_b64_tr_b8 v[106:107], v89 offset:17408\n"
      "buffer_load_dwordx4 a[96:99], %[va0], %[asrd], s65 offen offset:0 sc0 sc1\n"
      "v_mfma_f32_16x16x128_f8f6f4 v[244:247], v[120:127], a[8:15], v[244:247] cbsz:1\n"
      "ds_read_b64_tr_b8 v[108:109], v89 offset:24576\n"
      "v_mfma_f32_16x16x128_f8f6f4 v[240:243], v[120:127], a[0:7], v[240:243] cbsz:1\n"
      "ds_read_b64_tr_b8 v[110:111], v89 offset:25600\n"
      "buffer_load_dwordx4 a[100:103], %[va0], %[asrd], s65 offen offset:1024 sc0 sc1\n"
      "Ls5_%=:\n"
      "s_cmp_eq_u32 %[smt], 0\n"
      "s_cbranch_scc1 Ltw259_%=\n"
      "s_waitcnt vmcnt(14)\n"
      "Ltw259_%=:\n"
      "s_waitcnt vmcnt(18)\n"
      "s_barrier\n"
      "s_waitcnt lgkmcnt(4)\n"
      "v_mfma_f32_16x16x128_f8f6f4 v[128:131], v[96:103], a[32:39], v[128:131] cbsz:1\n"
      "ds_read_b64_tr_b8 v[112:113], v90 offset:16384\n"
      "v_mfma_f32_16x16x128_f8f6f4 v[132:135], v[96:103], a[40:47], v[132:135] cbsz:1\n"
      "ds_read_b64_tr_b8 v[114:115], v90 offset:17408\n"
      "v_readlane_b32 s90, v54, 10\n"
      "s_and_b32 s66, s90, 0xfffe0000\n"
      "s_and_b32 m0, s90, 0x1ffff\n"
      "s_nop 0\n"
      "buffer_load_dwordx4 v84, %[wsrd], s66 offen offset:0  lds\n"
      "v_mfma_f32_16x16x128_f8f6f4 v[136:139], v[96:103], a[48:55], v[136:139] cbsz:1\n"
      "ds_read_b64_tr_b8 v[116:117], v90 offset:24576\n"
      "v_mfma_f32_16x16x128_f8f6f4 v[140:143], v[96:103], a[56:63], v[140:143] cbsz:1\n"
      "ds_read_b64_tr_b8 v[118:119], v90 offset:25600\n"
      "buffer_load_dwordx4 v85, %[wsrd], s66 offen offset:1024  lds\n"
      "s_waitcnt lgkmcnt(4)\n"
      "v_mfma_f32_16x16x128_f8f6f4 v[156:159], v[104:111], a[56:63], v[156:159] cbsz:1\n"
      "ds_read_b64_tr_b8 v[120:121], v91 offset:16384\n"
      "s_load_dwordx4 s[56:59], %[pfl], s70\n"
      "s_load_dwordx4 s[60:63], %[pfl], s70 offset:64\n"
      "v_mfma_f32_16x16x128_f8f6f4 v[152:155], v[104:111], a[48:55], v[152:155] cbsz:1\n"
      "ds_read_b64_tr_b8 v[122:123], v91 offset:17408\n"
      "buffer_load_dwordx4 v86, %[wsrd], s66 offen offset:2048  lds\n"
      "v_mfma_f32_16x16x128_f8f6f4 v[148:151], v[104:111], a[40:47], v[148:151] cbsz:1\n"
      "ds_read_b64_tr_b8 v[124:125], v91 offset:24576\n"
      "v_mfma_f32_16x16x128_f8f6f4 v[144:147], v[104:111], a[32:39], v[144:147] cbsz:1\n"
      "ds_read_b64_tr_b8 v[126:127], v91 offset:25600\n"
      "buffer_load_dwordx4 v87, %[wsrd], s66 offen offset:3072  lds\n"
      "s_waitcnt lgkmcnt(4)\n"
      "v_mfma_f32_16x16x128_f8f6f4 v[160:163], v[112:119], a[32:39], v[160:163] cbsz:1\n"
      "ds_read_b64_tr_b8 v[96:97], v92 offset:16384\n"
      "v_mfma_f32_16x16x128_f8f6f4 v[164:167], v[112:119], a[40:47], v[164:167] cbsz:1\n"
      "ds_read_b64_tr_b8 v[98:99], v92 offset:17408\n"
      "buffer_load_dwordx4 a[104:107], %[va0], %[asrd], s65 offen offset:2048 sc0 sc1\n"
      "v_mfma_f32_16x16x128_f8f6f4 v[168:171], v[112:119], a[48:55], v[168:171] cbsz:1\n"
      "ds_read_b64_tr_b8 v[100:101], v92 offset:24576\n"
      "v_mfma_f32_16x16x128_f8f6f4 v[172:175], v[112:119], a[56:63], v[172:175] cbsz:1\n"
      "ds_read_b64_tr_b8 v[102:103], v92 offset:25600\n"
      "buffer_load_dwordx4 a[108:111], %[va0], %[asrd], s65 offen offset:3072 sc0 sc1\n"
      "s_waitcnt lgkmcnt(4)\n"
      "v_mfma_f32_16x16x128_f8f6f4 v[188:191], v[120:127], a[56:63], v[188:191] cbsz:1\n"
      "ds_read_b64_tr_b8 v[104:105], v93 offset:16384\n"
      "v_mfma_f32_16x16x128_f8f6f4 v[184:187], v[120:127], a[48:55], v[184:187] cbsz:1\n"
      "ds_read_b64_tr_b8 v[106:107], v93 offset:17408\n"
      "buffer_load_dwordx4 a[112:115], v59, %[asrd], s65 offen offset:0 sc0 sc1\n"
      "v_mfma_f32_16x16x128_f8f6f4 v[180:183], v[120:127], a[40:47], v[180:183] cbsz:1\n"
      "ds_read_b64_tr_b8 v[108:109], v93 offset:24576\n"
      "s_cmp_eq_u32 %[sps], 6\n"
      "s_cbranch_scc0 Lps260_%=\n"
      "buffer_store_dwordx4 v[64:67], %[vpd], %[psrd], s81 offen sc0 sc1\n"
      "buffer_store_dwordx4 v[68:71], %[vpd], %[psrd], s82 offen sc0 sc1\n"
      "Lps260_%=:\n"
      "v_mfma_f32_16x16x128_f8f6f4 v[176:179], v[120:127], a[32:39], v[176:179] cbsz:1\n"
      "ds_read_b64_tr_b8 v[110:111], v93 offset:25600\n"
      "buffer_load_dwordx4 a[116:119], v59, %[asrd], s65 offen offset:1024 sc0 sc1\n"
      "s_waitcnt lgkmcnt(4)\n"
      "v_mfma_f32_16x16x128_f8f6f4 v[192:195], v[96:103], a[32:39], v[192:195] cbsz:1\n"
      "ds_read_b64_tr_b8 v[112:113], v94 offset:16384\n"
      "ds_read_b64_tr_b8 v[114:115], v94 offset:17408\n"
      "v_mfma_f32_16x16x128_f8f6f4 v[196:199], v[96:103], a[40:47], v[196:199] cbsz:1\n"
      "ds_read_b64_tr_b8 v[116:117], v94 offset:24576\n"
      "ds_read_b64_tr_b8 v[118:119], v94 offset:25600\n"
      "buffer_load_dwordx4 a[120:123], v59, %[asrd], s65 offen offset:2048 sc0 sc1\n"
      "v_mfma_f32_16x16x128_f8f6f4 v[200:203], v[96:103], a[48:55], v[200:203] cbsz:1\n"
      "v_mfma_f32_16x16x128_f8f6f4 v[204:207], v[96:103], a[56:63], v[204:207] cbsz:1\n"
      "buffer_load_dwordx4 a[124:127], v59, %[asrd], s65 offen offset:3072 sc0 sc1\n"
      "s_waitcnt lgkmcnt(0)\n"
      "v_mfma_f32_16x16x128_f8f6f4 v[220:223], v[104:111], a[56:63], v[220:223] cbsz:1\n"
      "ds_read_b64_tr_b8 v[120:121], v95 offset:16384\n"
      "s_min_u32 s56, s56, s57\n"
      "s_min_u32 s58, s58, s59\n"
      "s_min_u32 s60, s60, s61\n"
      "s_min_u32 s62, s62, s63\n"
      "v_mfma_f32_16x16x128_f8f6f4 v[216:219], v[104:111], a[48:55], v[216:219] cbsz:1\n"
      "ds_read_b64_tr_b8 v[122:123], v95 offset:17408\n"
      "s_min_u32 s56, s56, s58\n"
      "s_min_u32 s60, s60, s62\n"
      "s_min_u32 s56, s56, s60\n"
      "s_cmp_ge_u32 s56, s68\n"
      "s_cbranch_scc0 Lsl261_%=\n"
      "Lrt262_%=:\n"
      "v_mfma_f32_16x16x128_f8f6f4 v[212:215], v[104:111], a[40:47], v[212:215] cbsz:1\n"
      "ds_read_b64_tr_b8 v[124:125], v95 offset:24576\n"
      "s_add_u32 s70, s70, 128\n"
      "s_min_u32 s70, s70, %[sfl]\n"
      "s_lshl_b32 s89, s89, 2\n"
      "s_load_dwordx4 s[56:59], %[pfl], s70\n"
      "s_load_dwordx4 s[60:63], %[pfl], s70 offset:64\n"
      "v_readlane_b32 s65, v63, 8\n"
      "s_nop 1\n"
      "v_mfma_f32_16x16x128_f8f6f4 v[208:211], v[104:111], a[32:39], v[208:211] cbsz:1\n"
      "ds_read_b64_tr_b8 v[126:127], v95 offset:25600\n"
      "s_waitcnt lgkmcnt(4)\n"
      "v_mfma_f32_16x16x128_f8f6f4 v[224:227], v[112:119], a[32:39], v[224:227] cbsz:1\n"
      "ds_read_b64_tr_b8 v[96:97], v88 offset:32768\n"
      "buffer_load_dwordx4 a[0:3], %[va0], %[asrd], s65 offen offset:0 sc0 sc1\n"
      "v_mfma_f32_16x16x128_f8f6f4 v[228:231], v[112:119], a[40:47], v[228:231] cbsz:1\n"
      "ds_read_b64_tr_b8 v[98:99], v88 offset:33792\n"
      "v_mfma_f32_16x16x128_f8f6f4 v[232:235], v[112:119], a[48:55], v[232:235] cbsz:1\n"
      "ds_read_b64_tr_b8 v[100:101], v88 offset:40960\n"
      "buffer_load_dwordx4 a[4:7], %[va0], %[asrd], s65 offen offset:1024 sc0 sc1\n"
      "v_mfma_f32_16x16x128_f8f6f4 v[236:239], v[112:119], a[56:63], v[236:239] cbsz:1\n"
      "ds_read_b64_tr_b8 v[102:103], v88 offset:41984\n"
      "s_waitcnt lgkmcnt(4)\n"
      "v_mfma_f32_16x16x128_f8f6f4 v[252:255], v[120:127], a[56:63], v[252:255] cbsz:1\n"
      "ds_read_b64_tr_b8 v[104:105], v89 offset:32768\n"
      "buffer_load_dwordx4 a[8:11], %[va0], %[asrd], s65 offen offset:2048 sc0 sc1\n"
      "v_mfma_f32_16x16x128_f8f6f4 v[248:251], v[120:127], a[48:55], v[248:251] cbsz:1\n"
      "ds_read_b64_tr_b8 v[106:107], v89 offset:33792\n"
      "v_mfma_f32_16x16x128_f8f6f4 v[244:247], v[120:127], a[40:47], v[244:247] cbsz:1\n"
      "ds_read_b64_tr_b8 v[108:109], v89 offset:40960\n"
      "buffer_load_dwordx4 a[12:15], %[va0], %[asrd], s65 offen offset:3072 sc0 sc1\n"
      "v_mfma_f32_16x16x128_f8f6f4 v[240:243], v[120:127], a[32:39], v[240:243] cbsz:1\n"
      "ds_read_b64_tr_b8 v[110:111], v89 offset:41984\n"
      "Ls6_%=:\n"
      "s_waitcnt vmcnt(16)\n"
      "s_barrier\n"
      "s_waitcnt lgkmcnt(4)\n"
      "v_mfma_f32_16x16x128_f8f6f4 v[128:131], v[96:103], a[64:71], v[128:131] cbsz:1\n"
      "ds_read_b64_tr_b8 v[112:113], v90 offset:32768\n"
      "v_mfma_f32_16x16x128_f8f6f4 v[132:135], v[96:103], a[72:79], v[132:135] cbsz:1\n"
      "ds_read_b64_tr_b8 v[114:115], v90 offset:33792\n"
      "v_readlane_b32 s90, v54, 11\n"
      "s_and_b32 s66, s90, 0xfffe0000\n"
      "s_and_b32 m0, s90, 0x1ffff\n"
      "s_nop 0\n"
      "buffer_load_dwordx4 v84, %[wsrd], s66 offen offset:0  lds\n"
      "v_mfma_f32_16x16x128_f8f6f4 v[136:139], v[96:103], a[80:87], v[136:139] cbsz:1\n"
      "ds_read_b64_tr_b8 v[116:117], v90 offset:40960\n"
      "v_mfma_f32_16x16x128_f8f6f4 v[140:143], v[96:103], a[88:95], v[140:143] cbsz:1\n"
      "ds_read_b64_tr_b8 v[118:119], v90 offset:41984\n"
      "buffer_load_dwordx4 v85, %[wsrd], s66 offen offset:1024  lds\n"
      "s_waitcnt lgkmcnt(4)\n"
      "v_mfma_f32_16x16x128_f8f6f4 v[156:159], v[104:111], a[88:95], v[156:159] cbsz:1\n"
      "ds_read_b64_tr_b8 v[120:121], v91 offset:32768\n"
      "v_mfma_f32_16x16x128_f8f6f4 v[152:155], v[104:111], a[80:87], v[152:155] cbsz:1\n"
      "ds_read_b64_tr_b8 v[122:123], v91 offset:33792\n"
      "buffer_load_dwordx4 v86, %[wsrd], s66 offen offset:2048  lds\n"
      "v_mfma_f32_16x16x128_f8f6f4 v[148:151], v[104:111], a[72:79], v[148:151] cbsz:1\n"
      "ds_read_b64_tr_b8 v[124:125], v91 offset:40960\n"
      "v_mfma_f32_16x16x128_f8f6f4 v[144:147], v[104:111], a[64:71], v[144:147] cbsz:1\n"
      "ds_read_b64_tr_b8 v[126:127], v91 offset:41984\n"
      "buffer_load_dwordx4 v87, %[wsrd], s66 offen offset:3072  lds\n"
      "s_waitcnt lgkmcnt(4)\n"
      "v_mfma_f32_16x16x128_f8f6f4 v[160:163], v[112:119], a[64:71], v[160:163] cbsz:1\n"
      "ds_read_b64_tr_b8 v[96:97], v92 offset:32768\n"
      "v_mfma_f32_16x16x128_f8f6f4 v[164:167], v[112:119], a[72:79], v[164:167] cbsz:1\n"
      "ds_read_b64_tr_b8 v[98:99], v92 offset:33792\n"
      "buffer_load_dwordx4 a[16:19], v59, %[asrd], s65 offen offset:0 sc0 sc1\n"
      "v_mfma_f32_16x16x128_f8f6f4 v[168:171], v[112:119], a[80:87], v[168:171] cbsz:1\n"
      "ds_read_b64_tr_b8 v[100:101], v92 offset:40960\n"
      "v_mfma_f32_16x16x128_f8f6f4 v[172:175], v[112:119], a[88:95], v[172:175] cbsz:1\n"
      "ds_read_b64_tr_b8 v[102:103], v92 offset:41984\n"
      "buffer_load_dwordx4 a[20:23], v59, %[asrd], s65 offen offset:1024 sc0 sc1\n"
      "s_waitcnt lgkmcnt(4)\n"
      "v_mfma_f32_16x16x128_f8f6f4 v[188:191], v[120:127], a[88:95], v[188:191] cbsz:1\n"
      "ds_read_b64_tr_b8 v[104:105], v93 offset:32768\n"
      "s_cmp_eq_u32 %[sps], 5\n"
      "s_cbranch_scc0 Lpb263_%=\n"
      "s_waitcnt vmcnt(28)\n"
      "s_mov_b64 exec, 1\n"
      "buffer_store_dword v62, off, %[psrd], %[spf] sc0 sc1\n"
      "s_mov_b64 exec, -1\n"
      "Lpb263_%=:\n"
      "s_cmp_eq_u32 %[sps], 7\n"
      "s_cbranch_scc0 Lps264_%=\n"
      "buffer_store_dwordx4 v[64:67], %[vpd], %[psrd], s81 offen sc0 sc1\n"
      "buffer_store_dwordx4 v[68:71], %[vpd], %[psrd], s82 offen sc0 sc1\n"
      "Lps264_%=:\n"
      "v_mfma_f32_16x16x128_f8f6f4 v[184:187], v[120:127], a[80:87], v[184:187] cbsz:1\n"
      "ds_read_b64_tr_b8 v[106:107], v93 offset:33792\n"
      "buffer_load_dwordx4 a[24:27], v59, %[asrd], s65 offen offset:2048 sc0 sc1\n"
      "v_mfma_f32_16x16x128_f8f6f4 v[180:183], v[120:127], a[72:79], v[180:183] cbsz:1\n"
      "ds_read_b64_tr_b8 v[108:109], v93 offset:40960\n"
      "v_mfma_f32_16x16x128_f8f6f4 v[176:179], v[120:127], a[64:71], v[176:179] cbsz:1\n"
      "ds_read_b64_tr_b8 v[110:111], v93 offset:41984\n"
      "buffer_load_dwordx4 a[28:31], v59, %[asrd], s65 offen offset:3072 sc0 sc1\n"
      "s_waitcnt lgkmcnt(4)\n"
      "v_mfma_f32_16x16x128_f8f6f4 v[192:195], v[96:103], a[64:71], v[192:195] cbsz:1\n"
      "ds_read_b64_tr_b8 v[112:113], v94 offset:32768\n"
      "v_mfma_f32_16x16x128_f8f6f4 v[196:199], v[96:103], a[72:79], v[196:199] cbsz:1\n"
      "ds_read_b64_tr_b8 v[114:115], v94 offset:33792\n"
      "v_mfma_f32_16x16x128_f8f6f4 v[200:203], v[96:103], a[80:87], v[200:203] cbsz:1\n"
      "ds_read_b64_tr_b8 v[116:117], v94 offset:40960\n"
      "v_readlane_b32 s65, v63, 9\n"
      "s_nop 1\n"
      "v_mfma_f32_16x16x128_f8f6f4 v[204:207], v[96:103], a[88:95], v[204:207] cbsz:1\n"
      "ds_read_b64_tr_b8 v[118:119], v94 offset:41984\n"
      "s_waitcnt lgkmcnt(4)\n"
      "v_mfma_f32_16x16x128_f8f6f4 v[220:223], v[104:111], a[88:95], v[220:223] cbsz:1\n"
      "ds_read_b64_tr_b8 v[120:121], v95 offset:32768\n"
      "buffer_load_dwordx4 a[32:35], %[va0], %[asrd], s65 offen offset:0 sc0 sc1\n"
      "v_mfma_f32_16x16x128_f8f6f4 v[216:219], v[104:111], a[80:87], v[216:219] cbsz:1\n"
      "ds_read_b64_tr_b8 v[122:123], v95 offset:33792\n"
      "v_mfma_f32_16x16x128_f8f6f4 v[212:215], v[104:111], a[72:79], v[212:215] cbsz:1\n"
      "ds_read_b64_tr_b8 v[124:125], v95 offset:40960\n"
      "buffer_load_dwordx4 a[36:39], %[va0], %[asrd], s65 offen offset:1024 sc0 sc1\n"
      "v_mfma_f32_16x16x128_f8f6f4 v[208:211], v[104:111], a[64:71], v[208:211] cbsz:1\n"
      "ds_read_b64_tr_b8 v[126:127], v95 offset:41984\n"
      "s_waitcnt lgkmcnt(4)\n"
      "v_mfma_f32_16x16x128_f8f6f4 v[224:227], v[112:119], a[64:71], v[224:227] cbsz:1\n"
      "ds_read_b64_tr_b8 v[96:97], v88 offset:49152\n"
      "buffer_load_dwordx4 a[40:43], %[va0], %[asrd], s65 offen offset:2048 sc0 sc1\n"
      "v_mfma_f32_16x16x128_f8f6f4 v[228:231], v[112:119], a[72:79], v[228:231] cbsz:1\n"
      "ds_read_b64_tr_b8 v[98:99], v88 offset:50176\n"
      "v_mfma_f32_16x16x128_f8f6f4 v[232:235], v[112:119], a[80:87], v[232:235] cbsz:1\n"
      "ds_read_b64_tr_b8 v[100:101], v88 offset:57344\n"
      "buffer_load_dwordx4 a[44:47], %[va0], %[asrd], s65 offen offset:3072 sc0 sc1\n"
      "v_mfma_f32_16x16x128_f8f6f4 v[236:239], v[112:119], a[88:95], v[236:239] cbsz:1\n"
      "ds_read_b64_tr_b8 v[102:103], v88 offset:58368\n"
      "v_xor_b32_e32 v88, 0x10000, v88\n"
      "s_waitcnt lgkmcnt(4)\n"
      "v_mfma_f32_16x16x128_f8f6f4 v[252:255], v[120:127], a[88:95], v[252:255] cbsz:1\n"
      "ds_read_b64_tr_b8 v[104:105], v89 offset:49152\n"
      "ds_read_b64_tr_b8 v[106:107], v89 offset:50176\n"
      "buffer_load_dwordx4 a[48:51], v59, %[asrd], s65 offen offset:0 sc0 sc1\n"
      "v_mfma_f32_16x16x128_f8f6f4 v[248:251], v[120:127], a[80:87], v[248:251] cbsz:1\n"
      "ds_read_b64_tr_b8 v[108:109], v89 offset:57344\n"
      "ds_read_b64_tr_b8 v[110:111], v89 offset:58368\n"
      "v_xor_b32_e32 v89, 0x10000, v89\n"
      "v_mfma_f32_16x16x128_f8f6f4 v[244:247], v[120:127], a[72:79], v[244:247] cbsz:1\n"
      "buffer_load_dwordx4 a[52:55], v59, %[asrd], s65 offen offset:1024 sc0 sc1\n"
      "v_mfma_f32_16x16x128_f8f6f4 v[240:243], v[120:127], a[64:71], v[240:243] cbsz:1\n"
      "Ls7_%=:\n"
      "s_waitcnt vmcnt(18)\n"
      "s_barrier\n"
      "s_waitcnt lgkmcnt(0)\n"
      "v_mfma_f32_16x16x128_f8f6f4 v[128:131], v[96:103], a[96:103], v[128:131] cbsz:1\n"
      "ds_read_b64_tr_b8 v[112:113], v90 offset:49152\n"
      "s_min_u32 s56, s56, s57\n"
      "s_min_u32 s58, s58, s59\n"
      "s_min_u32 s60, s60, s61\n"
      "s_min_u32 s62, s62, s63\n"
      "v_mfma_f32_16x16x128_f8f6f4 v[132:135], v[96:103], a[104:111], v[132:135] cbsz:1\n"
      "ds_read_b64_tr_b8 v[114:115], v90 offset:50176\n"
      "s_min_u32 s56, s56, s58\n"
      "s_min_u32 s60, s60, s62\n"
      "s_min_u32 s56, s56, s60\n"
      "s_cmp_ge_u32 s56, s68\n"
      "s_cbranch_scc0 Lsl265_%=\n"
      "Lrt266_%=:\n"
      "v_readlane_b32 s90, v54, 12\n"
      "s_and_b32 s66, s90, 0xfffe0000\n"
      "s_and_b32 m0, s90, 0x1ffff\n"
      "s_nop 0\n"
      "buffer_load_dwordx4 v84, %[wsrd], s66 offen offset:0  lds\n"
      "v_mfma_f32_16x16x128_f8f6f4 v[136:139], v[96:103], a[112:119], v[136:139] cbsz:1\n"
      "ds_read_b64_tr_b8 v[116:117], v90 offset:57344\n"
      "s_add_u32 s70, s70, 128\n"
      "s_min_u32 s70, s70, %[sfl]\n"
      "s_lshl_b32 s89, s89, 2\n"
      "s_load_dwordx4 s[56:59], %[pfl], s70\n"
      "s_load_dwordx4 s[60:63], %[pfl], s70 offset:64\n"
      "v_mfma_f32_16x16x128_f8f6f4 v[140:143], v[96:103], a[120:127], v[140:143] cbsz:1\n"
      "ds_read_b64_tr_b8 v[118:119], v90 offset:58368\n"
      "v_xor_b32_e32 v90, 0x10000, v90\n"
      "buffer_load_dwordx4 v85, %[wsrd], s66 offen offset:1024  lds\n"
      "s_waitcnt lgkmcnt(4)\n"
      "v_mfma_f32_16x16x128_f8f6f4 v[156:159], v[104:111], a[120:127], v[156:159] cbsz:1\n"
      "ds_read_b64_tr_b8 v[120:121], v91 offset:49152\n"
      "v_mfma_f32_16x16x128_f8f6f4 v[152:155], v[104:111], a[112:119], v[152:155] cbsz:1\n"
      "ds_read_b64_tr_b8 v[122:123], v91 offset:50176\n"
      "buffer_load_dwordx4 v86, %[wsrd], s66 offen offset:2048  lds\n"
      "v_mfma_f32_16x16x128_f8f6f4 v[148:151], v[104:111], a[104:111], v[148:151] cbsz:1\n"
      "ds_read_b64_tr_b8 v[124:125], v91 offset:57344\n"
      "v_mfma_f32_16x16x128_f8f6f4 v[144:147], v[104:111], a[96:103], v[144:147] cbsz:1\n"
      "ds_read_b64_tr_b8 v[126:127], v91 offset:58368\n"
      "v_xor_b32_e32 v91, 0x10000, v91\n"
      "buffer_load_dwordx4 v87, %[wsrd], s66 offen offset:3072  lds\n"
      "s_waitcnt lgkmcnt(4)\n"
      "v_mfma_f32_16x16x128_f8f6f4 v[160:163], v[112:119], a[96:103], v[160:163] cbsz:1\n"
      "ds_read_b64_tr_b8 v[96:97], v92 offset:49152\n"
      "v_mfma_f32_16x16x128_f8f6f4 v[164:167], v[112:119], a[104:111], v[164:167] cbsz:1\n"
      "ds_read_b64_tr_b8 v[98:99], v92 offset:50176\n"
      "buffer_load_dwordx4 a[56:59], v59, %[asrd], s65 offen offset:2048 sc0 sc1\n"
      "v_mfma_f32_16x16x128_f8f6f4 v[168:171], v[112:119], a[112:119], v[168:171] cbsz:1\n"
      "ds_read_b64_tr_b8 v[100:101], v92 offset:57344\n"
      "v_mfma_f32_16x16x128_f8f6f4 v[172:175], v[112:119], a[120:127], v[172:175] cbsz:1\n"
      "ds_read_b64_tr_b8 v[102:103], v92 offset:58368\n"
      "v_xor_b32_e32 v92, 0x10000, v92\n"
      "buffer_load_dwordx4 a[60:63], v59, %[asrd], s65 offen offset:3072 sc0 sc1\n"
      "s_waitcnt lgkmcnt(4)\n"
      "v_mfma_f32_16x16x128_f8f6f4 v[188:191], v[120:127], a[120:127], v[188:191] cbsz:1\n"
      "ds_read_b64_tr_b8 v[104:105], v93 offset:49152\n"
      "v_readlane_b32 s65, v63, 10\n"
      "s_nop 1\n"
      "v_mfma_f32_16x16x128_f8f6f4 v[184:187], v[120:127], a[112:119], v[184:187] cbsz:1\n"
      "ds_read_b64_tr_b8 v[106:107], v93 offset:50176\n"
      "v_mfma_f32_16x16x128_f8f6f4 v[180:183], v[120:127], a[104:111], v[180:183] cbsz:1\n"
      "ds_read_b64_tr_b8 v[108:109], v93 offset:57344\n"
      "buffer_load_dwordx4 a[64:67], %[va0], %[asrd], s65 offen offset:0 sc0 sc1\n"
      "v_mfma_f32_16x16x128_f8f6f4 v[176:179], v[120:127], a[96:103], v[176:179] cbsz:1\n"
      "ds_read_b64_tr_b8 v[110:111], v93 offset:58368\n"
      "v_xor_b32_e32 v93, 0x10000, v93\n"
      "s_waitcnt lgkmcnt(4)\n"
      "v_mfma_f32_16x16x128_f8f6f4 v[192:195], v[96:103], a[96:103], v[192:195] cbsz:1\n"
      "ds_read_b64_tr_b8 v[112:113], v94 offset:49152\n"
      "buffer_load_dwordx4 a[68:71], %[va0], %[asrd], s65 offen offset:1024 sc0 sc1\n"
      "v_mfma_f32_16x16x128_f8f6f4 v[196:199], v[96:103], a[104:111], v[196:199] cbsz:1\n"
      "ds_read_b64_tr_b8 v[114:115], v94 offset:50176\n"
      "s_cmp_eq_u32 %[sps], 6\n"
      "s_cbranch_scc0 Lpb267_%=\n"
      "s_waitcnt vmcnt(29)\n"
      "s_mov_b64 exec, 1\n"
      "buffer_store_dword v62, off, %[psrd], %[spf] sc0 sc1\n"
      "s_mov_b64 exec, -1\n"
      "Lpb267_%=:\n"
      "v_mfma_f32_16x16x128_f8f6f4 v[200:203], v[96:103], a[112:119], v[200:203] cbsz:1\n"
      "ds_read_b64_tr_b8 v[116:117], v94 offset:57344\n"
      "buffer_load_dwordx4 a[72:75], %[va0], %[asrd], s65 offen offset:2048 sc0 sc1\n"
      "v_mfma_f32_16x16x128_f8f6f4 v[204:207], v[96:103], a[120:127], v[204:207] cbsz:1\n"
      "ds_read_b64_tr_b8 v[118:119], v94 offset:58368\n"
      "v_xor_b32_e32 v94, 0x10000, v94\n"
      "s_waitcnt lgkmcnt(4)\n"
      "v_mfma_f32_16x16x128_f8f6f4 v[220:223], v[104:111], a[120:127], v[220:223] cbsz:1\n"
      "ds_read_b64_tr_b8 v[120:121], v95 offset:49152\n"
      "buffer_load_dwordx4 a[76:79], %[va0], %[asrd], s65 offen offset:3072 sc0 sc1\n"
      "v_mfma_f32_16x16x128_f8f6f4 v[216:219], v[104:111], a[112:119], v[216:219] cbsz:1\n"
      "ds_read_b64_tr_b8 v[122:123], v95 offset:50176\n"
      "v_mfma_f32_16x16x128_f8f6f4 v[212:215], v[104:111], a[104:111], v[212:215] cbsz:1\n"
      "ds_read_b64_tr_b8 v[124:125], v95 offset:57344\n"
      "buffer_load_dwordx4 a[80:83], v59, %[asrd], s65 offen offset:0 sc0 sc1\n"
      "v_mfma_f32_16x16x128_f8f6f4 v[208:211], v[104:111], a[96:103], v[208:211] cbsz:1\n"
      "ds_read_b64_tr_b8 v[126:127], v95 offset:58368\n"
      "v_xor_b32_e32 v95, 0x10000, v95\n"
      "s_waitcnt lgkmcnt(4)\n"
      "v_mfma_f32_16x16x128_f8f6f4 v[224:227], v[112:119], a[96:103], v[224:227] cbsz:1\n"
      "ds_read_b64_tr_b8 v[96:97], v88 offset:0\n"
      "buffer_load_dwordx4 a[84:87], v59, %[asrd], s65 offen offset:1024 sc0 sc1\n"
      "v_mfma_f32_16x16x128_f8f6f4 v[228:231], v[112:119], a[104:111], v[228:231] cbsz:1\n"
      "ds_read_b64_tr_b8 v[98:99], v88 offset:1024\n"
      "v_mfma_f32_16x16x128_f8f6f4 v[232:235], v[112:119], a[112:119], v[232:235] cbsz:1\n"
      "ds_read_b64_tr_b8 v[100:101], v88 offset:8192\n"
      "buffer_load_dwordx4 a[88:91], v59, %[asrd], s65 offen offset:2048 sc0 sc1\n"
      "v_mfma_f32_16x16x128_f8f6f4 v[236:239], v[112:119], a[120:127], v[236:239] cbsz:1\n"
      "ds_read_b64_tr_b8 v[102:103], v88 offset:9216\n"
      "s_waitcnt lgkmcnt(4)\n"
      "v_mfma_f32_16x16x128_f8f6f4 v[252:255], v[120:127], a[120:127], v[252:255] cbsz:1\n"
      "ds_read_b64_tr_b8 v[104:105], v89 offset:0\n"
      "buffer_load_dwordx4 a[92:95], v59, %[asrd], s65 offen offset:3072 sc0 sc1\n"
      "v_mfma_f32_16x16x128_f8f6f4 v[248:251], v[120:127], a[112:119], v[248:251] cbsz:1\n"
      "ds_read_b64_tr_b8 v[106:107], v89 offset:1024\n"
      "v_mfma_f32_16x16x128_f8f6f4 v[244:247], v[120:127], a[104:111], v[244:247] cbsz:1\n"
      "ds_read_b64_tr_b8 v[108:109], v89 offset:8192\n"
      "v_mfma_f32_16x16x128_f8f6f4 v[240:243], v[120:127], a[96:103], v[240:243] cbsz:1\n"
      "ds_read_b64_tr_b8 v[110:111], v89 offset:9216\n"
      "s_cmp_eq_u32 %[sps], 7\n"
      "s_cbranch_scc0 Lpb268_%=\n"
      "s_waitcnt vmcnt(22)\n"
      "s_mov_b64 exec, 1\n"
      "buffer_store_dword v62, off, %[psrd], %[spf] sc0 sc1\n"
      "s_mov_b64 exec, -1\n"
      "Lpb268_%=:\n"
      "s_mov_b32 s64, 8\n"
      "Ljoin_%=:\n"
      "s_cmp_eq_u32 %[smt], 1\n"
      "s_cbranch_scc1 Lloop1_%=\n"
      ".p2align 6\n"
      "Lloop0_%=:\n"
      "s_waitcnt vmcnt(20)\n"
      "s_barrier\n"
      "s_waitcnt lgkmcnt(4)\n"
      "v_mfma_f32_16x16x128_f8f6f4 v[128:131], v[96:103], a[0:7], v[128:131] cbsz:1\n"
      "ds_read_b64_tr_b8 v[112:113], v90 offset:0\n"
      "v_mfma_f32_16x16x128_f8f6f4 v[132:135], v[96:103], a[8:15], v[132:135] cbsz:1\n"
      "ds_read_b64_tr_b8 v[114:115], v90 offset:1024\n"
      "s_add_u32 s71, s64, 5\n"
      "v_readlane_b32 s90, v54, s71\n"
      "s_and_b32 s66, s90, 0xfffe0000\n"
      "s_and_b32 m0, s90, 0x1ffff\n"
      "s_nop 0\n"
      "v_mfma_f32_16x16x128_f8f6f4 v[136:139], v[96:103], a[16:23], v[136:139] cbsz:1\n"
      "ds_read_b64_tr_b8 v[116:117], v90 offset:8192\n"
      "buffer_load_dwordx4 v84, %[wsrd], s66 offen offset:0  lds\n"
      "v_mfma_f32_16x16x128_f8f6f4 v[140:143], v[96:103], a[24:31], v[140:143] cbsz:1\n"
      "ds_read_b64_tr_b8 v[118:119], v90 offset:9216\n"
      "s_waitcnt lgkmcnt(4)\n"
      "v_mfma_f32_16x16x128_f8f6f4 v[156:159], v[104:111], a[24:31], v[156:159] cbsz:1\n"
      "ds_read_b64_tr_b8 v[120:121], v91 offset:0\n"
      "buffer_load_dwordx4 v85, %[wsrd], s66 offen offset:1024  lds\n"
      "v_mfma_f32_16x16x128_f8f6f4 v[152:155], v[104:111], a[16:23], v[152:155] cbsz:1\n"
      "ds_read_b64_tr_b8 v[122:123], v91 offset:1024\n"
      "v_mfma_f32_16x16x128_f8f6f4 v[148:151], v[104:111], a[8:15], v[148:151] cbsz:1\n"
      "ds_read_b64_tr_b8 v[124:125], v91 offset:8192\n"
      "buffer_load_dwordx4 v86, %[wsrd], s66 offen offset:2048  lds\n"
      "v_mfma_f32_16x16x128_f8f6f4 v[144:147], v[104:111], a[0:7], v[144:147] cbsz:1\n"
      "ds_read_b64_tr_b8 v[126:127], v91 offset:9216\n"
      "s_waitcnt lgkmcnt(4)\n"
      "v_mfma_f32_16x16x128_f8f6f4 v[160:163], v[112:119], a[0:7], v[160:163] cbsz:1\n"
      "ds_read_b64_tr_b8 v[96:97], v92 offset:0\n"
      "buffer_load_dwordx4 v87, %[wsrd], s66 offen offset:3072  lds\n"
      "v_mfma_f32_16x16x128_f8f6f4 v[164:167], v[112:119], a[8:15], v[164:167] cbsz:1\n"
      "ds_read_b64_tr_b8 v[98:99], v92 offset:1024\n"
      "v_mfma_f32_16x16x128_f8f6f4 v[168:171], v[112:119], a[16:23], v[168:171] cbsz:1\n"
      "ds_read_b64_tr_b8 v[100:101], v92 offset:8192\n"
      "v_mfma_f32_16x16x128_f8f6f4 v[172:175], v[112:119], a[24:31], v[172:175] cbsz:1\n"
      "ds_read_b64_tr_b8 v[102:103], v92 offset:9216\n"
      "s_waitcnt lgkmcnt(4)\n"
      "v_mfma_f32_16x16x128_f8f6f4 v[188:191], v[120:127], a[24:31], v[188:191] cbsz:1\n"
      "ds_read_b64_tr_b8 v[104:105], v93 offset:0\n"
      "v_mfma_f32_16x16x128_f8f6f4 v[184:187], v[120:127], a[16:23], v[184:187] cbsz:1\n"
      "ds_read_b64_tr_b8 v[106:107], v93 offset:1024\n"
      "v_mfma_f32_16x16x128_f8f6f4 v[180:183], v[120:127], a[8:15], v[180:183] cbsz:1\n"
      "ds_read_b64_tr_b8 v[108:109], v93 offset:8192\n"
      "v_mfma_f32_16x16x128_f8f6f4 v[176:179], v[120:127], a[0:7], v[176:179] cbsz:1\n"
      "ds_read_b64_tr_b8 v[110:111], v93 offset:9216\n"
      "s_waitcnt lgkmcnt(4)\n"
      "v_mfma_f32_16x16x128_f8f6f4 v[192:195], v[96:103], a[0:7], v[192:195] cbsz:1\n"
      "ds_read_b64_tr_b8 v[112:113], v94 offset:0\n"
      "v_mfma_f32_16x16x128_f8f6f4 v[196:199], v[96:103], a[8:15], v[196:199] cbsz:1\n"
      "ds_read_b64_tr_b8 v[114:115], v94 offset:1024\n"
      "v_mfma_f32_16x16x128_f8f6f4 v[200:203], v[96:103], a[16:23], v[200:203] cbsz:1\n"
      "ds_read_b64_tr_b8 v[116:117], v94 offset:8192\n"
      "v_mfma_f32_16x16x128_f8f6f4 v[204:207], v[96:103], a[24:31], v[204:207] cbsz:1\n"
      "ds_read_b64_tr_b8 v[118:119], v94 offset:9216\n"
      "s_waitcnt lgkmcnt(4)\n"
      "v_mfma_f32_16x16x128_f8f6f4 v[220:223], v[104:111], a[24:31], v[220:223] cbsz:1\n"
      "ds_read_b64_tr_b8 v[120:121], v95 offset:0\n"
      "v_mfma_f32_16x16x128_f8f6f4 v[216:219], v[104:111], a[16:23], v[216:219] cbsz:1\n"
      "ds_read_b64_tr_b8 v[122:123], v95 offset:1024\n"
      "v_mfma_f32_16x16x128_f8f6f4 v[212:215], v[104:111], a[8:15], v[212:215] cbsz:1\n"
      "ds_read_b64_tr_b8 v[124:125], v95 offset:8192\n"
      "s_add_u32 s71, s64, 3\n"
      "v_readlane_b32 s65, v63, s71\n"
      "s_nop 1\n"
      "v_mfma_f32_16x16x128_f8f6f4 v[208:211], v[104:111], a[0:7], v[208:211] cbsz:1\n"
      "ds_read_b64_tr_b8 v[126:127], v95 offset:9216\n"
      "s_waitcnt lgkmcnt(4)\n"
      "v_mfma_f32_16x16x128_f8f6f4 v[224:227], v[112:119], a[0:7], v[224:227] cbsz:1\n"
      "ds_read_b64_tr_b8 v[96:97], v88 offset:16384\n"
      "buffer_load_dwordx4 a[96:99], %[va0], %[asrd], s65 offen offset:0 sc0 sc1\n"
      "v_mfma_f32_16x16x128_f8f6f4 v[228:231], v[112:119], a[8:15], v[228:231] cbsz:1\n"
      "ds_read_b64_tr_b8 v[98:99], v88 offset:17408\n"
      "v_mfma_f32_16x16x128_f8f6f4 v[232:235], v[112:119], a[16:23], v[232:235] cbsz:1\n"
      "ds_read_b64_tr_b8 v[100:101], v88 offset:24576\n"
      "buffer_load_dwordx4 a[100:103], %[va0], %[asrd], s65 offen offset:1024 sc0 sc1\n"
      "v_mfma_f32_16x16x128_f8f6f4 v[236:239], v[112:119], a[24:31], v[236:239] cbsz:1\n"
      "ds_read_b64_tr_b8 v[102:103], v88 offset:25600\n"
      "s_waitcnt lgkmcnt(4)\n"
      "v_mfma_f32_16x16x128_f8f6f4 v[252:255], v[120:127], a[24:31], v[252:255] cbsz:1\n"
      "ds_read_b64_tr_b8 v[104:105], v89 offset:16384\n"
      "ds_read_b64_tr_b8 v[106:107], v89 offset:17408\n"
      "buffer_load_dwordx4 a[104:107], %[va0], %[asrd], s65 offen offset:2048 sc0 sc1\n"
      "v_mfma_f32_16x16x128_f8f6f4 v[248:251], v[120:127], a[16:23], v[248:251] cbsz:1\n"
      "ds_read_b64_tr_b8 v[108:109], v89 offset:24576\n"
      "ds_read_b64_tr_b8 v[110:111], v89 offset:25600\n"
      "v_mfma_f32_16x16x128_f8f6f4 v[244:247], v[120:127], a[8:15], v[244:247] cbsz:1\n"
      "buffer_load_dwordx4 a[108:111], %[va0], %[asrd], s65 offen offset:3072 sc0 sc1\n"
      "v_mfma_f32_16x16x128_f8f6f4 v[240:243], v[120:127], a[0:7], v[240:243] cbsz:1\n"
      "s_add_u32 s64, s64, 1\n"
      "s_waitcnt vmcnt(16)\n"
      "s_barrier\n"
      "s_waitcnt lgkmcnt(0)\n"
      "v_mfma_f32_16x16x128_f8f6f4 v[128:131], v[96:103], a[32:39], v[128:131] cbsz:1\n"
      "ds_read_b64_tr_b8 v[112:113], v90 offset:16384\n"
      "s_min_u32 s56, s56, s57\n"
      "s_min_u32 s58, s58, s59\n"
      "s_min_u32 s60, s60, s61\n"
      "s_min_u32 s62, s62, s63\n"
      "buffer_load_dwordx4 a[112:115], v59, %[asrd], s65 offen offset:0 sc0 sc1\n"
      "v_mfma_f32_16x16x128_f8f6f4 v[132:135], v[96:103], a[40:47], v[132:135] cbsz:1\n"
      "ds_read_b64_tr_b8 v[114:115], v90 offset:17408\n"
      "s_min_u32 s56, s56, s58\n"
      "s_min_u32 s60, s60, s62\n"
      "s_min_u32 s56, s56, s60\n"
      "s_cmp_ge_u32 s56, s68\n"
      "s_cbranch_scc0 Lsl269_%=\n"
      "Lrt270_%=:\n"
      "v_mfma_f32_16x16x128_f8f6f4 v[136:139], v[96:103], a[48:55], v[136:139] cbsz:1\n"
      "ds_read_b64_tr_b8 v[116:117], v90 offset:24576\n"
      "s_add_u32 s70, s70, 128\n"
      "s_min_u32 s70, s70, %[sfl]\n"
      "s_lshl_b32 s89, s89, 2\n"
      "s_load_dwordx4 s[56:59], %[pfl], s70\n"
      "s_load_dwordx4 s[60:63], %[pfl], s70 offset:64\n"
      "buffer_load_dwordx4 a[116:119], v59, %[asrd], s65 offen offset:1024 sc0 sc1\n"
      "v_mfma_f32_16x16x128_f8f6f4 v[140:143], v[96:103], a[56:63], v[140:143] cbsz:1\n"
      "ds_read_b64_tr_b8 v[118:119], v90 offset:25600\n"
      "s_waitcnt lgkmcnt(4)\n"
      "v_mfma_f32_16x16x128_f8f6f4 v[156:159], v[104:111], a[56:63], v[156:159] cbsz:1\n"
      "ds_read_b64_tr_b8 v[120:121], v91 offset:16384\n"
      "buffer_load_dwordx4 a[120:123], v59, %[asrd], s65 offen offset:2048 sc0 sc1\n"
      "v_mfma_f32_16x16x128_f8f6f4 v[152:155], v[104:111], a[48:55], v[152:155] cbsz:1\n"
      "ds_read_b64_tr_b8 v[122:123], v91 offset:17408\n"
      "v_mfma_f32_16x16x128_f8f6f4 v[148:151], v[104:111], a[40:47], v[148:151] cbsz:1\n"
      "ds_read_b64_tr_b8 v[124:125], v91 offset:24576\n"
      "buffer_load_dwordx4 a[124:127], v59, %[asrd], s65 offen offset:3072 sc0 sc1\n"
      "v_mfma_f32_16x16x128_f8f6f4 v[144:147], v[104:111], a[32:39], v[144:147] cbsz:1\n"
      "ds_read_b64_tr_b8 v[126:127], v91 offset:25600\n"
      "s_add_u32 s71, s64, 3\n"
      "v_readlane_b32 s65, v63, s71\n"
      "s_nop 1\n"
      "s_waitcnt lgkmcnt(4)\n"
      "v_mfma_f32_16x16x128_f8f6f4 v[160:163], v[112:119], a[32:39], v[160:163] cbsz:1\n"
      "ds_read_b64_tr_b8 v[96:97], v92 offset:16384\n"
      "v_mfma_f32_16x16x128_f8f6f4 v[164:167], v[112:119], a[40:47], v[164:167] cbsz:1\n"
      "ds_read_b64_tr_b8 v[98:99], v92 offset:17408\n"
      "buffer_load_dwordx4 a[0:3], %[va0], %[asrd], s65 offen offset:0 sc0 sc1\n"
      "v_mfma_f32_16x16x128_f8f6f4 v[168:171], v[112:119], a[48:55], v[168:171] cbsz:1\n"
      "ds_read_b64_tr_b8 v[100:101], v92 offset:24576\n"
      "v_mfma_f32_16x16x128_f8f6f4 v[172:175], v[112:119], a[56:63], v[172:175] cbsz:1\n"
      "ds_read_b64_tr_b8 v[102:103], v92 offset:25600\n"
      "buffer_load_dwordx4 a[4:7], %[va0], %[asrd], s65 offen offset:1024 sc0 sc1\n"
      "s_waitcnt lgkmcnt(4)\n"
      "v_mfma_f32_16x16x128_f8f6f4 v[188:191], v[120:127], a[56:63], v[188:191] cbsz:1\n"
      "ds_read_b64_tr_b8 v[104:105], v93 offset:16384\n"
      "v_mfma_f32_16x16x128_f8f6f4 v[184:187], v[120:127], a[48:55], v[184:187] cbsz:1\n"
      "ds_read_b64_tr_b8 v[106:107], v93 offset:17408\n"
      "buffer_load_dwordx4 a[8:11], %[va0], %[asrd], s65 offen offset:2048 sc0 sc1\n"
      "v_mfma_f32_16x16x128_f8f6f4 v[180:183], v[120:127], a[40:47], v[180:183] cbsz:1\n"
      "ds_read_b64_tr_b8 v[108:109], v93 offset:24576\n"
      "v_mfma_f32_16x16x128_f8f6f4 v[176:179], v[120:127], a[32:39], v[176:179] cbsz:1\n"
      "ds_read_b64_tr_b8 v[110:111], v93 offset:25600\n"
      "buffer_load_dwordx4 a[12:15], %[va0], %[asrd], s65 offen offset:3072 sc0 sc1\n"
      "s_waitcnt lgkmcnt(4)\n"
      "v_mfma_f32_16x16x128_f8f6f4 v[192:195], v[96:103], a[32:39], v[192:195] cbsz:1\n"
      "ds_read_b64_tr_b8 v[112:113], v94 offset:16384\n"
      "v_mfma_f32_16x16x128_f8f6f4 v[196:199], v[96:103], a[40:47], v[196:199] cbsz:1\n"
      "ds_read_b64_tr_b8 v[114:115], v94 offset:17408\n"
      "buffer_load_dwordx4 a[16:19], v59, %[asrd], s65 offen offset:0 sc0 sc1\n"
      "v_mfma_f32_16x16x128_f8f6f4 v[200:203], v[96:103], a[48:55], v[200:203] cbsz:1\n"
      "ds_read_b64_tr_b8 v[116:117], v94 offset:24576\n"
      "v_mfma_f32_16x16x128_f8f6f4 v[204:207], v[96:103], a[56:63], v[204:207] cbsz:1\n"
      "ds_read_b64_tr_b8 v[118:119], v94 offset:25600\n"
      "buffer_load_dwordx4 a[20:23], v59, %[asrd], s65 offen offset:1024 sc0 sc1\n"
      "s_waitcnt lgkmcnt(4)\n"
      "v_mfma_f32_16x16x128_f8f6f4 v[220:223], v[104:111], a[56:63], v[220:223] cbsz:1\n"
      "ds_read_b64_tr_b8 v[120:121], v95 offset:16384\n"
      "v_mfma_f32_16x16x128_f8f6f4 v[216:219], v[104:111], a[48:55], v[216:219] cbsz:1\n"
      "ds_read_b64_tr_b8 v[122:123], v95 offset:17408\n"
      "buffer_load_dwordx4 a[24:27], v59, %[asrd], s65 offen offset:2048 sc0 sc1\n"
      "v_mfma_f32_16x16x128_f8f6f4 v[212:215], v[104:111], a[40:47], v[212:215] cbsz:1\n"
      "ds_read_b64_tr_b8 v[124:125], v95 offset:24576\n"
      "v_mfma_f32_16x16x128_f8f6f4 v[208:211], v[104:111], a[32:39], v[208:211] cbsz:1\n"
      "ds_read_b64_tr_b8 v[126:127], v95 offset:25600\n"
      "buffer_load_dwordx4 a[28:31], v59, %[asrd], s65 offen offset:3072 sc0 sc1\n"
      "s_waitcnt lgkmcnt(4)\n"
      "v_mfma_f32_16x16x128_f8f6f4 v[224:227], v[112:119], a[32:39], v[224:227] cbsz:1\n"
      "ds_read_b64_tr_b8 v[96:97], v88 offset:32768\n"
      "s_add_u32 s71, s64, 5\n"
      "v_readlane_b32 s90, v54, s71\n"
      "s_and_b32 s66, s90, 0xfffe0000\n"
      "s_and_b32 m0, s90, 0x1ffff\n"
      "s_nop 0\n"
      "v_mfma_f32_16x16x128_f8f6f4 v[228:231], v[112:119], a[40:47], v[228:231] cbsz:1\n"
      "ds_read_b64_tr_b8 v[98:99], v88 offset:33792\n"
      "buffer_load_dwordx4 v84, %[wsrd], s66 offen offset:0  lds\n"
      "v_mfma_f32_16x16x128_f8f6f4 v[232:235], v[112:119], a[48:55], v[232:235] cbsz:1\n"
      "ds_read_b64_tr_b8 v[100:101], v88 offset:40960\n"
      "v_mfma_f32_16x16x128_f8f6f4 v[236:239], v[112:119], a[56:63], v[236:239] cbsz:1\n"
      "ds_read_b64_tr_b8 v[102:103], v88 offset:41984\n"
      "buffer_load_dwordx4 v85, %[wsrd], s66 offen offset:1024  lds\n"
      "s_waitcnt lgkmcnt(4)\n"
      "v_mfma_f32_16x16x128_f8f6f4 v[252:255], v[120:127], a[56:63], v[252:255] cbsz:1\n"
      "ds_read_b64_tr_b8 v[104:105], v89 offset:32768\n"
      "v_mfma_f32_16x16x128_f8f6f4 v[248:251], v[120:127], a[48:55], v[248:251] cbsz:1\n"
      "ds_read_b64_tr_b8 v[106:107], v89 offset:33792\n"
      "buffer_load_dwordx4 v86, %[wsrd], s66 offen offset:2048  lds\n"
      "v_mfma_f32_16x16x128_f8f6f4 v[244:247], v[120:127], a[40:47], v[244:247] cbsz:1\n"
      "ds_read_b64_tr_b8 v[108:109], v89 offset:40960\n"
      "v_mfma_f32_16x16x128_f8f6f4 v[240:243], v[120:127], a[32:39], v[240:243] cbsz:1\n"
      "ds_read_b64_tr_b8 v[110:111], v89 offset:41984\n"
      "s_add_u32 s64, s64, 1\n"
      "buffer_load_dwordx4 v87, %[wsrd], s66 offen offset:3072  lds\n"
      "s_waitcnt vmcnt(24)\n"
      "s_barrier\n"
      "s_waitcnt lgkmcnt(4)\n"
      "v_mfma_f32_16x16x128_f8f6f4 v[128:131], v[96:103], a[64:71], v[128:131] cbsz:1\n"
      "ds_read_b64_tr_b8 v[112:113], v90 offset:32768\n"
      "v_mfma_f32_16x16x128_f8f6f4 v[132:135], v[96:103], a[72:79], v[132:135] cbsz:1\n"
      "ds_read_b64_tr_b8 v[114:115], v90 offset:33792\n"
      "s_add_u32 s71, s64, 5\n"
      "v_readlane_b32 s90, v54, s71\n"
      "s_and_b32 s66, s90, 0xfffe0000\n"
      "s_and_b32 m0, s90, 0x1ffff\n"
      "s_nop 0\n"
      "v_mfma_f32_16x16x128_f8f6f4 v[136:139], v[96:103], a[80:87], v[136:139] cbsz:1\n"
      "ds_read_b64_tr_b8 v[116:117], v90 offset:40960\n"
      "buffer_load_dwordx4 v84, %[wsrd], s66 offen offset:0  lds\n"
      "v_mfma_f32_16x16x128_f8f6f4 v[140:143], v[96:103], a[88:95], v[140:143] cbsz:1\n"
      "ds_read_b64_tr_b8 v[118:119], v90 offset:41984\n"
      "s_waitcnt lgkmcnt(4)\n"
      "v_mfma_f32_16x16x128_f8f6f4 v[156:159], v[104:111], a[88:95], v[156:159] cbsz:1\n"
      "ds_read_b64_tr_b8 v[120:121], v91 offset:32768\n"
      "buffer_load_dwordx4 v85, %[wsrd], s66 offen offset:1024  lds\n"
      "v_mfma_f32_16x16x128_f8f6f4 v[152:155], v[104:111], a[80:87], v[152:155] cbsz:1\n"
      "ds_read_b64_tr_b8 v[122:123], v91 offset:33792\n"
      "v_mfma_f32_16x16x128_f8f6f4 v[148:151], v[104:111], a[72:79], v[148:151] cbsz:1\n"
      "ds_read_b64_tr_b8 v[124:125], v91 offset:40960\n"
      "buffer_load_dwordx4 v86, %[wsrd], s66 offen offset:2048  lds\n"
      "v_mfma_f32_16x16x128_f8f6f4 v[144:147], v[104:111], a[64:71], v[144:147] cbsz:1\n"
      "ds_read_b64_tr_b8 v[126:127], v91 offset:41984\n"
      "s_waitcnt lgkmcnt(4)\n"
      "v_mfma_f32_16x16x128_f8f6f4 v[160:163], v[112:119], a[64:71], v[160:163] cbsz:1\n"
      "ds_read_b64_tr_b8 v[96:97], v92 offset:32768\n"
      "buffer_load_dwordx4 v87, %[wsrd], s66 offen offset:3072  lds\n"
      "v_mfma_f32_16x16x128_f8f6f4 v[164:167], v[112:119], a[72:79], v[164:167] cbsz:1\n"
      "ds_read_b64_tr_b8 v[98:99], v92 offset:33792\n"
      "v_mfma_f32_16x16x128_f8f6f4 v[168:171], v[112:119], a[80:87], v[168:171] cbsz:1\n"
      "ds_read_b64_tr_b8 v[100:101], v92 offset:40960\n"
      "v_mfma_f32_16x16x128_f8f6f4 v[172:175], v[112:119], a[88:95], v[172:175] cbsz:1\n"
      "ds_read_b64_tr_b8 v[102:103], v92 offset:41984\n"
      "s_waitcnt lgkmcnt(4)\n"
      "v_mfma_f32_16x16x128_f8f6f4 v[188:191], v[120:127], a[88:95], v[188:191] cbsz:1\n"
      "ds_read_b64_tr_b8 v[104:105], v93 offset:32768\n"
      "v_mfma_f32_16x16x128_f8f6f4 v[184:187], v[120:127], a[80:87], v[184:187] cbsz:1\n"
      "ds_read_b64_tr_b8 v[106:107], v93 offset:33792\n"
      "v_mfma_f32_16x16x128_f8f6f4 v[180:183], v[120:127], a[72:79], v[180:183] cbsz:1\n"
      "ds_read_b64_tr_b8 v[108:109], v93 offset:40960\n"
      "v_mfma_f32_16x16x128_f8f6f4 v[176:179], v[120:127], a[64:71], v[176:179] cbsz:1\n"
      "ds_read_b64_tr_b8 v[110:111], v93 offset:41984\n"
      "s_waitcnt lgkmcnt(4)\n"
      "v_mfma_f32_16x16x128_f8f6f4 v[192:195], v[96:103], a[64:71], v[192:195] cbsz:1\n"
      "ds_read_b64_tr_b8 v[112:113], v94 offset:32768\n"
      "v_mfma_f32_16x16x128_f8f6f4 v[196:199], v[96:103], a[72:79], v[196:199] cbsz:1\n"
      "ds_read_b64_tr_b8 v[114:115], v94 offset:33792\n"
      "v_mfma_f32_16x16x128_f8f6f4 v[200:203], v[96:103], a[80:87], v[200:203] cbsz:1\n"
      "ds_read_b64_tr_b8 v[116:117], v94 offset:40960\n"
      "v_mfma_f32_16x16x128_f8f6f4 v[204:207], v[96:103], a[88:95], v[204:207] cbsz:1\n"
      "ds_read_b64_tr_b8 v[118:119], v94 offset:41984\n"
      "s_waitcnt lgkmcnt(4)\n"
      "v_mfma_f32_16x16x128_f8f6f4 v[220:223], v[104:111], a[88:95], v[220:223] cbsz:1\n"
      "ds_read_b64_tr_b8 v[120:121], v95 offset:32768\n"
      "v_mfma_f32_16x16x128_f8f6f4 v[216:219], v[104:111], a[80:87], v[216:219] cbsz:1\n"
      "ds_read_b64_tr_b8 v[122:123], v95 offset:33792\n"
      "v_mfma_f32_16x16x128_f8f6f4 v[212:215], v[104:111], a[72:79], v[212:215] cbsz:1\n"
      "ds_read_b64_tr_b8 v[124:125], v95 offset:40960\n"
      "s_add_u32 s71, s64, 3\n"
      "v_readlane_b32 s65, v63, s71\n"
      "s_nop 1\n"
      "v_mfma_f32_16x16x128_f8f6f4 v[208:211], v[104:111], a[64:71], v[208:211] cbsz:1\n"
      "ds_read_b64_tr_b8 v[126:127], v95 offset:41984\n"
      "s_waitcnt lgkmcnt(4)\n"
      "v_mfma_f32_16x16x128_f8f6f4 v[224:227], v[112:119], a[64:71], v[224:227] cbsz:1\n"
      "ds_read_b64_tr_b8 v[96:97], v88 offset:49152\n"
      "buffer_load_dwordx4 a[32:35], %[va0], %[asrd], s65 offen offset:0 sc0 sc1\n"
      "v_mfma_f32_16x16x128_f8f6f4 v[228:231], v[112:119], a[72:79], v[228:231] cbsz:1\n"
      "ds_read_b64_tr_b8 v[98:99], v88 offset:50176\n"
      "v_mfma_f32_16x16x128_f8f6f4 v[232:235], v[112:119], a[80:87], v[232:235] cbsz:1\n"
      "ds_read_b64_tr_b8 v[100:101], v88 offset:57344\n"
      "buffer_load_dwordx4 a[36:39], %[va0], %[asrd], s65 offen offset:1024 sc0 sc1\n"
      "v_mfma_f32_16x16x128_f8f6f4 v[236:239], v[112:119], a[88:95], v[236:239] cbsz:1\n"
      "ds_read_b64_tr_b8 v[102:103], v88 offset:58368\n"
      "v_xor_b32_e32 v88, 0x10000, v88\n"
      "s_waitcnt lgkmcnt(4)\n"
      "v_mfma_f32_16x16x128_f8f6f4 v[252:255], v[120:127], a[88:95], v[252:255] cbsz:1\n"
      "ds_read_b64_tr_b8 v[104:105], v89 offset:49152\n"
      "ds_read_b64_tr_b8 v[106:107], v89 offset:50176\n"
      "buffer_load_dwordx4 a[40:43], %[va0], %[asrd], s65 offen offset:2048 sc0 sc1\n"
      "v_mfma_f32_16x16x128_f8f6f4 v[248:251], v[120:127], a[80:87], v[248:251] cbsz:1\n"
      "ds_read_b64_tr_b8 v[108:109], v89 offset:57344\n"
      "ds_read_b64_tr_b8 v[110:111], v89 offset:58368\n"
      "v_xor_b32_e32 v89, 0x10000, v89\n"
      "v_mfma_f32_16x16x128_f8f6f4 v[244:247], v[120:127], a[72:79], v[244:247] cbsz:1\n"
      "buffer_load_dwordx4 a[44:47], %[va0], %[asrd], s65 offen offset:3072 sc0 sc1\n"
      "v_mfma_f32_16x16x128_f8f6f4 v[240:243], v[120:127], a[64:71], v[240:243] cbsz:1\n"
      "s_add_u32 s64, s64, 1\n"
      "s_waitcnt vmcnt(20)\n"
      "s_barrier\n"
      "s_waitcnt lgkmcnt(0)\n"
      "v_mfma_f32_16x16x128_f8f6f4 v[128:131], v[96:103], a[96:103], v[128:131] cbsz:1\n"
      "ds_read_b64_tr_b8 v[112:113], v90 offset:49152\n"
      "s_min_u32 s56, s56, s57\n"
      "s_min_u32 s58, s58, s59\n"
      "s_min_u32 s60, s60, s61\n"
      "s_min_u32 s62, s62, s63\n"
      "buffer_load_dwordx4 a[48:51], v59, %[asrd], s65 offen offset:0 sc0 sc1\n"
      "v_mfma_f32_16x16x128_f8f6f4 v[132:135], v[96:103], a[104:111], v[132:135] cbsz:1\n"
      "ds_read_b64_tr_b8 v[114:115], v90 offset:50176\n"
      "s_min_u32 s56, s56, s58\n"
      "s_min_u32 s60, s60, s62\n"
      "s_min_u32 s56, s56, s60\n"
      "s_cmp_ge_u32 s56, s68\n"
      "s_cbranch_scc0 Lsl271_%=\n"
      "Lrt272_%=:\n"
      "v_mfma_f32_16x16x128_f8f6f4 v[136:139], v[96:103], a[112:119], v[136:139] cbsz:1\n"
      "ds_read_b64_tr_b8 v[116:117], v90 offset:57344\n"
      "s_add_u32 s70, s70, 128\n"
      "s_min_u32 s70, s70, %[sfl]\n"
      "s_lshl_b32 s89, s89, 2\n"
      "s_load_dwordx4 s[56:59], %[pfl], s70\n"
      "s_load_dwordx4 s[60:63], %[pfl], s70 offset:64\n"
      "buffer_load_dwordx4 a[52:55], v59, %[asrd], s65 offen offset:1024 sc0 sc1\n"
      "v_mfma_f32_16x16x128_f8f6f4 v[140:143], v[96:103], a[120:127], v[140:143] cbsz:1\n"
      "ds_read_b64_tr_b8 v[118:119], v90 offset:58368\n"
      "v_xor_b32_e32 v90, 0x10000, v90\n"
      "s_waitcnt lgkmcnt(4)\n"
      "v_mfma_f32_16x16x128_f8f6f4 v[156:159], v[104:111], a[120:127], v[156:159] cbsz:1\n"
      "ds_read_b64_tr_b8 v[120:121], v91 offset:49152\n"
      "buffer_load_dwordx4 a[56:59], v59, %[asrd], s65 offen offset:2048 sc0 sc1\n"
      "v_mfma_f32_16x16x128_f8f6f4 v[152:155], v[104:111], a[112:119], v[152:155] cbsz:1\n"
      "ds_read_b64_tr_b8 v[122:123], v91 offset:50176\n"
      "v_mfma_f32_16x16x128_f8f6f4 v[148:151], v[104:111], a[104:111], v[148:151] cbsz:1\n"
      "ds_read_b64_tr_b8 v[124:125], v91 offset:57344\n"
      "buffer_load_dwordx4 a[60:63], v59, %[asrd], s65 offen offset:3072 sc0 sc1\n"
      "v_mfma_f32_16x16x128_f8f6f4 v[144:147], v[104:111], a[96:103], v[144:147] cbsz:1\n"
      "ds_read_b64_tr_b8 v[126:127], v91 offset:58368\n"
      "v_xor_b32_e32 v91, 0x10000, v91\n"
      "s_add_u32 s71, s64, 3\n"
      "v_readlane_b32 s65, v63, s71\n"
      "s_nop 1\n"
      "s_waitcnt lgkmcnt(4)\n"
      "v_mfma_f32_16x16x128_f8f6f4 v[160:163], v[112:119], a[96:103], v[160:163] cbsz:1\n"
      "ds_read_b64_tr_b8 v[96:97], v92 offset:49152\n"
      "v_mfma_f32_16x16x128_f8f6f4 v[164:167], v[112:119], a[104:111], v[164:167] cbsz:1\n"
      "ds_read_b64_tr_b8 v[98:99], v92 offset:50176\n"
      "buffer_load_dwordx4 a[64:67], %[va0], %[asrd], s65 offen offset:0 sc0 sc1\n"
      "v_mfma_f32_16x16x128_f8f6f4 v[168:171], v[112:119], a[112:119], v[168:171] cbsz:1\n"
      "ds_read_b64_tr_b8 v[100:101], v92 offset:57344\n"
      "v_mfma_f32_16x16x128_f8f6f4 v[172:175], v[112:119], a[120:127], v[172:175] cbsz:1\n"
      "ds_read_b64_tr_b8 v[102:103], v92 offset:58368\n"
      "v_xor_b32_e32 v92, 0x10000, v92\n"
      "buffer_load_dwordx4 a[68:71], %[va0], %[asrd], s65 offen offset:1024 sc0 sc1\n"
      "s_waitcnt lgkmcnt(4)\n"
      "v_mfma_f32_16x16x128_f8f6f4 v[188:191], v[120:127], a[120:127], v[188:191] cbsz:1\n"
      "ds_read_b64_tr_b8 v[104:105], v93 offset:49152\n"
      "v_mfma_f32_16x16x128_f8f6f4 v[184:187], v[120:127], a[112:119], v[184:187] cbsz:1\n"
      "ds_read_b64_tr_b8 v[106:107], v93 offset:50176\n"
      "buffer_load_dwordx4 a[72:75], %[va0], %[asrd], s65 offen offset:2048 sc0 sc1\n"
      "v_mfma_f32_16x16x128_f8f6f4 v[180:183], v[120:127], a[104:111], v[180:183] cbsz:1\n"
      "ds_read_b64_tr_b8 v[108:109], v93 offset:57344\n"
      "v_mfma_f32_16x16x128_f8f6f4 v[176:179], v[120:127], a[96:103], v[176:179] cbsz:1\n"
      "ds_read_b64_tr_b8 v[110:111], v93 offset:58368\n"
      "v_xor_b32_e32 v93, 0x10000, v93\n"
      "buffer_load_dwordx4 a[76:79], %[va0], %[asrd], s65 offen offset:3072 sc0 sc1\n"
      "s_waitcnt lgkmcnt(4)\n"
      "v_mfma_f32_16x16x128_f8f6f4 v[192:195], v[96:103], a[96:103], v[192:195] cbsz:1\n"
      "ds_read_b64_tr_b8 v[112:113], v94 offset:49152\n"
      "v_mfma_f32_16x16x128_f8f6f4 v[196:199], v[96:103], a[104:111], v[196:199] cbsz:1\n"
      "ds_read_b64_tr_b8 v[114:115], v94 offset:50176\n"
      "buffer_load_dwordx4 a[80:83], v59, %[asrd], s65 offen offset:0 sc0 sc1\n"
      "v_mfma_f32_16x16x128_f8f6f4 v[200:203], v[96:103], a[112:119], v[200:203] cbsz:1\n"
      "ds_read_b64_tr_b8 v[116:117], v94 offset:57344\n"
      "v_mfma_f32_16x16x128_f8f6f4 v[204:207], v[96:103], a[120:127], v[204:207] cbsz:1\n"
      "ds_read_b64_tr_b8 v[118:119], v94 offset:58368\n"
      "v_xor_b32_e32 v94, 0x10000, v94\n"
      "buffer_load_dwordx4 a[84:87], v59, %[asrd], s65 offen offset:1024 sc0 sc1\n"
      "s_waitcnt lgkmcnt(4)\n"
      "v_mfma_f32_16x16x128_f8f6f4 v[220:223], v[104:111], a[120:127], v[220:223] cbsz:1\n"
      "ds_read_b64_tr_b8 v[120:121], v95 offset:49152\n"
      "v_mfma_f32_16x16x128_f8f6f4 v[216:219], v[104:111], a[112:119], v[216:219] cbsz:1\n"
      "ds_read_b64_tr_b8 v[122:123], v95 offset:50176\n"
      "buffer_load_dwordx4 a[88:91], v59, %[asrd], s65 offen offset:2048 sc0 sc1\n"
      "v_mfma_f32_16x16x128_f8f6f4 v[212:215], v[104:111], a[104:111], v[212:215] cbsz:1\n"
      "ds_read_b64_tr_b8 v[124:125], v95 offset:57344\n"
      "v_mfma_f32_16x16x128_f8f6f4 v[208:211], v[104:111], a[96:103], v[208:211] cbsz:1\n"
      "ds_read_b64_tr_b8 v[126:127], v95 offset:58368\n"
      "v_xor_b32_e32 v95, 0x10000, v95\n"
      "buffer_load_dwordx4 a[92:95], v59, %[asrd], s65 offen offset:3072 sc0 sc1\n"
      "s_waitcnt lgkmcnt(4)\n"
      "v_mfma_f32_16x16x128_f8f6f4 v[224:227], v[112:119], a[96:103], v[224:227] cbsz:1\n"
      "ds_read_b64_tr_b8 v[96:97], v88 offset:0\n"
      "s_add_u32 s71, s64, 5\n"
      "v_readlane_b32 s90, v54, s71\n"
      "s_and_b32 s66, s90, 0xfffe0000\n"
      "s_and_b32 m0, s90, 0x1ffff\n"
      "s_nop 0\n"
      "v_mfma_f32_16x16x128_f8f6f4 v[228:231], v[112:119], a[104:111], v[228:231] cbsz:1\n"
      "ds_read_b64_tr_b8 v[98:99], v88 offset:1024\n"
      "buffer_load_dwordx4 v84, %[wsrd], s66 offen offset:0  lds\n"
      "v_mfma_f32_16x16x128_f8f6f4 v[232:235], v[112:119], a[112:119], v[232:235] cbsz:1\n"
      "ds_read_b64_tr_b8 v[100:101], v88 offset:8192\n"
      "v_mfma_f32_16x16x128_f8f6f4 v[236:239], v[112:119], a[120:127], v[236:239] cbsz:1\n"
      "ds_read_b64_tr_b8 v[102:103], v88 offset:9216\n"
      "buffer_load_dwordx4 v85, %[wsrd], s66 offen offset:1024  lds\n"
      "s_waitcnt lgkmcnt(4)\n"
      "v_mfma_f32_16x16x128_f8f6f4 v[252:255], v[120:127], a[120:127], v[252:255] cbsz:1\n"
      "ds_read_b64_tr_b8 v[104:105], v89 offset:0\n"
      "v_mfma_f32_16x16x128_f8f6f4 v[248:251], v[120:127], a[112:119], v[248:251] cbsz:1\n"
      "ds_read_b64_tr_b8 v[106:107], v89 offset:1024\n"
      "buffer_load_dwordx4 v86, %[wsrd], s66 offen offset:2048  lds\n"
      "v_mfma_f32_16x16x128_f8f6f4 v[244:247], v[120:127], a[104:111], v[244:247] cbsz:1\n"
      "ds_read_b64_tr_b8 v[108:109], v89 offset:8192\n"
      "v_mfma_f32_16x16x128_f8f6f4 v[240:243], v[120:127], a[96:103], v[240:243] cbsz:1\n"
      "ds_read_b64_tr_b8 v[110:111], v89 offset:9216\n"
      "s_add_u32 s64, s64, 1\n"
      "buffer_load_dwordx4 v87, %[wsrd], s66 offen offset:3072  lds\n"
      "s_add_u32 s69, s69, 1\n"
      "s_cmp_lt_u32 s69, 4\n"
      "s_cbranch_scc1 Lloop0_%=\n"
      "s_branch Lpost_%=\n"
      ".p2align 6\n"
      "Lloop1_%=:\n"
      "s_waitcnt vmcnt(20)\n"
      "s_barrier\n"
      "s_waitcnt lgkmcnt(4)\n"
      "v_mfma_f32_16x16x128_f8f6f4 v[128:131], v[96:103], a[0:7], v[128:131] cbsz:1\n"
      "ds_read_b64_tr_b8 v[112:113], v90 offset:0\n"
      "buffer_load_dwordx4 a[80:83], v59, %[asrd], s65 offen offset:0 sc0 sc1\n"
      "v_mfma_f32_16x16x128_f8f6f4 v[132:135], v[96:103], a[8:15], v[132:135] cbsz:1\n"
      "ds_read_b64_tr_b8 v[114:115], v90 offset:1024\n"
      "v_mfma_f32_16x16x128_f8f6f4 v[136:139], v[96:103], a[16:23], v[136:139] cbsz:1\n"
      "ds_read_b64_tr_b8 v[116:117], v90 offset:8192\n"
      "buffer_load_dwordx4 a[84:87], v59, %[asrd], s65 offen offset:1024 sc0 sc1\n"
      "v_mfma_f32_16x16x128_f8f6f4 v[140:143], v[96:103], a[24:31], v[140:143] cbsz:1\n"
      "ds_read_b64_tr_b8 v[118:119], v90 offset:9216\n"
      "s_waitcnt lgkmcnt(4)\n"
      "v_mfma_f32_16x16x128_f8f6f4 v[156:159], v[104:111], a[24:31], v[156:159] cbsz:1\n"
      "ds_read_b64_tr_b8 v[120:121], v91 offset:0\n"
      "buffer_load_dwordx4 a[88:91], v59, %[asrd], s65 offen offset:2048 sc0 sc1\n"
      "v_mfma_f32_16x16x128_f8f6f4 v[152:155], v[104:111], a[16:23], v[152:155] cbsz:1\n"
      "ds_read_b64_tr_b8 v[122:123], v91 offset:1024\n"
      "v_mfma_f32_16x16x128_f8f6f4 v[148:151], v[104:111], a[8:15], v[148:151] cbsz:1\n"
      "ds_read_b64_tr_b8 v[124:125], v91 offset:8192\n"
      "buffer_load_dwordx4 a[92:95], v59, %[asrd], s65 offen offset:3072 sc0 sc1\n"
      "v_mfma_f32_16x16x128_f8f6f4 v[144:147], v[104:111], a[0:7], v[144:147] cbsz:1\n"
      "ds_read_b64_tr_b8 v[126:127], v91 offset:9216\n"
      "s_add_u32 s71, s64, 3\n"
      "v_readlane_b32 s65, v63, s71\n"
      "s_nop 1\n"
      "s_waitcnt lgkmcnt(4)\n"
      "v_mfma_f32_16x16x128_f8f6f4 v[160:163], v[112:119], a[0:7], v[160:163] cbsz:1\n"
      "ds_read_b64_tr_b8 v[96:97], v92 offset:0\n"
      "v_mfma_f32_16x16x128_f8f6f4 v[164:167], v[112:119], a[8:15], v[164:167] cbsz:1\n"
      "ds_read_b64_tr_b8 v[98:99], v92 offset:1024\n"
      "buffer_load_dwordx4 a[96:99], %[va0], %[asrd], s65 offen offset:0 sc0 sc1\n"
      "v_mfma_f32_16x16x128_f8f6f4 v[168:171], v[112:119], a[16:23], v[168:171] cbsz:1\n"
      "ds_read_b64_tr_b8 v[100:101], v92 offset:8192\n"
      "v_mfma_f32_16x16x128_f8f6f4 v[172:175], v[112:119], a[24:31], v[172:175] cbsz:1\n"
      "ds_read_b64_tr_b8 v[102:103], v92 offset:9216\n"
      "buffer_load_dwordx4 a[100:103], %[va0], %[asrd], s65 offen offset:1024 sc0 sc1\n"
      "s_waitcnt lgkmcnt(4)\n"
      "v_mfma_f32_16x16x128_f8f6f4 v[188:191], v[120:127], a[24:31], v[188:191] cbsz:1\n"
      "ds_read_b64_tr_b8 v[104:105], v93 offset:0\n"
      "v_mfma_f32_16x16x128_f8f6f4 v[184:187], v[120:127], a[16:23], v[184:187] cbsz:1\n"
      "ds_read_b64_tr_b8 v[106:107], v93 offset:1024\n"
      "buffer_load_dwordx4 a[104:107], %[va0], %[asrd], s65 offen offset:2048 sc0 sc1\n"
      "v_mfma_f32_16x16x128_f8f6f4 v[180:183], v[120:127], a[8:15], v[180:183] cbsz:1\n"
      "ds_read_b64_tr_b8 v[108:109], v93 offset:8192\n"
      "v_mfma_f32_16x16x128_f8f6f4 v[176:179], v[120:127], a[0:7], v[176:179] cbsz:1\n"
      "ds_read_b64_tr_b8 v[110:111], v93 offset:9216\n"
      "buffer_load_dwordx4 a[108:111], %[va0], %[asrd], s65 offen offset:3072 sc0 sc1\n"
      "s_waitcnt lgkmcnt(4)\n"
      "v_mfma_f32_16x16x128_f8f6f4 v[192:195], v[96:103], a[0:7], v[192:195] cbsz:1\n"
      "ds_read_b64_tr_b8 v[112:113], v94 offset:0\n"
      "v_mfma_f32_16x16x128_f8f6f4 v[196:199], v[96:103], a[8:15], v[196:199] cbsz:1\n"
      "ds_read_b64_tr_b8 v[114:115], v94 offset:1024\n"
      "buffer_load_dwordx4 a[112:115], v59, %[asrd], s65 offen offset:0 sc0 sc1\n"
      "v_mfma_f32_16x16x128_f8f6f4 v[200:203], v[96:103], a[16:23], v[200:203] cbsz:1\n"
      "ds_read_b64_tr_b8 v[116:117], v94 offset:8192\n"
      "v_mfma_f32_16x16x128_f8f6f4 v[204:207], v[96:103], a[24:31], v[204:207] cbsz:1\n"
      "ds_read_b64_tr_b8 v[118:119], v94 offset:9216\n"
      "buffer_load_dwordx4 a[116:119], v59, %[asrd], s65 offen offset:1024 sc0 sc1\n"
      "s_waitcnt lgkmcnt(4)\n"
      "v_mfma_f32_16x16x128_f8f6f4 v[220:223], v[104:111], a[24:31], v[220:223] cbsz:1\n"
      "ds_read_b64_tr_b8 v[120:121], v95 offset:0\n"
      "v_mfma_f32_16x16x128_f8f6f4 v[216:219], v[104:111], a[16:23], v[216:219] cbsz:1\n"
      "ds_read_b64_tr_b8 v[122:123], v95 offset:1024\n"
      "buffer_load_dwordx4 a[120:123], v59, %[asrd], s65 offen offset:2048 sc0 sc1\n"
      "v_mfma_f32_16x16x128_f8f6f4 v[212:215], v[104:111], a[8:15], v[212:215] cbsz:1\n"
      "ds_read_b64_tr_b8 v[124:125], v95 offset:8192\n"
      "v_mfma_f32_16x16x128_f8f6f4 v[208:211], v[104:111], a[0:7], v[208:211] cbsz:1\n"
      "ds_read_b64_tr_b8 v[126:127], v95 offset:9216\n"
      "buffer_load_dwordx4 a[124:127], v59, %[asrd], s65 offen offset:3072 sc0 sc1\n"
      "s_waitcnt lgkmcnt(4)\n"
      "v_mfma_f32_16x16x128_f8f6f4 v[224:227], v[112:119], a[0:7], v[224:227] cbsz:1\n"
      "ds_read_b64_tr_b8 v[96:97], v88 offset:16384\n"
      "s_add_u32 s71, s64, 5\n"
      "v_readlane_b32 s90, v54, s71\n"
      "s_and_b32 s66, s90, 0xfffe0000\n"
      "s_and_b32 m0, s90, 0x1ffff\n"
      "s_nop 0\n"
      "v_mfma_f32_16x16x128_f8f6f4 v[228:231], v[112:119], a[8:15], v[228:231] cbsz:1\n"
      "ds_read_b64_tr_b8 v[98:99], v88 offset:17408\n"
      "buffer_load_dwordx4 v84, %[wsrd], s66 offen offset:0  lds\n"
      "v_mfma_f32_16x16x128_f8f6f4 v[232:235], v[112:119], a[16:23], v[232:235] cbsz:1\n"
      "ds_read_b64_tr_b8 v[100:101], v88 offset:24576\n"
      "v_mfma_f32_16x16x128_f8f6f4 v[236:239], v[112:119], a[24:31], v[236:239] cbsz:1\n"
      "ds_read_b64_tr_b8 v[102:103], v88 offset:25600\n"
      "buffer_load_dwordx4 v85, %[wsrd], s66 offen offset:1024  lds\n"
      "s_waitcnt lgkmcnt(4)\n"
      "v_mfma_f32_16x16x128_f8f6f4 v[252:255], v[120:127], a[24:31], v[252:255] cbsz:1\n"
      "ds_read_b64_tr_b8 v[104:105], v89 offset:16384\n"
      "ds_read_b64_tr_b8 v[106:107], v89 offset:17408\n"
      "v_mfma_f32_16x16x128_f8f6f4 v[248:251], v[120:127], a[16:23], v[248:251] cbsz:1\n"
      "ds_read_b64_tr_b8 v[108:109], v89 offset:24576\n"
      "ds_read_b64_tr_b8 v[110:111], v89 offset:25600\n"
      "buffer_load_dwordx4 v86, %[wsrd], s66 offen offset:2048  lds\n"
      "v_mfma_f32_16x16x128_f8f6f4 v[244:247], v[120:127], a[8:15], v[244:247] cbsz:1\n"
      "v_mfma_f32_16x16x128_f8f6f4 v[240:243], v[120:127], a[0:7], v[240:243] cbsz:1\n"
      "s_add_u32 s64, s64, 1\n"
      "buffer_load_dwordx4 v87, %[wsrd], s66 offen offset:3072  lds\n"
      "s_waitcnt vmcnt(24)\n"
      "s_barrier\n"
      "s_waitcnt lgkmcnt(0)\n"
      "v_mfma_f32_16x16x128_f8f6f4 v[128:131], v[96:103], a[32:39], v[128:131] cbsz:1\n"
      "ds_read_b64_tr_b8 v[112:113], v90 offset:16384\n"
      "s_min_u32 s56, s56, s57\n"
      "s_min_u32 s58, s58, s59\n"
      "s_min_u32 s60, s60, s61\n"
      "s_min_u32 s62, s62, s63\n"
      "v_mfma_f32_16x16x128_f8f6f4 v[132:135], v[96:103], a[40:47], v[132:135] cbsz:1\n"
      "ds_read_b64_tr_b8 v[114:115], v90 offset:17408\n"
      "s_min_u32 s56, s56, s58\n"
      "s_min_u32 s60, s60, s62\n"
      "s_min_u32 s56, s56, s60\n"
      "s_cmp_ge_u32 s56, s68\n"
      "s_cbranch_scc0 Lsl273_%=\n"
      "Lrt274_%=:\n"
      "s_add_u32 s71, s64, 5\n"
      "v_readlane_b32 s90, v54, s71\n"
      "s_and_b32 s66, s90, 0xfffe0000\n"
      "s_and_b32 m0, s90, 0x1ffff\n"
      "s_nop 0\n"
      "v_mfma_f32_16x16x128_f8f6f4 v[136:139], v[96:103], a[48:55], v[136:139] cbsz:1\n"
      "ds_read_b64_tr_b8 v[116:117], v90 offset:24576\n"
      "s_add_u32 s70, s70, 128\n"
      "s_min_u32 s70, s70, %[sfl]\n"
      "s_lshl_b32 s89, s89, 2\n"
      "s_load_dwordx4 s[56:59], %[pfl], s70\n"
      "s_load_dwordx4 s[60:63], %[pfl], s70 offset:64\n"
      "buffer_load_dwordx4 v84, %[wsrd], s66 offen offset:0  lds\n"
      "v_mfma_f32_16x16x128_f8f6f4 v[140:143], v[96:103], a[56:63], v[140:143] cbsz:1\n"
      "ds_read_b64_tr_b8 v[118:119], v90 offset:25600\n"
      "s_waitcnt lgkmcnt(4)\n"
      "v_mfma_f32_16x16x128_f8f6f4 v[156:159], v[104:111], a[56:63], v[156:159] cbsz:1\n"
      "ds_read_b64_tr_b8 v[120:121], v91 offset:16384\n"
      "buffer_load_dwordx4 v85, %[wsrd], s66 offen offset:1024  lds\n"
      "v_mfma_f32_16x16x128_f8f6f4 v[152:155], v[104:111], a[48:55], v[152:155] cbsz:1\n"
      "ds_read_b64_tr_b8 v[122:123], v91 offset:17408\n"
      "v_mfma_f32_16x16x128_f8f6f4 v[148:151], v[104:111], a[40:47], v[148:151] cbsz:1\n"
      "ds_read_b64_tr_b8 v[124:125], v91 offset:24576\n"
      "buffer_load_dwordx4 v86, %[wsrd], s66 offen offset:2048  lds\n"
      "v_mfma_f32_16x16x128_f8f6f4 v[144:147], v[104:111], a[32:39], v[144:147] cbsz:1\n"
      "ds_read_b64_tr_b8 v[126:127], v91 offset:25600\n"
      "s_waitcnt lgkmcnt(4)\n"
      "v_mfma_f32_16x16x128_f8f6f4 v[160:163], v[112:119], a[32:39], v[160:163] cbsz:1\n"
      "ds_read_b64_tr_b8 v[96:97], v92 offset:16384\n"
      "buffer_load_dwordx4 v87, %[wsrd], s66 offen offset:3072  lds\n"
      "v_mfma_f32_16x16x128_f8f6f4 v[164:167], v[112:119], a[40:47], v[164:167] cbsz:1\n"
      "ds_read_b64_tr_b8 v[98:99], v92 offset:17408\n"
      "v_mfma_f32_16x16x128_f8f6f4 v[168:171], v[112:119], a[48:55], v[168:171] cbsz:1\n"
      "ds_read_b64_tr_b8 v[100:101], v92 offset:24576\n"
      "v_mfma_f32_16x16x128_f8f6f4 v[172:175], v[112:119], a[56:63], v[172:175] cbsz:1\n"
      "ds_read_b64_tr_b8 v[102:103], v92 offset:25600\n"
      "s_waitcnt lgkmcnt(4)\n"
      "v_mfma_f32_16x16x128_f8f6f4 v[188:191], v[120:127], a[56:63], v[188:191] cbsz:1\n"
      "ds_read_b64_tr_b8 v[104:105], v93 offset:16384\n"
      "v_mfma_f32_16x16x128_f8f6f4 v[184:187], v[120:127], a[48:55], v[184:187] cbsz:1\n"
      "ds_read_b64_tr_b8 v[106:107], v93 offset:17408\n"
      "v_mfma_f32_16x16x128_f8f6f4 v[180:183], v[120:127], a[40:47], v[180:183] cbsz:1\n"
      "ds_read_b64_tr_b8 v[108:109], v93 offset:24576\n"
      "v_mfma_f32_16x16x128_f8f6f4 v[176:179], v[120:127], a[32:39], v[176:179] cbsz:1\n"
      "ds_read_b64_tr_b8 v[110:111], v93 offset:25600\n"
      "s_waitcnt lgkmcnt(4)\n"
      "v_mfma_f32_16x16x128_f8f6f4 v[192:195], v[96:103], a[32:39], v[192:195] cbsz:1\n"
      "ds_read_b64_tr_b8 v[112:113], v94 offset:16384\n"
      "v_mfma_f32_16x16x128_f8f6f4 v[196:199], v[96:103], a[40:47], v[196:199] cbsz:1\n"
      "ds_read_b64_tr_b8 v[114:115], v94 offset:17408\n"
      "v_mfma_f32_16x16x128_f8f6f4 v[200:203], v[96:103], a[48:55], v[200:203] cbsz:1\n"
      "ds_read_b64_tr_b8 v[116:117], v94 offset:24576\n"
      "v_mfma_f32_16x16x128_f8f6f4 v[204:207], v[96:103], a[56:63], v[204:207] cbsz:1\n"
      "ds_read_b64_tr_b8 v[118:119], v94 offset:25600\n"
      "s_waitcnt lgkmcnt(4)\n"
      "v_mfma_f32_16x16x128_f8f6f4 v[220:223], v[104:111], a[56:63], v[220:223] cbsz:1\n"
      "ds_read_b64_tr_b8 v[120:121], v95 offset:16384\n"
      "v_mfma_f32_16x16x128_f8f6f4 v[216:219], v[104:111], a[48:55], v[216:219] cbsz:1\n"
      "ds_read_b64_tr_b8 v[122:123], v95 offset:17408\n"
      "v_mfma_f32_16x16x128_f8f6f4 v[212:215], v[104:111], a[40:47], v[212:215] cbsz:1\n"
      "ds_read_b64_tr_b8 v[124:125], v95 offset:24576\n"
      "s_add_u32 s71, s64, 3\n"
      "v_readlane_b32 s65, v63, s71\n"
      "s_nop 1\n"
      "v_mfma_f32_16x16x128_f8f6f4 v[208:211], v[104:111], a[32:39], v[208:211] cbsz:1\n"
      "ds_read_b64_tr_b8 v[126:127], v95 offset:25600\n"
      "s_waitcnt lgkmcnt(4)\n"
      "v_mfma_f32_16x16x128_f8f6f4 v[224:227], v[112:119], a[32:39], v[224:227] cbsz:1\n"
      "ds_read_b64_tr_b8 v[96:97], v88 offset:32768\n"
      "buffer_load_dwordx4 a[0:3], %[va0], %[asrd], s65 offen offset:0 sc0 sc1\n"
      "v_mfma_f32_16x16x128_f8f6f4 v[228:231], v[112:119], a[40:47], v[228:231] cbsz:1\n"
      "ds_read_b64_tr_b8 v[98:99], v88 offset:33792\n"
      "v_mfma_f32_16x16x128_f8f6f4 v[232:235], v[112:119], a[48:55], v[232:235] cbsz:1\n"
      "ds_read_b64_tr_b8 v[100:101], v88 offset:40960\n"
      "buffer_load_dwordx4 a[4:7], %[va0], %[asrd], s65 offen offset:1024 sc0 sc1\n"
      "v_mfma_f32_16x16x128_f8f6f4 v[236:239], v[112:119], a[56:63], v[236:239] cbsz:1\n"
      "ds_read_b64_tr_b8 v[102:103], v88 offset:41984\n"
      "s_waitcnt lgkmcnt(4)\n"
      "v_mfma_f32_16x16x128_f8f6f4 v[252:255], v[120:127], a[56:63], v[252:255] cbsz:1\n"
      "ds_read_b64_tr_b8 v[104:105], v89 offset:32768\n"
      "buffer_load_dwordx4 a[8:11], %[va0], %[asrd], s65 offen offset:2048 sc0 sc1\n"
      "v_mfma_f32_16x16x128_f8f6f4 v[248:251], v[120:127], a[48:55], v[248:251] cbsz:1\n"
      "ds_read_b64_tr_b8 v[106:107], v89 offset:33792\n"
      "v_mfma_f32_16x16x128_f8f6f4 v[244:247], v[120:127], a[40:47], v[244:247] cbsz:1\n"
      "ds_read_b64_tr_b8 v[108:109], v89 offset:40960\n"
      "buffer_load_dwordx4 a[12:15], %[va0], %[asrd], s65 offen offset:3072 sc0 sc1\n"
      "v_mfma_f32_16x16x128_f8f6f4 v[240:243], v[120:127], a[32:39], v[240:243] cbsz:1\n"
      "ds_read_b64_tr_b8 v[110:111], v89 offset:41984\n"
      "s_add_u32 s64, s64, 1\n"
      "s_waitcnt vmcnt(20)\n"
      "s_barrier\n"
      "s_waitcnt lgkmcnt(4)\n"
      "v_mfma_f32_16x16x128_f8f6f4 v[128:131], v[96:103], a[64:71], v[128:131] cbsz:1\n"
      "ds_read_b64_tr_b8 v[112:113], v90 offset:32768\n"
      "buffer_load_dwordx4 a[16:19], v59, %[asrd], s65 offen offset:0 sc0 sc1\n"
      "v_mfma_f32_16x16x128_f8f6f4 v[132:135], v[96:103], a[72:79], v[132:135] cbsz:1\n"
      "ds_read_b64_tr_b8 v[114:115], v90 offset:33792\n"
      "v_mfma_f32_16x16x128_f8f6f4 v[136:139], v[96:103], a[80:87], v[136:139] cbsz:1\n"
      "ds_read_b64_tr_b8 v[116:117], v90 offset:40960\n"
      "buffer_load_dwordx4 a[20:23], v59, %[asrd], s65 offen offset:1024 sc0 sc1\n"
      "v_mfma_f32_16x16x128_f8f6f4 v[140:143], v[96:103], a[88:95], v[140:143] cbsz:1\n"
      "ds_read_b64_tr_b8 v[118:119], v90 offset:41984\n"
      "s_waitcnt lgkmcnt(4)\n"
      "v_mfma_f32_16x16x128_f8f6f4 v[156:159], v[104:111], a[88:95], v[156:159] cbsz:1\n"
      "ds_read_b64_tr_b8 v[120:121], v91 offset:32768\n"
      "buffer_load_dwordx4 a[24:27], v59, %[asrd], s65 offen offset:2048 sc0 sc1\n"
      "v_mfma_f32_16x16x128_f8f6f4 v[152:155], v[104:111], a[80:87], v[152:155] cbsz:1\n"
      "ds_read_b64_tr_b8 v[122:123], v91 offset:33792\n"
      "v_mfma_f32_16x16x128_f8f6f4 v[148:151], v[104:111], a[72:79], v[148:151] cbsz:1\n"
      "ds_read_b64_tr_b8 v[124:125], v91 offset:40960\n"
      "buffer_load_dwordx4 a[28:31], v59, %[asrd], s65 offen offset:3072 sc0 sc1\n"
      "v_mfma_f32_16x16x128_f8f6f4 v[144:147], v[104:111], a[64:71], v[144:147] cbsz:1\n"
      "ds_read_b64_tr_b8 v[126:127], v91 offset:41984\n"
      "s_add_u32 s71, s64, 3\n"
      "v_readlane_b32 s65, v63, s71\n"
      "s_nop 1\n"
      "s_waitcnt lgkmcnt(4)\n"
      "v_mfma_f32_16x16x128_f8f6f4 v[160:163], v[112:119], a[64:71], v[160:163] cbsz:1\n"
      "ds_read_b64_tr_b8 v[96:97], v92 offset:32768\n"
      "v_mfma_f32_16x16x128_f8f6f4 v[164:167], v[112:119], a[72:79], v[164:167] cbsz:1\n"
      "ds_read_b64_tr_b8 v[98:99], v92 offset:33792\n"
      "buffer_load_dwordx4 a[32:35], %[va0], %[asrd], s65 offen offset:0 sc0 sc1\n"
      "v_mfma_f32_16x16x128_f8f6f4 v[168:171], v[112:119], a[80:87], v[168:171] cbsz:1\n"
      "ds_read_b64_tr_b8 v[100:101], v92 offset:40960\n"
      "v_mfma_f32_16x16x128_f8f6f4 v[172:175], v[112:119], a[88:95], v[172:175] cbsz:1\n"
      "ds_read_b64_tr_b8 v[102:103], v92 offset:41984\n"
      "buffer_load_dwordx4 a[36:39], %[va0], %[asrd], s65 offen offset:1024 sc0 sc1\n"
      "s_waitcnt lgkmcnt(4)\n"
      "v_mfma_f32_16x16x128_f8f6f4 v[188:191], v[120:127], a[88:95], v[188:191] cbsz:1\n"
      "ds_read_b64_tr_b8 v[104:105], v93 offset:32768\n"
      "v_mfma_f32_16x16x128_f8f6f4 v[184:187], v[120:127], a[80:87], v[184:187] cbsz:1\n"
      "ds_read_b64_tr_b8 v[106:107], v93 offset:33792\n"
      "buffer_load_dwordx4 a[40:43], %[va0], %[asrd], s65 offen offset:2048 sc0 sc1\n"
      "v_mfma_f32_16x16x128_f8f6f4 v[180:183], v[120:127], a[72:79], v[180:183] cbsz:1\n"
      "ds_read_b64_tr_b8 v[108:109], v93 offset:40960\n"
      "v_mfma_f32_16x16x128_f8f6f4 v[176:179], v[120:127], a[64:71], v[176:179] cbsz:1\n"
      "ds_read_b64_tr_b8 v[110:111], v93 offset:41984\n"
      "buffer_load_dwordx4 a[44:47], %[va0], %[asrd], s65 offen offset:3072 sc0 sc1\n"
      "s_waitcnt lgkmcnt(4)\n"
      "v_mfma_f32_16x16x128_f8f6f4 v[192:195], v[96:103], a[64:71], v[192:195] cbsz:1\n"
      "ds_read_b64_tr_b8 v[112:113], v94 offset:32768\n"
      "v_mfma_f32_16x16x128_f8f6f4 v[196:199], v[96:103], a[72:79], v[196:199] cbsz:1\n"
      "ds_read_b64_tr_b8 v[114:115], v94 offset:33792\n"
      "buffer_load_dwordx4 a[48:51], v59, %[asrd], s65 offen offset:0 sc0 sc1\n"
      "v_mfma_f32_16x16x128_f8f6f4 v[200:203], v[96:103], a[80:87], v[200:203] cbsz:1\n"
      "ds_read_b64_tr_b8 v[116:117], v94 offset:40960\n"
      "v_mfma_f32_16x16x128_f8f6f4 v[204:207], v[96:103], a[88:95], v[204:207] cbsz:1\n"
      "ds_read_b64_tr_b8 v[118:119], v94 offset:41984\n"
      "buffer_load_dwordx4 a[52:55], v59, %[asrd], s65 offen offset:1024 sc0 sc1\n"
      "s_waitcnt lgkmcnt(4)\n"
      "v_mfma_f32_16x16x128_f8f6f4 v[220:223], v[104:111], a[88:95], v[220:223] cbsz:1\n"
      "ds_read_b64_tr_b8 v[120:121], v95 offset:32768\n"
      "v_mfma_f32_16x16x128_f8f6f4 v[216:219], v[104:111], a[80:87], v[216:219] cbsz:1\n"
      "ds_read_b64_tr_b8 v[122:123], v95 offset:33792\n"
      "buffer_load_dwordx4 a[56:59], v59, %[asrd], s65 offen offset:2048 sc0 sc1\n"
      "v_mfma_f32_16x16x128_f8f6f4 v[212:215], v[104:111], a[72:79], v[212:215] cbsz:1\n"
      "ds_read_b64_tr_b8 v[124:125], v95 offset:40960\n"
      "v_mfma_f32_16x16x128_f8f6f4 v[208:211], v[104:111], a[64:71], v[208:211] cbsz:1\n"
      "ds_read_b64_tr_b8 v[126:127], v95 offset:41984\n"
      "buffer_load_dwordx4 a[60:63], v59, %[asrd], s65 offen offset:3072 sc0 sc1\n"
      "s_waitcnt lgkmcnt(4)\n"
      "v_mfma_f32_16x16x128_f8f6f4 v[224:227], v[112:119], a[64:71], v[224:227] cbsz:1\n"
      "ds_read_b64_tr_b8 v[96:97], v88 offset:49152\n"
      "s_add_u32 s71, s64, 5\n"
      "v_readlane_b32 s90, v54, s71\n"
      "s_and_b32 s66, s90, 0xfffe0000\n"
      "s_and_b32 m0, s90, 0x1ffff\n"
      "s_nop 0\n"
      "v_mfma_f32_16x16x128_f8f6f4 v[228:231], v[112:119], a[72:79], v[228:231] cbsz:1\n"
      "ds_read_b64_tr_b8 v[98:99], v88 offset:50176\n"
      "buffer_load_dwordx4 v84, %[wsrd], s66 offen offset:0  lds\n"
      "v_mfma_f32_16x16x128_f8f6f4 v[232:235], v[112:119], a[80:87], v[232:235] cbsz:1\n"
      "ds_read_b64_tr_b8 v[100:101], v88 offset:57344\n"
      "v_mfma_f32_16x16x128_f8f6f4 v[236:239], v[112:119], a[88:95], v[236:239] cbsz:1\n"
      "ds_read_b64_tr_b8 v[102:103], v88 offset:58368\n"
      "v_xor_b32_e32 v88, 0x10000, v88\n"
      "buffer_load_dwordx4 v85, %[wsrd], s66 offen offset:1024  lds\n"
      "s_waitcnt lgkmcnt(4)\n"
      "v_mfma_f32_16x16x128_f8f6f4 v[252:255], v[120:127], a[88:95], v[252:255] cbsz:1\n"
      "ds_read_b64_tr_b8 v[104:105], v89 offset:49152\n"
      "ds_read_b64_tr_b8 v[106:107], v89 offset:50176\n"
      "v_mfma_f32_16x16x128_f8f6f4 v[248:251], v[120:127], a[80:87], v[248:251] cbsz:1\n"
      "ds_read_b64_tr_b8 v[108:109], v89 offset:57344\n"
      "ds_read_b64_tr_b8 v[110:111], v89 offset:58368\n"
      "v_xor_b32_e32 v89, 0x10000, v89\n"
      "buffer_load_dwordx4 v86, %[wsrd], s66 offen offset:2048  lds\n"
      "v_mfma_f32_16x16x128_f8f6f4 v[244:247], v[120:127], a[72:79], v[244:247] cbsz:1\n"
      "v_mfma_f32_16x16x128_f8f6f4 v[240:243], v[120:127], a[64:71], v[240:243] cbsz:1\n"
      "s_add_u32 s64, s64, 1\n"
      "buffer_load_dwordx4 v87, %[wsrd], s66 offen offset:3072  lds\n"
      "s_waitcnt vmcnt(24)\n"
      "s_barrier\n"
      "s_waitcnt lgkmcnt(0)\n"
      "v_mfma_f32_16x16x128_f8f6f4 v[128:131], v[96:103], a[96:103], v[128:131] cbsz:1\n"
      "ds_read_b64_tr_b8 v[112:113], v90 offset:49152\n"
      "s_min_u32 s56, s56, s57\n"
      "s_min_u32 s58, s58, s59\n"
      "s_min_u32 s60, s60, s61\n"
      "s_min_u32 s62, s62, s63\n"
      "v_mfma_f32_16x16x128_f8f6f4 v[132:135], v[96:103], a[104:111], v[132:135] cbsz:1\n"
      "ds_read_b64_tr_b8 v[114:115], v90 offset:50176\n"
      "s_min_u32 s56, s56, s58\n"
      "s_min_u32 s60, s60, s62\n"
      "s_min_u32 s56, s56, s60\n"
      "s_cmp_ge_u32 s56, s68\n"
      "s_cbranch_scc0 Lsl275_%=\n"
      "Lrt276_%=:\n"
      "s_add_u32 s71, s64, 5\n"
      "v_readlane_b32 s90, v54, s71\n"
      "s_and_b32 s66, s90, 0xfffe0000\n"
      "s_and_b32 m0, s90, 0x1ffff\n"
      "s_nop 0\n"
      "v_mfma_f32_16x16x128_f8f6f4 v[136:139], v[96:103], a[112:119], v[136:139] cbsz:1\n"
      "ds_read_b64_tr_b8 v[116:117], v90 offset:57344\n"
      "s_add_u32 s70, s70, 128\n"
      "s_min_u32 s70, s70, %[sfl]\n"
      "s_lshl_b32 s89, s89, 2\n"
      "s_load_dwordx4 s[56:59], %[pfl], s70\n"
      "s_load_dwordx4 s[60:63], %[pfl], s70 offset:64\n"
      "buffer_load_dwordx4 v84, %[wsrd], s66 offen offset:0  lds\n"
      "v_mfma_f32_16x16x128_f8f6f4 v[140:143], v[96:103], a[120:127], v[140:143] cbsz:1\n"
      "ds_read_b64_tr_b8 v[118:119], v90 offset:58368\n"
      "v_xor_b32_e32 v90, 0x10000, v90\n"
      "s_waitcnt lgkmcnt(4)\n"
      "v_mfma_f32_16x16x128_f8f6f4 v[156:159], v[104:111], a[120:127], v[156:159] cbsz:1\n"
      "ds_read_b64_tr_b8 v[120:121], v91 offset:49152\n"
      "buffer_load_dwordx4 v85, %[wsrd], s66 offen offset:1024  lds\n"
      "v_mfma_f32_16x16x128_f8f6f4 v[152:155], v[104:111], a[112:119], v[152:155] cbsz:1\n"
      "ds_read_b64_tr_b8 v[122:123], v91 offset:50176\n"
      "v_mfma_f32_16x16x128_f8f6f4 v[148:151], v[104:111], a[104:111], v[148:151] cbsz:1\n"
      "ds_read_b64_tr_b8 v[124:125], v91 offset:57344\n"
      "buffer_load_dwordx4 v86, %[wsrd], s66 offen offset:2048  lds\n"
      "v_mfma_f32_16x16x128_f8f6f4 v[144:147], v[104:111], a[96:103], v[144:147] cbsz:1\n"
      "ds_read_b64_tr_b8 v[126:127], v91 offset:58368\n"
      "v_xor_b32_e32 v91, 0x10000, v91\n"
      "s_waitcnt lgkmcnt(4)\n"
      "v_mfma_f32_16x16x128_f8f6f4 v[160:163], v[112:119], a[96:103], v[160:163] cbsz:1\n"
      "ds_read_b64_tr_b8 v[96:97], v92 offset:49152\n"
      "buffer_load_dwordx4 v87, %[wsrd], s66 offen offset:3072  lds\n"
      "v_mfma_f32_16x16x128_f8f6f4 v[164:167], v[112:119], a[104:111], v[164:167] cbsz:1\n"
      "ds_read_b64_tr_b8 v[98:99], v92 offset:50176\n"
      "v_mfma_f32_16x16x128_f8f6f4 v[168:171], v[112:119], a[112:119], v[168:171] cbsz:1\n"
      "ds_read_b64_tr_b8 v[100:101], v92 offset:57344\n"
      "v_mfma_f32_16x16x128_f8f6f4 v[172:175], v[112:119], a[120:127], v[172:175] cbsz:1\n"
      "ds_read_b64_tr_b8 v[102:103], v92 offset:58368\n"
      "v_xor_b32_e32 v92, 0x10000, v92\n"
      "s_waitcnt lgkmcnt(4)\n"
      "v_mfma_f32_16x16x128_f8f6f4 v[188:191], v[120:127], a[120:127], v[188:191] cbsz:1\n"
      "ds_read_b64_tr_b8 v[104:105], v93 offset:49152\n"
      "v_mfma_f32_16x16x128_f8f6f4 v[184:187], v[120:127], a[112:119], v[184:187] cbsz:1\n"
      "ds_read_b64_tr_b8 v[106:107], v93 offset:50176\n"
      "v_mfma_f32_16x16x128_f8f6f4 v[180:183], v[120:127], a[104:111], v[180:183] cbsz:1\n"
      "ds_read_b64_tr_b8 v[108:109], v93 offset:57344\n"
      "v_mfma_f32_16x16x128_f8f6f4 v[176:179], v[120:127], a[96:103], v[176:179] cbsz:1\n"
      "ds_read_b64_tr_b8 v[110:111], v93 offset:58368\n"
      "v_xor_b32_e32 v93, 0x10000, v93\n"
      "s_waitcnt lgkmcnt(4)\n"
      "v_mfma_f32_16x16x128_f8f6f4 v[192:195], v[96:103], a[96:103], v[192:195] cbsz:1\n"
      "ds_read_b64_tr_b8 v[112:113], v94 offset:49152\n"
      "v_mfma_f32_16x16x128_f8f6f4 v[196:199], v[96:103], a[104:111], v[196:199] cbsz:1\n"
      "ds_read_b64_tr_b8 v[114:115], v94 offset:50176\n"
      "v_mfma_f32_16x16x128_f8f6f4 v[200:203], v[96:103], a[112:119], v[200:203] cbsz:1\n"
      "ds_read_b64_tr_b8 v[116:117], v94 offset:57344\n"
      "v_mfma_f32_16x16x128_f8f6f4 v[204:207], v[96:103], a[120:127], v[204:207] cbsz:1\n"
      "ds_read_b64_tr_b8 v[118:119], v94 offset:58368\n"
      "v_xor_b32_e32 v94, 0x10000, v94\n"
      "s_waitcnt lgkmcnt(4)\n"
      "v_mfma_f32_16x16x128_f8f6f4 v[220:223], v[104:111], a[120:127], v[220:223] cbsz:1\n"
      "ds_read_b64_tr_b8 v[120:121], v95 offset:49152\n"
      "v_mfma_f32_16x16x128_f8f6f4 v[216:219], v[104:111], a[112:119], v[216:219] cbsz:1\n"
      "ds_read_b64_tr_b8 v[122:123], v95 offset:50176\n"
      "v_mfma_f32_16x16x128_f8f6f4 v[212:215], v[104:111], a[104:111], v[212:215] cbsz:1\n"
      "ds_read_b64_tr_b8 v[124:125], v95 offset:57344\n"
      "s_add_u32 s71, s64, 3\n"
      "v_readlane_b32 s65, v63, s71\n"
      "s_nop 1\n"
      "v_mfma_f32_16x16x128_f8f6f4 v[208:211], v[104:111], a[96:103], v[208:211] cbsz:1\n"
      "ds_read_b64_tr_b8 v[126:127], v95 offset:58368\n"
      "v_xor_b32_e32 v95, 0x10000, v95\n"
      "s_waitcnt lgkmcnt(4)\n"
      "v_mfma_f32_16x16x128_f8f6f4 v[224:227], v[112:119], a[96:103], v[224:227] cbsz:1\n"
      "ds_read_b64_tr_b8 v[96:97], v88 offset:0\n"
      "buffer_load_dwordx4 a[64:67], %[va0], %[asrd], s65 offen offset:0 sc0 sc1\n"
      "v_mfma_f32_16x16x128_f8f6f4 v[228:231], v[112:119], a[104:111], v[228:231] cbsz:1\n"
      "ds_read_b64_tr_b8 v[98:99], v88 offset:1024\n"
      "v_mfma_f32_16x16x128_f8f6f4 v[232:235], v[112:119], a[112:119], v[232:235] cbsz:1\n"
      "ds_read_b64_tr_b8 v[100:101], v88 offset:8192\n"
      "buffer_load_dwordx4 a[68:71], %[va0], %[asrd], s65 offen offset:1024 sc0 sc1\n"
      "v_mfma_f32_16x16x128_f8f6f4 v[236:239], v[112:119], a[120:127], v[236:239] cbsz:1\n"
      "ds_read_b64_tr_b8 v[102:103], v88 offset:9216\n"
      "s_waitcnt lgkmcnt(4)\n"
      "v_mfma_f32_16x16x128_f8f6f4 v[252:255], v[120:127], a[120:127], v[252:255] cbsz:1\n"
      "ds_read_b64_tr_b8 v[104:105], v89 offset:0\n"
      "buffer_load_dwordx4 a[72:75], %[va0], %[asrd], s65 offen offset:2048 sc0 sc1\n"
      "v_mfma_f32_16x16x128_f8f6f4 v[248:251], v[120:127], a[112:119], v[248:251] cbsz:1\n"
      "ds_read_b64_tr_b8 v[106:107], v89 offset:1024\n"
      "v_mfma_f32_16x16x128_f8f6f4 v[244:247], v[120:127], a[104:111], v[244:247] cbsz:1\n"
      "ds_read_b64_tr_b8 v[108:109], v89 offset:8192\n"
      "buffer_load_dwordx4 a[76:79], %[va0], %[asrd], s65 offen offset:3072 sc0 sc1\n"
      "v_mfma_f32_16x16x128_f8f6f4 v[240:243], v[120:127], a[96:103], v[240:243] cbsz:1\n"
      "ds_read_b64_tr_b8 v[110:111], v89 offset:9216\n"
      "s_add_u32 s64, s64, 1\n"
      "s_add_u32 s69, s69, 1\n"
      "s_cmp_lt_u32 s69, 4\n"
      "s_cbranch_scc1 Lloop1_%=\n"
      "Lpost_%=:\n"
      "Ls24_%=:\n"
      "s_cmp_eq_u32 %[smt], 0\n"
      "s_cbranch_scc1 Ltw277_%=\n"
      "s_waitcnt vmcnt(20)\n"
      "Ltw277_%=:\n"
      "s_waitcnt vmcnt(24)\n"
      "s_barrier\n"
      "s_waitcnt lgkmcnt(4)\n"
      "v_mfma_f32_16x16x128_f8f6f4 v[128:131], v[96:103], a[0:7], v[128:131] cbsz:1\n"
      "ds_read_b64_tr_b8 v[112:113], v90 offset:0\n"
      "v_mfma_f32_16x16x128_f8f6f4 v[132:135], v[96:103], a[8:15], v[132:135] cbsz:1\n"
      "ds_read_b64_tr_b8 v[114:115], v90 offset:1024\n"
      "buffer_load_dwordx4 a[80:83], v59, %[asrd], s65 offen offset:0 sc0 sc1\n"
      "v_mfma_f32_16x16x128_f8f6f4 v[136:139], v[96:103], a[16:23], v[136:139] cbsz:1\n"
      "ds_read_b64_tr_b8 v[116:117], v90 offset:8192\n"
      "v_mfma_f32_16x16x128_f8f6f4 v[140:143], v[96:103], a[24:31], v[140:143] cbsz:1\n"
      "ds_read_b64_tr_b8 v[118:119], v90 offset:9216\n"
      "buffer_load_dwordx4 a[84:87], v59, %[asrd], s65 offen offset:1024 sc0 sc1\n"
      "s_waitcnt lgkmcnt(4)\n"
      "v_mfma_f32_16x16x128_f8f6f4 v[156:159], v[104:111], a[24:31], v[156:159] cbsz:1\n"
      "ds_read_b64_tr_b8 v[120:121], v91 offset:0\n"
      "v_mfma_f32_16x16x128_f8f6f4 v[152:155], v[104:111], a[16:23], v[152:155] cbsz:1\n"
      "ds_read_b64_tr_b8 v[122:123], v91 offset:1024\n"
      "buffer_load_dwordx4 a[88:91], v59, %[asrd], s65 offen offset:2048 sc0 sc1\n"
      "v_mfma_f32_16x16x128_f8f6f4 v[148:151], v[104:111], a[8:15], v[148:151] cbsz:1\n"
      "ds_read_b64_tr_b8 v[124:125], v91 offset:8192\n"
      "v_mfma_f32_16x16x128_f8f6f4 v[144:147], v[104:111], a[0:7], v[144:147] cbsz:1\n"
      "ds_read_b64_tr_b8 v[126:127], v91 offset:9216\n"
      "buffer_load_dwordx4 a[92:95], v59, %[asrd], s65 offen offset:3072 sc0 sc1\n"
      "s_waitcnt lgkmcnt(4)\n"
      "v_mfma_f32_16x16x128_f8f6f4 v[160:163], v[112:119], a[0:7], v[160:163] cbsz:1\n"
      "ds_read_b64_tr_b8 v[96:97], v92 offset:0\n"
      "s_getpc_b64 s[74:75]\n"
      "Lct278_%=:\n"
      "s_add_u32 s74, s74, Ls25_%=-Lct278_%=\n"
      "s_addc_u32 s75, s75, 0\n"
      "v_min_u32_e32 v58, Lend_%=-Ls25_%=-4, %[vct]\n"
      "s_mov_b64 exec, 3\n"
      "global_load_dword v57, v58, s[74:75]\n"
      "s_mov_b64 exec, -1\n"
      "v_mfma_f32_16x16x128_f8f6f4 v[164:167], v[112:119], a[8:15], v[164:167] cbsz:1\n"
      "ds_read_b64_tr_b8 v[98:99], v92 offset:1024\n"
      "v_readlane_b32 s90, v54, 29\n"
      "s_and_b32 s66, s90, 0xfffe0000\n"
      "s_and_b32 m0, s90, 0x1ffff\n"
      "s_nop 0\n"
      "buffer_load_dwordx4 v84, %[wsrd], s66 offen offset:0  lds\n"
      "v_mfma_f32_16x16x128_f8f6f4 v[168:171], v[112:119], a[16:23], v[168:171] cbsz:1\n"
      "ds_read_b64_tr_b8 v[100:101], v92 offset:8192\n"
      "v_mfma_f32_16x16x128_f8f6f4 v[172:175], v[112:119], a[24:31], v[172:175] cbsz:1\n"
      "ds_read_b64_tr_b8 v[102:103], v92 offset:9216\n"
      "buffer_load_dwordx4 v85, %[wsrd], s66 offen offset:1024  lds\n"
      "s_waitcnt lgkmcnt(4)\n"
      "v_mfma_f32_16x16x128_f8f6f4 v[188:191], v[120:127], a[24:31], v[188:191] cbsz:1\n"
      "ds_read_b64_tr_b8 v[104:105], v93 offset:0\n"
      "v_mfma_f32_16x16x128_f8f6f4 v[184:187], v[120:127], a[16:23], v[184:187] cbsz:1\n"
      "ds_read_b64_tr_b8 v[106:107], v93 offset:1024\n"
      "buffer_load_dwordx4 v86, %[wsrd], s66 offen offset:2048  lds\n"
      "v_mfma_f32_16x16x128_f8f6f4 v[180:183], v[120:127], a[8:15], v[180:183] cbsz:1\n"
      "ds_read_b64_tr_b8 v[108:109], v93 offset:8192\n"
      "v_mfma_f32_16x16x128_f8f6f4 v[176:179], v[120:127], a[0:7], v[176:179] cbsz:1\n"
      "ds_read_b64_tr_b8 v[110:111], v93 offset:9216\n"
      "buffer_load_dwordx4 v87, %[wsrd], s66 offen offset:3072  lds\n"
      "v_readlane_b32 s65, v63, 27\n"
      "s_nop 1\n"
      "s_waitcnt lgkmcnt(4)\n"
      "v_mfma_f32_16x16x128_f8f6f4 v[192:195], v[96:103], a[0:7], v[192:195] cbsz:1\n"
      "ds_read_b64_tr_b8 v[112:113], v94 offset:0\n"
      "v_mfma_f32_16x16x128_f8f6f4 v[196:199], v[96:103], a[8:15], v[196:199] cbsz:1\n"
      "ds_read_b64_tr_b8 v[114:115], v94 offset:1024\n"
      "buffer_load_dwordx4 a[96:99], %[va0], %[asrd], s65 offen offset:0 sc0 sc1\n"
      "v_mfma_f32_16x16x128_f8f6f4 v[200:203], v[96:103], a[16:23], v[200:203] cbsz:1\n"
      "ds_read_b64_tr_b8 v[116:117], v94 offset:8192\n"
      "v_mfma_f32_16x16x128_f8f6f4 v[204:207], v[96:103], a[24:31], v[204:207] cbsz:1\n"
      "ds_read_b64_tr_b8 v[118:119], v94 offset:9216\n"
      "buffer_load_dwordx4 a[100:103], %[va0], %[asrd], s65 offen offset:1024 sc0 sc1\n"
      "s_waitcnt lgkmcnt(4)\n"
      "v_mfma_f32_16x16x128_f8f6f4 v[220:223], v[104:111], a[24:31], v[220:223] cbsz:1\n"
      "ds_read_b64_tr_b8 v[120:121], v95 offset:0\n"
      "v_mfma_f32_16x16x128_f8f6f4 v[216:219], v[104:111], a[16:23], v[216:219] cbsz:1\n"
      "ds_read_b64_tr_b8 v[122:123], v95 offset:1024\n"
      "buffer_load_dwordx4 a[104:107], %[va0], %[asrd], s65 offen offset:2048 sc0 sc1\n"
      "v_mfma_f32_16x16x128_f8f6f4 v[212:215], v[104:111], a[8:15], v[212:215] cbsz:1\n"
      "ds_read_b64_tr_b8 v[124:125], v95 offset:8192\n"
      "v_mfma_f32_16x16x128_f8f6f4 v[208:211], v[104:111], a[0:7], v[208:211] cbsz:1\n"
      "ds_read_b64_tr_b8 v[126:127], v95 offset:9216\n"
      "buffer_load_dwordx4 a[108:111], %[va0], %[asrd], s65 offen offset:3072 sc0 sc1\n"
      "s_waitcnt lgkmcnt(4)\n"
      "v_mfma_f32_16x16x128_f8f6f4 v[224:227], v[112:119], a[0:7], v[224:227] cbsz:1\n"
      "ds_read_b64_tr_b8 v[96:97], v88 offset:16384\n"
      "v_mfma_f32_16x16x128_f8f6f4 v[228:231], v[112:119], a[8:15], v[228:231] cbsz:1\n"
      "ds_read_b64_tr_b8 v[98:99], v88 offset:17408\n"
      "buffer_load_dwordx4 a[112:115], v59, %[asrd], s65 offen offset:0 sc0 sc1\n"
      "v_mfma_f32_16x16x128_f8f6f4 v[232:235], v[112:119], a[16:23], v[232:235] cbsz:1\n"
      "ds_read_b64_tr_b8 v[100:101], v88 offset:24576\n"
      "v_mfma_f32_16x16x128_f8f6f4 v[236:239], v[112:119], a[24:31], v[236:239] cbsz:1\n"
      "ds_read_b64_tr_b8 v[102:103], v88 offset:25600\n"
      "buffer_load_dwordx4 a[116:119], v59, %[asrd], s65 offen offset:1024 sc0 sc1\n"
      "s_waitcnt lgkmcnt(4)\n"
      "v_mfma_f32_16x16x128_f8f6f4 v[252:255], v[120:127], a[24:31], v[252:255] cbsz:1\n"
      "ds_read_b64_tr_b8 v[104:105], v89 offset:16384\n"
      "ds_read_b64_tr_b8 v[106:107], v89 offset:17408\n"
      "v_mfma_f32_16x16x128_f8f6f4 v[248:251], v[120:127], a[16:23], v[248:251] cbsz:1\n"
      "ds_read_b64_tr_b8 v[108:109], v89 offset:24576\n"
      "ds_read_b64_tr_b8 v[110:111], v89 offset:25600\n"
      "buffer_load_dwordx4 a[120:123], v59, %[asrd], s65 offen offset:2048 sc0 sc1\n"
      "v_mfma_f32_16x16x128_f8f6f4 v[244:247], v[120:127], a[8:15], v[244:247] cbsz:1\n"
      "v_mfma_f32_16x16x128_f8f6f4 v[240:243], v[120:127], a[0:7], v[240:243] cbsz:1\n"
      "buffer_load_dwordx4 a[124:127], v59, %[asrd], s65 offen offset:3072 sc0 sc1\n"
      "Ls25_%=:\n"
      "s_cmp_eq_u32 %[smt], 0\n"
      "s_cbranch_scc1 Ltw279_%=\n"
      "s_waitcnt vmcnt(25)\n"
      "Ltw279_%=:\n"
      "s_waitcnt vmcnt(29)\n"
      "s_barrier\n"
      "s_waitcnt lgkmcnt(0)\n"
      "v_mfma_f32_16x16x128_f8f6f4 v[128:131], v[96:103], a[32:39], v[128:131] cbsz:1\n"
      "ds_read_b64_tr_b8 v[112:113], v90 offset:16384\n"
      "s_min_u32 s56, s56, s57\n"
      "s_min_u32 s58, s58, s59\n"
      "s_min_u32 s60, s60, s61\n"
      "s_min_u32 s62, s62, s63\n"
      "v_mfma_f32_16x16x128_f8f6f4 v[132:135], v[96:103], a[40:47], v[132:135] cbsz:1\n"
      "ds_read_b64_tr_b8 v[114:115], v90 offset:17408\n"
      "s_min_u32 s56, s56, s58\n"
      "s_min_u32 s60, s60, s62\n"
      "s_min_u32 s56, s56, s60\n"
      "s_cmp_ge_u32 s56, s68\n"
      "s_cbranch_scc0 Lsl280_%=\n"
      "Lrt281_%=:\n"
      "v_mfma_f32_16x16x128_f8f6f4 v[136:139], v[96:103], a[48:55], v[136:139] cbsz:1\n"
      "ds_read_b64_tr_b8 v[116:117], v90 offset:24576\n"
      "s_add_u32 s70, s70, 128\n"
      "s_min_u32 s70, s70, %[sfl]\n"
      "s_lshl_b32 s89, s89, 2\n"
      "s_load_dwordx4 s[56:59], %[pfl], s70\n"
      "s_load_dwordx4 s[60:63], %[pfl], s70 offset:64\n"
      "v_readlane_b32 s90, v54, 30\n"
      "s_and_b32 s66, s90, 0xfffe0000\n"
      "s_and_b32 m0, s90, 0x1ffff\n"
      "s_nop 0\n"
      "buffer_load_dwordx4 v84, %[wsrd], s66 offen offset:0  lds\n"
      "v_mfma_f32_16x16x128_f8f6f4 v[140:143], v[96:103], a[56:63], v[140:143] cbsz:1\n"
      "ds_read_b64_tr_b8 v[118:119], v90 offset:25600\n"
      "s_waitcnt lgkmcnt(4)\n"
      "v_mfma_f32_16x16x128_f8f6f4 v[156:159], v[104:111], a[56:63], v[156:159] cbsz:1\n"
      "ds_read_b64_tr_b8 v[120:121], v91 offset:16384\n"
      "buffer_load_dwordx4 v85, %[wsrd], s66 offen offset:1024  lds\n"
      "v_mfma_f32_16x16x128_f8f6f4 v[152:155], v[104:111], a[48:55], v[152:155] cbsz:1\n"
      "ds_read_b64_tr_b8 v[122:123], v91 offset:17408\n"
      "v_mfma_f32_16x16x128_f8f6f4 v[148:151], v[104:111], a[40:47], v[148:151] cbsz:1\n"
      "ds_read_b64_tr_b8 v[124:125], v91 offset:24576\n"
      "buffer_load_dwordx4 v86, %[wsrd], s66 offen offset:2048  lds\n"
      "v_mfma_f32_16x16x128_f8f6f4 v[144:147], v[104:111], a[32:39], v[144:147] cbsz:1\n"
      "ds_read_b64_tr_b8 v[126:127], v91 offset:25600\n"
      "s_waitcnt lgkmcnt(4)\n"
      "v_mfma_f32_16x16x128_f8f6f4 v[160:163], v[112:119], a[32:39], v[160:163] cbsz:1\n"
      "ds_read_b64_tr_b8 v[96:97], v92 offset:16384\n"
      "buffer_load_dwordx4 v87, %[wsrd], s66 offen offset:3072  lds\n"
      "v_mfma_f32_16x16x128_f8f6f4 v[164:167], v[112:119], a[40:47], v[164:167] cbsz:1\n"
      "ds_read_b64_tr_b8 v[98:99], v92 offset:17408\n"
      "v_mfma_f32_16x16x128_f8f6f4 v[168:171], v[112:119], a[48:55], v[168:171] cbsz:1\n"
      "ds_read_b64_tr_b8 v[100:101], v92 offset:24576\n"
      "v_mfma_f32_16x16x128_f8f6f4 v[172:175], v[112:119], a[56:63], v[172:175] cbsz:1\n"
      "ds_read_b64_tr_b8 v[102:103], v92 offset:25600\n"
      "s_waitcnt lgkmcnt(4)\n"
      "v_mfma_f32_16x16x128_f8f6f4 v[188:191], v[120:127], a[56:63], v[188:191] cbsz:1\n"
      "ds_read_b64_tr_b8 v[104:105], v93 offset:16384\n"
      "v_mfma_f32_16x16x128_f8f6f4 v[184:187], v[120:127], a[48:55], v[184:187] cbsz:1\n"
      "ds_read_b64_tr_b8 v[106:107], v93 offset:17408\n"
      "v_mfma_f32_16x16x128_f8f6f4 v[180:183], v[120:127], a[40:47], v[180:183] cbsz:1\n"
      "ds_read_b64_tr_b8 v[108:109], v93 offset:24576\n"
      "v_readlane_b32 s65, v63, 28\n"
      "s_nop 1\n"
      "v_mfma_f32_16x16x128_f8f6f4 v[176:179], v[120:127], a[32:39], v[176:179] cbsz:1\n"
      "ds_read_b64_tr_b8 v[110:111], v93 offset:25600\n"
      "s_waitcnt lgkmcnt(4)\n"
      "v_mfma_f32_16x16x128_f8f6f4 v[192:195], v[96:103], a[32:39], v[192:195] cbsz:1\n"
      "ds_read_b64_tr_b8 v[112:113], v94 offset:16384\n"
      "buffer_load_dwordx4 a[0:3], %[va0], %[asrd], s65 offen offset:0 sc0 sc1\n"
      "v_mfma_f32_16x16x128_f8f6f4 v[196:199], v[96:103], a[40:47], v[196:199] cbsz:1\n"
      "ds_read_b64_tr_b8 v[114:115], v94 offset:17408\n"
      "v_mfma_f32_16x16x128_f8f6f4 v[200:203], v[96:103], a[48:55], v[200:203] cbsz:1\n"
      "ds_read_b64_tr_b8 v[116:117], v94 offset:24576\n"
      "buffer_load_dwordx4 a[4:7], %[va0], %[asrd], s65 offen offset:1024 sc0 sc1\n"
      "v_mfma_f32_16x16x128_f8f6f4 v[204:207], v[96:103], a[56:63], v[204:207] cbsz:1\n"
      "ds_read_b64_tr_b8 v[118:119], v94 offset:25600\n"
      "s_waitcnt lgkmcnt(4)\n"
      "v_mfma_f32_16x16x128_f8f6f4 v[220:223], v[104:111], a[56:63], v[220:223] cbsz:1\n"
      "ds_read_b64_tr_b8 v[120:121], v95 offset:16384\n"
      "buffer_load_dwordx4 a[8:11], %[va0], %[asrd], s65 offen offset:2048 sc0 sc1\n"
      "v_mfma_f32_16x16x128_f8f6f4 v[216:219], v[104:111], a[48:55], v[216:219] cbsz:1\n"
      "ds_read_b64_tr_b8 v[122:123], v95 offset:17408\n"
      "v_mfma_f32_16x16x128_f8f6f4 v[212:215], v[104:111], a[40:47], v[212:215] cbsz:1\n"
      "ds_read_b64_tr_b8 v[124:125], v95 offset:24576\n"
      "buffer_load_dwordx4 a[12:15], %[va0], %[asrd], s65 offen offset:3072 sc0 sc1\n"
      "v_mfma_f32_16x16x128_f8f6f4 v[208:211], v[104:111], a[32:39], v[208:211] cbsz:1\n"
      "ds_read_b64_tr_b8 v[126:127], v95 offset:25600\n"
      "s_waitcnt lgkmcnt(4)\n"
      "v_mfma_f32_16x16x128_f8f6f4 v[224:227], v[112:119], a[32:39], v[224:227] cbsz:1\n"
      "ds_read_b64_tr_b8 v[96:97], v88 offset:32768\n"
      "buffer_load_dwordx4 a[16:19], v59, %[asrd], s65 offen offset:0 sc0 sc1\n"
      "v_mfma_f32_16x16x128_f8f6f4 v[228:231], v[112:119], a[40:47], v[228:231] cbsz:1\n"
      "ds_read_b64_tr_b8 v[98:99], v88 offset:33792\n"
      "v_mfma_f32_16x16x128_f8f6f4 v[232:235], v[112:119], a[48:55], v[232:235] cbsz:1\n"
      "ds_read_b64_tr_b8 v[100:101], v88 offset:40960\n"
      "buffer_load_dwordx4 a[20:23], v59, %[asrd], s65 offen offset:1024 sc0 sc1\n"
      "v_mfma_f32_16x16x128_f8f6f4 v[236:239], v[112:119], a[56:63], v[236:239] cbsz:1\n"
      "ds_read_b64_tr_b8 v[102:103], v88 offset:41984\n"
      "s_waitcnt lgkmcnt(4)\n"
      "v_mfma_f32_16x16x128_f8f6f4 v[252:255], v[120:127], a[56:63], v[252:255] cbsz:1\n"
      "ds_read_b64_tr_b8 v[104:105], v89 offset:32768\n"
      "ds_read_b64_tr_b8 v[106:107], v89 offset:33792\n"
      "buffer_load_dwordx4 a[24:27], v59, %[asrd], s65 offen offset:2048 sc0 sc1\n"
      "v_mfma_f32_16x16x128_f8f6f4 v[248:251], v[120:127], a[48:55], v[248:251] cbsz:1\n"
      "ds_read_b64_tr_b8 v[108:109], v89 offset:40960\n"
      "ds_read_b64_tr_b8 v[110:111], v89 offset:41984\n"
      "v_mfma_f32_16x16x128_f8f6f4 v[244:247], v[120:127], a[40:47], v[244:247] cbsz:1\n"
      "buffer_load_dwordx4 a[28:31], v59, %[asrd], s65 offen offset:3072 sc0 sc1\n"
      "v_mfma_f32_16x16x128_f8f6f4 v[240:243], v[120:127], a[32:39], v[240:243] cbsz:1\n"
      "Ls26_%=:\n"
      "s_waitcnt vmcnt(25)\n"
      "s_barrier\n"
      "s_waitcnt lgkmcnt(0)\n"
      "v_mfma_f32_16x16x128_f8f6f4 v[128:131], v[96:103], a[64:71], v[128:131] cbsz:1\n"
      "ds_read_b64_tr_b8 v[112:113], v90 offset:32768\n"
      "s_min_u32 s56, s56, s57\n"
      "s_min_u32 s58, s58, s59\n"
      "s_min_u32 s60, s60, s61\n"
      "s_min_u32 s62, s62, s63\n"
      "v_mfma_f32_16x16x128_f8f6f4 v[132:135], v[96:103], a[72:79], v[132:135] cbsz:1\n"
      "ds_read_b64_tr_b8 v[114:115], v90 offset:33792\n"
      "s_min_u32 s56, s56, s58\n"
      "s_min_u32 s60, s60, s62\n"
      "s_min_u32 s56, s56, s60\n"
      "s_cmp_ge_u32 s56, s68\n"
      "s_cbranch_scc0 Lsl282_%=\n"
      "Lrt283_%=:\n"
      "v_readlane_b32 s65, v63, 29\n"
      "s_nop 1\n"
      "v_mfma_f32_16x16x128_f8f6f4 v[136:139], v[96:103], a[80:87], v[136:139] cbsz:1\n"
      "ds_read_b64_tr_b8 v[116:117], v90 offset:40960\n"
      "s_add_u32 s70, s70, 128\n"
      "s_min_u32 s70, s70, %[sfl]\n"
      "s_lshl_b32 s89, s89, 2\n"
      "s_load_dwordx4 s[56:59], %[pfl], s70\n"
      "s_load_dwordx4 s[60:63], %[pfl], s70 offset:64\n"
      "v_mfma_f32_16x16x128_f8f6f4 v[140:143], v[96:103], a[88:95], v[140:143] cbsz:1\n"
      "ds_read_b64_tr_b8 v[118:119], v90 offset:41984\n"
      "buffer_load_dwordx4 a[32:35], %[va0], %[asrd], s65 offen offset:0 sc0 sc1\n"
      "s_waitcnt lgkmcnt(4)\n"
      "v_mfma_f32_16x16x128_f8f6f4 v[156:159], v[104:111], a[88:95], v[156:159] cbsz:1\n"
      "ds_read_b64_tr_b8 v[120:121], v91 offset:32768\n"
      "v_mfma_f32_16x16x128_f8f6f4 v[152:155], v[104:111], a[80:87], v[152:155] cbsz:1\n"
      "ds_read_b64_tr_b8 v[122:123], v91 offset:33792\n"
      "buffer_load_dwordx4 a[36:39], %[va0], %[asrd], s65 offen offset:1024 sc0 sc1\n"
      "v_mfma_f32_16x16x128_f8f6f4 v[148:151], v[104:111], a[72:79], v[148:151] cbsz:1\n"
      "ds_read_b64_tr_b8 v[124:125], v91 offset:40960\n"
      "v_mfma_f32_16x16x128_f8f6f4 v[144:147], v[104:111], a[64:71], v[144:147] cbsz:1\n"
      "ds_read_b64_tr_b8 v[126:127], v91 offset:41984\n"
      "buffer_load_dwordx4 a[40:43], %[va0], %[asrd], s65 offen offset:2048 sc0 sc1\n"
      "s_waitcnt lgkmcnt(4)\n"
      "v_mfma_f32_16x16x128_f8f6f4 v[160:163], v[112:119], a[64:71], v[160:163] cbsz:1\n"
      "ds_read_b64_tr_b8 v[96:97], v92 offset:32768\n"
      "v_mfma_f32_16x16x128_f8f6f4 v[164:167], v[112:119], a[72:79], v[164:167] cbsz:1\n"
      "ds_read_b64_tr_b8 v[98:99], v92 offset:33792\n"
      "buffer_load_dwordx4 a[44:47], %[va0], %[asrd], s65 offen offset:3072 sc0 sc1\n"
      "v_mfma_f32_16x16x128_f8f6f4 v[168:171], v[112:119], a[80:87], v[168:171] cbsz:1\n"
      "ds_read_b64_tr_b8 v[100:101], v92 offset:40960\n"
      "v_mfma_f32_16x16x128_f8f6f4 v[172:175], v[112:119], a[88:95], v[172:175] cbsz:1\n"
      "ds_read_b64_tr_b8 v[102:103], v92 offset:41984\n"
      "buffer_load_dwordx4 a[48:51], v59, %[asrd], s65 offen offset:0 sc0 sc1\n"
      "s_waitcnt lgkmcnt(4)\n"
      "v_mfma_f32_16x16x128_f8f6f4 v[188:191], v[120:127], a[88:95], v[188:191] cbsz:1\n"
      "ds_read_b64_tr_b8 v[104:105], v93 offset:32768\n"
      "v_mfma_f32_16x16x128_f8f6f4 v[184:187], v[120:127], a[80:87], v[184:187] cbsz:1\n"
      "ds_read_b64_tr_b8 v[106:107], v93 offset:33792\n"
      "buffer_load_dwordx4 a[52:55], v59, %[asrd], s65 offen offset:1024 sc0 sc1\n"
      "v_mfma_f32_16x16x128_f8f6f4 v[180:183], v[120:127], a[72:79], v[180:183] cbsz:1\n"
      "ds_read_b64_tr_b8 v[108:109], v93 offset:40960\n"
      "v_mfma_f32_16x16x128_f8f6f4 v[176:179], v[120:127], a[64:71], v[176:179] cbsz:1\n"
      "ds_read_b64_tr_b8 v[110:111], v93 offset:41984\n"
      "buffer_load_dwordx4 a[56:59], v59, %[asrd], s65 offen offset:2048 sc0 sc1\n"
      "s_waitcnt lgkmcnt(4)\n"
      "v_mfma_f32_16x16x128_f8f6f4 v[192:195], v[96:103], a[64:71], v[192:195] cbsz:1\n"
      "ds_read_b64_tr_b8 v[112:113], v94 offset:32768\n"
      "v_mfma_f32_16x16x128_f8f6f4 v[196:199], v[96:103], a[72:79], v[196:199] cbsz:1\n"
      "ds_read_b64_tr_b8 v[114:115], v94 offset:33792\n"
      "buffer_load_dwordx4 a[60:63], v59, %[asrd], s65 offen offset:3072 sc0 sc1\n"
      "v_readlane_b32 s65, v63, 30\n"
      "s_nop 1\n"
      "v_mfma_f32_16x16x128_f8f6f4 v[200:203], v[96:103], a[80:87], v[200:203] cbsz:1\n"
      "ds_read_b64_tr_b8 v[116:117], v94 offset:40960\n"
      "v_mfma_f32_16x16x128_f8f6f4 v[204:207], v[96:103], a[88:95], v[204:207] cbsz:1\n"
      "ds_read_b64_tr_b8 v[118:119], v94 offset:41984\n"
      "buffer_load_dwordx4 a[128:131], %[va0], %[asrd], s65 offen offset:0 sc0 sc1\n"
      "s_waitcnt lgkmcnt(4)\n"
      "v_mfma_f32_16x16x128_f8f6f4 v[220:223], v[104:111], a[88:95], v[220:223] cbsz:1\n"
      "ds_read_b64_tr_b8 v[120:121], v95 offset:32768\n"
      "v_mfma_f32_16x16x128_f8f6f4 v[216:219], v[104:111], a[80:87], v[216:219] cbsz:1\n"
      "ds_read_b64_tr_b8 v[122:123], v95 offset:33792\n"
      "buffer_load_dwordx4 a[132:135], %[va0], %[asrd], s65 offen offset:1024 sc0 sc1\n"
      "v_mfma_f32_16x16x128_f8f6f4 v[212:215], v[104:111], a[72:79], v[212:215] cbsz:1\n"
      "ds_read_b64_tr_b8 v[124:125], v95 offset:40960\n"
      "v_mfma_f32_16x16x128_f8f6f4 v[208:211], v[104:111], a[64:71], v[208:211] cbsz:1\n"
      "ds_read_b64_tr_b8 v[126:127], v95 offset:41984\n"
      "v_readlane_b32 s90, v54, 31\n"
      "s_and_b32 s66, s90, 0xfffe0000\n"
      "s_and_b32 m0, s90, 0x1ffff\n"
      "s_nop 0\n"
      "buffer_load_dwordx4 v84, %[wsrd], s66 offen offset:0  lds\n"
      "s_waitcnt lgkmcnt(4)\n"
      "v_mfma_f32_16x16x128_f8f6f4 v[224:227], v[112:119], a[64:71], v[224:227] cbsz:1\n"
      "ds_read_b64_tr_b8 v[96:97], v88 offset:49152\n"
      "v_mfma_f32_16x16x128_f8f6f4 v[228:231], v[112:119], a[72:79], v[228:231] cbsz:1\n"
      "ds_read_b64_tr_b8 v[98:99], v88 offset:50176\n"
      "buffer_load_dwordx4 v85, %[wsrd], s66 offen offset:1024  lds\n"
      "v_mfma_f32_16x16x128_f8f6f4 v[232:235], v[112:119], a[80:87], v[232:235] cbsz:1\n"
      "ds_read_b64_tr_b8 v[100:101], v88 offset:57344\n"
      "v_mfma_f32_16x16x128_f8f6f4 v[236:239], v[112:119], a[88:95], v[236:239] cbsz:1\n"
      "ds_read_b64_tr_b8 v[102:103], v88 offset:58368\n"
      "v_xor_b32_e32 v88, 0x10000, v88\n"
      "buffer_load_dwordx4 v86, %[wsrd], s66 offen offset:2048  lds\n"
      "s_waitcnt lgkmcnt(4)\n"
      "v_mfma_f32_16x16x128_f8f6f4 v[252:255], v[120:127], a[88:95], v[252:255] cbsz:1\n"
      "ds_read_b64_tr_b8 v[104:105], v89 offset:49152\n"
      "v_mfma_f32_16x16x128_f8f6f4 v[248:251], v[120:127], a[80:87], v[248:251] cbsz:1\n"
      "ds_read_b64_tr_b8 v[106:107], v89 offset:50176\n"
      "buffer_load_dwordx4 v87, %[wsrd], s66 offen offset:3072  lds\n"
      "v_mfma_f32_16x16x128_f8f6f4 v[244:247], v[120:127], a[72:79], v[244:247] cbsz:1\n"
      "ds_read_b64_tr_b8 v[108:109], v89 offset:57344\n"
      "v_mfma_f32_16x16x128_f8f6f4 v[240:243], v[120:127], a[64:71], v[240:243] cbsz:1\n"
      "ds_read_b64_tr_b8 v[110:111], v89 offset:58368\n"
      "v_xor_b32_e32 v89, 0x10000, v89\n"
      "Ls27_%=:\n"
      "s_waitcnt vmcnt(26)\n"
      "s_barrier\n"
      "s_waitcnt lgkmcnt(4)\n"
      "v_mfma_f32_16x16x128_f8f6f4 v[128:131], v[96:103], a[96:103], v[128:131] cbsz:1\n"
      "ds_read_b64_tr_b8 v[112:113], v90 offset:49152\n"
      "v_mfma_f32_16x16x128_f8f6f4 v[132:135], v[96:103], a[104:111], v[132:135] cbsz:1\n"
      "ds_read_b64_tr_b8 v[114:115], v90 offset:50176\n"
      "buffer_load_dwordx4 a[136:139], %[va0], %[asrd], s65 offen offset:2048 sc0 sc1\n"
      "v_mfma_f32_16x16x128_f8f6f4 v[136:139], v[96:103], a[112:119], v[136:139] cbsz:1\n"
      "ds_read_b64_tr_b8 v[116:117], v90 offset:57344\n"
      "v_mfma_f32_16x16x128_f8f6f4 v[140:143], v[96:103], a[120:127], v[140:143] cbsz:1\n"
      "ds_read_b64_tr_b8 v[118:119], v90 offset:58368\n"
      "v_xor_b32_e32 v90, 0x10000, v90\n"
      "buffer_load_dwordx4 a[140:143], %[va0], %[asrd], s65 offen offset:3072 sc0 sc1\n"
      "s_waitcnt lgkmcnt(4)\n"
      "v_mfma_f32_16x16x128_f8f6f4 v[156:159], v[104:111], a[120:127], v[156:159] cbsz:1\n"
      "ds_read_b64_tr_b8 v[120:121], v91 offset:49152\n"
      "v_mfma_f32_16x16x128_f8f6f4 v[152:155], v[104:111], a[112:119], v[152:155] cbsz:1\n"
      "ds_read_b64_tr_b8 v[122:123], v91 offset:50176\n"
      "buffer_load_dwordx4 a[144:147], v59, %[asrd], s65 offen offset:0 sc0 sc1\n"
      "v_mfma_f32_16x16x128_f8f6f4 v[148:151], v[104:111], a[104:111], v[148:151] cbsz:1\n"
      "ds_read_b64_tr_b8 v[124:125], v91 offset:57344\n"
      "v_mfma_f32_16x16x128_f8f6f4 v[144:147], v[104:111], a[96:103], v[144:147] cbsz:1\n"
      "ds_read_b64_tr_b8 v[126:127], v91 offset:58368\n"
      "v_xor_b32_e32 v91, 0x10000, v91\n"
      "buffer_load_dwordx4 a[148:151], v59, %[asrd], s65 offen offset:1024 sc0 sc1\n"
      "s_waitcnt lgkmcnt(4)\n"
      "v_mfma_f32_16x16x128_f8f6f4 v[160:163], v[112:119], a[96:103], v[160:163] cbsz:1\n"
      "ds_read_b64_tr_b8 v[96:97], v92 offset:49152\n"
      "v_mfma_f32_16x16x128_f8f6f4 v[164:167], v[112:119], a[104:111], v[164:167] cbsz:1\n"
      "ds_read_b64_tr_b8 v[98:99], v92 offset:50176\n"
      "buffer_load_dwordx4 a[152:155], v59, %[asrd], s65 offen offset:2048 sc0 sc1\n"
      "v_mfma_f32_16x16x128_f8f6f4 v[168:171], v[112:119], a[112:119], v[168:171] cbsz:1\n"
      "ds_read_b64_tr_b8 v[100:101], v92 offset:57344\n"
      "v_mfma_f32_16x16x128_f8f6f4 v[172:175], v[112:119], a[120:127], v[172:175] cbsz:1\n"
      "ds_read_b64_tr_b8 v[102:103], v92 offset:58368\n"
      "v_xor_b32_e32 v92, 0x10000, v92\n"
      "buffer_load_dwordx4 a[156:159], v59, %[asrd], s65 offen offset:3072 sc0 sc1\n"
      "s_waitcnt lgkmcnt(4)\n"
      "v_mfma_f32_16x16x128_f8f6f4 v[188:191], v[120:127], a[120:127], v[188:191] cbsz:1\n"
      "ds_read_b64_tr_b8 v[104:105], v93 offset:49152\n"
      "v_readlane_b32 s65, v63, 31\n"
      "s_nop 1\n"
      "v_mfma_f32_16x16x128_f8f6f4 v[184:187], v[120:127], a[112:119], v[184:187] cbsz:1\n"
      "ds_read_b64_tr_b8 v[106:107], v93 offset:50176\n"
      "v_mfma_f32_16x16x128_f8f6f4 v[180:183], v[120:127], a[104:111], v[180:183] cbsz:1\n"
      "ds_read_b64_tr_b8 v[108:109], v93 offset:57344\n"
      "buffer_load_dwordx4 a[160:163], %[va0], %[asrd], s65 offen offset:0 sc0 sc1\n"
      "v_mfma_f32_16x16x128_f8f6f4 v[176:179], v[120:127], a[96:103], v[176:179] cbsz:1\n"
      "ds_read_b64_tr_b8 v[110:111], v93 offset:58368\n"
      "v_xor_b32_e32 v93, 0x10000, v93\n"
      "s_waitcnt lgkmcnt(4)\n"
      "v_mfma_f32_16x16x128_f8f6f4 v[192:195], v[96:103], a[96:103], v[192:195] cbsz:1\n"
      "ds_read_b64_tr_b8 v[112:113], v94 offset:49152\n"
      "buffer_load_dwordx4 a[164:167], %[va0], %[asrd], s65 offen offset:1024 sc0 sc1\n"
      "v_mfma_f32_16x16x128_f8f6f4 v[196:199], v[96:103], a[104:111], v[196:199] cbsz:1\n"
      "ds_read_b64_tr_b8 v[114:115], v94 offset:50176\n"
      "v_mfma_f32_16x16x128_f8f6f4 v[200:203], v[96:103], a[112:119], v[200:203] cbsz:1\n"
      "ds_read_b64_tr_b8 v[116:117], v94 offset:57344\n"
      "buffer_load_dwordx4 a[168:171], %[va0], %[asrd], s65 offen offset:2048 sc0 sc1\n"
      "v_mfma_f32_16x16x128_f8f6f4 v[204:207], v[96:103], a[120:127], v[204:207] cbsz:1\n"
      "ds_read_b64_tr_b8 v[118:119], v94 offset:58368\n"
      "v_xor_b32_e32 v94, 0x10000, v94\n"
      "s_waitcnt lgkmcnt(4)\n"
      "v_mfma_f32_16x16x128_f8f6f4 v[220:223], v[104:111], a[120:127], v[220:223] cbsz:1\n"
      "ds_read_b64_tr_b8 v[120:121], v95 offset:49152\n"
      "buffer_load_dwordx4 a[172:175], %[va0], %[asrd], s65 offen offset:3072 sc0 sc1\n"
      "v_mfma_f32_16x16x128_f8f6f4 v[216:219], v[104:111], a[112:119], v[216:219] cbsz:1\n"
      "ds_read_b64_tr_b8 v[122:123], v95 offset:50176\n"
      "v_mfma_f32_16x16x128_f8f6f4 v[212:215], v[104:111], a[104:111], v[212:215] cbsz:1\n"
      "ds_read_b64_tr_b8 v[124:125], v95 offset:57344\n"
      "buffer_load_dwordx4 a[176:179], v59, %[asrd], s65 offen offset:0 sc0 sc1\n"
      "v_mfma_f32_16x16x128_f8f6f4 v[208:211], v[104:111], a[96:103], v[208:211] cbsz:1\n"
      "ds_read_b64_tr_b8 v[126:127], v95 offset:58368\n"
      "v_xor_b32_e32 v95, 0x10000, v95\n"
      "s_waitcnt lgkmcnt(4)\n"
      "v_mfma_f32_16x16x128_f8f6f4 v[224:227], v[112:119], a[96:103], v[224:227] cbsz:1\n"
      "ds_read_b64_tr_b8 v[96:97], v88 offset:0\n"
      "buffer_load_dwordx4 a[180:183], v59, %[asrd], s65 offen offset:1024 sc0 sc1\n"
      "v_mfma_f32_16x16x128_f8f6f4 v[228:231], v[112:119], a[104:111], v[228:231] cbsz:1\n"
      "ds_read_b64_tr_b8 v[98:99], v88 offset:1024\n"
      "v_mfma_f32_16x16x128_f8f6f4 v[232:235], v[112:119], a[112:119], v[232:235] cbsz:1\n"
      "ds_read_b64_tr_b8 v[100:101], v88 offset:8192\n"
      "buffer_load_dwordx4 a[184:187], v59, %[asrd], s65 offen offset:2048 sc0 sc1\n"
      "v_mfma_f32_16x16x128_f8f6f4 v[236:239], v[112:119], a[120:127], v[236:239] cbsz:1\n"
      "ds_read_b64_tr_b8 v[102:103], v88 offset:9216\n"
      "s_waitcnt lgkmcnt(4)\n"
      "v_mfma_f32_16x16x128_f8f6f4 v[252:255], v[120:127], a[120:127], v[252:255] cbsz:1\n"
      "ds_read_b64_tr_b8 v[104:105], v89 offset:0\n"
      "buffer_load_dwordx4 a[188:191], v59, %[asrd], s65 offen offset:3072 sc0 sc1\n"
      "v_mfma_f32_16x16x128_f8f6f4 v[248:251], v[120:127], a[112:119], v[248:251] cbsz:1\n"
      "ds_read_b64_tr_b8 v[106:107], v89 offset:1024\n"
      "v_mfma_f32_16x16x128_f8f6f4 v[244:247], v[120:127], a[104:111], v[244:247] cbsz:1\n"
      "ds_read_b64_tr_b8 v[108:109], v89 offset:8192\n"
      "v_mfma_f32_16x16x128_f8f6f4 v[240:243], v[120:127], a[96:103], v[240:243] cbsz:1\n"
      "ds_read_b64_tr_b8 v[110:111], v89 offset:9216\n"
      "s_waitcnt vmcnt(14)\n"
      "s_barrier\n"
      "s_waitcnt lgkmcnt(4)\n"
      "v_mfma_f32_16x16x128_f8f6f4 v[128:131], v[96:103], a[0:7], v[128:131] cbsz:1\n"
      "ds_read_b64_tr_b8 v[112:113], v88 offset:16384\n"
      "v_mfma_f32_16x16x128_f8f6f4 v[132:135], v[96:103], a[8:15], v[132:135] cbsz:1\n"
      "ds_read_b64_tr_b8 v[114:115], v88 offset:17408\n"
      "v_mul_f32_e32 v61, v61, v60\n"
      "v_or_b32_e32 v61, s83, v61\n"
      "v_mfma_f32_16x16x128_f8f6f4 v[136:139], v[96:103], a[16:23], v[136:139] cbsz:1\n"
      "ds_read_b64_tr_b8 v[116:117], v88 offset:24576\n"
      "v_mov_b32_e32 v80, %[vew]\n"
      "v_xor_b32_e32 v81, 64, %[vew]\n"
      "v_mov_b32_e32 v82, %[ver]\n"
      "v_mfma_f32_16x16x128_f8f6f4 v[140:143], v[96:103], a[24:31], v[140:143] cbsz:1\n"
      "ds_read_b64_tr_b8 v[118:119], v88 offset:25600\n"
      "s_waitcnt lgkmcnt(4)\n"
      "v_mfma_f32_16x16x128_f8f6f4 v[156:159], v[104:111], a[24:31], v[156:159] cbsz:1\n"
      "ds_read_b64_tr_b8 v[120:121], v89 offset:16384\n"
      "v_mfma_f32_16x16x128_f8f6f4 v[152:155], v[104:111], a[16:23], v[152:155] cbsz:1\n"
      "ds_read_b64_tr_b8 v[122:123], v89 offset:17408\n"
      "v_mfma_f32_16x16x128_f8f6f4 v[148:151], v[104:111], a[8:15], v[148:151] cbsz:1\n"
      "ds_read_b64_tr_b8 v[124:125], v89 offset:24576\n"
      "v_mfma_f32_16x16x128_f8f6f4 v[144:147], v[104:111], a[0:7], v[144:147] cbsz:1\n"
      "ds_read_b64_tr_b8 v[126:127], v89 offset:25600\n"
      "s_waitcnt vmcnt(20)\n"
      "s_waitcnt lgkmcnt(4)\n"
      "v_mfma_f32_16x16x128_f8f6f4 v[128:131], v[112:119], a[32:39], v[128:131] cbsz:1\n"
      "ds_read_b64_tr_b8 v[96:97], v88 offset:32768\n"
      "v_mfma_f32_16x16x128_f8f6f4 v[132:135], v[112:119], a[40:47], v[132:135] cbsz:1\n"
      "ds_read_b64_tr_b8 v[98:99], v88 offset:33792\n"
      "v_mfma_f32_16x16x128_f8f6f4 v[136:139], v[112:119], a[48:55], v[136:139] cbsz:1\n"
      "ds_read_b64_tr_b8 v[100:101], v88 offset:40960\n"
      "v_mfma_f32_16x16x128_f8f6f4 v[140:143], v[112:119], a[56:63], v[140:143] cbsz:1\n"
      "ds_read_b64_tr_b8 v[102:103], v88 offset:41984\n"
      "s_waitcnt lgkmcnt(4)\n"
      "v_mfma_f32_16x16x128_f8f6f4 v[156:159], v[120:127], a[56:63], v[156:159] cbsz:1\n"
      "ds_read_b64_tr_b8 v[104:105], v89 offset:32768\n"
      "v_mfma_f32_16x16x128_f8f6f4 v[152:155], v[120:127], a[48:55], v[152:155] cbsz:1\n"
      "ds_read_b64_tr_b8 v[106:107], v89 offset:33792\n"
      "v_mfma_f32_16x16x128_f8f6f4 v[148:151], v[120:127], a[40:47], v[148:151] cbsz:1\n"
      "ds_read_b64_tr_b8 v[108:109], v89 offset:40960\n"
      "v_mfma_f32_16x16x128_f8f6f4 v[144:147], v[120:127], a[32:39], v[144:147] cbsz:1\n"
      "ds_read_b64_tr_b8 v[110:111], v89 offset:41984\n"
      "s_waitcnt vmcnt(8)\n"
      "s_waitcnt lgkmcnt(4)\n"
      "v_mfma_f32_16x16x128_f8f6f4 v[128:131], v[96:103], a[128:135], v[128:131] cbsz:1\n"
      "ds_read_b64_tr_b8 v[112:113], v88 offset:49152\n"
      "v_mfma_f32_16x16x128_f8f6f4 v[132:135], v[96:103], a[136:143], v[132:135] cbsz:1\n"
      "ds_read_b64_tr_b8 v[114:115], v88 offset:50176\n"
      "v_mfma_f32_16x16x128_f8f6f4 v[136:139], v[96:103], a[144:151], v[136:139] cbsz:1\n"
      "ds_read_b64_tr_b8 v[116:117], v88 offset:57344\n"
      "v_mfma_f32_16x16x128_f8f6f4 v[140:143], v[96:103], a[152:159], v[140:143] cbsz:1\n"
      "ds_read_b64_tr_b8 v[118:119], v88 offset:58368\n"
      "s_waitcnt lgkmcnt(4)\n"
      "v_mfma_f32_16x16x128_f8f6f4 v[156:159], v[104:111], a[152:159], v[156:159] cbsz:1\n"
      "ds_read_b64_tr_b8 v[120:121], v89 offset:49152\n"
      "v_mfma_f32_16x16x128_f8f6f4 v[152:155], v[104:111], a[144:151], v[152:155] cbsz:1\n"
      "ds_read_b64_tr_b8 v[122:123], v89 offset:50176\n"
      "v_mfma_f32_16x16x128_f8f6f4 v[148:151], v[104:111], a[136:143], v[148:151] cbsz:1\n"
      "ds_read_b64_tr_b8 v[124:125], v89 offset:57344\n"
      "v_mfma_f32_16x16x128_f8f6f4 v[144:147], v[104:111], a[128:135], v[144:147] cbsz:1\n"
      "ds_read_b64_tr_b8 v[126:127], v89 offset:58368\n"
      "s_waitcnt vmcnt(0)\n"
      "s_waitcnt lgkmcnt(4)\n"
      "v_mfma_f32_16x16x128_f8f6f4 v[128:131], v[112:119], a[160:167], v[128:131] cbsz:1\n"
      "ds_read_b64_tr_b8 v[96:97], v90 offset:0\n"
      "v_mfma_f32_16x16x128_f8f6f4 v[132:135], v[112:119], a[168:175], v[132:135] cbsz:1\n"
      "ds_read_b64_tr_b8 v[98:99], v90 offset:1024\n"
      "v_mfma_f32_16x16x128_f8f6f4 v[136:139], v[112:119], a[176:183], v[136:139] cbsz:1\n"
      "ds_read_b64_tr_b8 v[100:101], v90 offset:8192\n"
      "v_mfma_f32_16x16x128_f8f6f4 v[140:143], v[112:119], a[184:191], v[140:143] cbsz:1\n"
      "ds_read_b64_tr_b8 v[102:103], v90 offset:9216\n"
      "s_waitcnt lgkmcnt(4)\n"
      "v_mfma_f32_16x16x128_f8f6f4 v[156:159], v[120:127], a[184:191], v[156:159] cbsz:1\n"
      "ds_read_b64_tr_b8 v[104:105], v91 offset:0\n"
      "v_mfma_f32_16x16x128_f8f6f4 v[152:155], v[120:127], a[176:183], v[152:155] cbsz:1\n"
      "ds_read_b64_tr_b8 v[106:107], v91 offset:1024\n"
      "v_mul_f32_e32 v128, v61, v128\n"
      "v_mul_f32_e32 v129, v61, v129\n"
      "v_mul_f32_e32 v130, v61, v130\n"
      "v_mul_f32_e32 v131, v61, v131\n"
      "ds_write_b128 v80, v[128:131] offset:0\n"
      "v_mfma_f32_16x16x128_f8f6f4 v[148:151], v[120:127], a[168:175], v[148:151] cbsz:1\n"
      "ds_read_b64_tr_b8 v[108:109], v91 offset:8192\n"
      "v_mul_f32_e32 v132, v61, v132\n"
      "v_mul_f32_e32 v133, v61, v133\n"
      "v_mul_f32_e32 v134, v61, v134\n"
      "v_mul_f32_e32 v135, v61, v135\n"
      "ds_write_b128 v80, v[132:135] offset:2048\n"
      "v_mfma_f32_16x16x128_f8f6f4 v[144:147], v[120:127], a[160:167], v[144:147] cbsz:1\n"
      "ds_read_b64_tr_b8 v[110:111], v91 offset:9216\n"
      "v_mul_f32_e32 v136, v61, v136\n"
      "v_mul_f32_e32 v137, v61, v137\n"
      "v_mul_f32_e32 v138, v61, v138\n"
      "v_mul_f32_e32 v139, v61, v139\n"
      "ds_write_b128 v80, v[136:139] offset:4096\n"
      "s_waitcnt lgkmcnt(7)\n"
      "v_mfma_f32_16x16x128_f8f6f4 v[160:163], v[96:103], a[0:7], v[160:163] cbsz:1\n"
      "ds_read_b64_tr_b8 v[112:113], v90 offset:16384\n"
      "v_mul_f32_e32 v140, v61, v140\n"
      "v_mul_f32_e32 v141, v61, v141\n"
      "v_mul_f32_e32 v142, v61, v142\n"
      "v_mul_f32_e32 v143, v61, v143\n"
      "ds_write_b128 v80, v[140:143] offset:6144\n"
      "v_mfma_f32_16x16x128_f8f6f4 v[164:167], v[96:103], a[8:15], v[164:167] cbsz:1\n"
      "ds_read_b64_tr_b8 v[114:115], v90 offset:17408\n"
      "v_mul_f32_e32 v156, v61, v156\n"
      "v_mul_f32_e32 v157, v61, v157\n"
      "v_mul_f32_e32 v158, v61, v158\n"
      "v_mul_f32_e32 v159, v61, v159\n"
      "ds_write_b128 v81, v[156:159] offset:6144\n"
      "v_mfma_f32_16x16x128_f8f6f4 v[168:171], v[96:103], a[16:23], v[168:171] cbsz:1\n"
      "ds_read_b64_tr_b8 v[116:117], v90 offset:24576\n"
      "v_mul_f32_e32 v152, v61, v152\n"
      "v_mul_f32_e32 v153, v61, v153\n"
      "v_mul_f32_e32 v154, v61, v154\n"
      "v_mul_f32_e32 v155, v61, v155\n"
      "ds_write_b128 v81, v[152:155] offset:4096\n"
      "v_mfma_f32_16x16x128_f8f6f4 v[172:175], v[96:103], a[24:31], v[172:175] cbsz:1\n"
      "ds_read_b64_tr_b8 v[118:119], v90 offset:25600\n"
      "v_mul_f32_e32 v148, v61, v148\n"
      "v_mul_f32_e32 v149, v61, v149\n"
      "v_mul_f32_e32 v150, v61, v150\n"
      "v_mul_f32_e32 v151, v61, v151\n"
      "ds_write_b128 v81, v[148:151] offset:2048\n"
      "s_waitcnt lgkmcnt(9)\n"
      "v_mfma_f32_16x16x128_f8f6f4 v[188:191], v[104:111], a[24:31], v[188:191] cbsz:1\n"
      "ds_read_b64_tr_b8 v[120:121], v91 offset:16384\n"
      "v_mul_f32_e32 v144, v61, v144\n"
      "v_mul_f32_e32 v145, v61, v145\n"
      "v_mul_f32_e32 v146, v61, v146\n"
      "v_mul_f32_e32 v147, v61, v147\n"
      "ds_write_b128 v81, v[144:147] offset:0\n"
      "v_mfma_f32_16x16x128_f8f6f4 v[184:187], v[104:111], a[16:23], v[184:187] cbsz:1\n"
      "ds_read_b64_tr_b8 v[122:123], v91 offset:17408\n"
      "ds_read_b128 v[128:131], v82 offset:0\n"
      "ds_read_b128 v[132:135], v82 offset:1024\n"
      "ds_read_b128 v[136:139], v82 offset:2048\n"
      "ds_read_b128 v[140:143], v82 offset:3072\n"
      "v_mfma_f32_16x16x128_f8f6f4 v[180:183], v[104:111], a[8:15], v[180:183] cbsz:1\n"
      "ds_read_b64_tr_b8 v[124:125], v91 offset:24576\n"
      "ds_read_b128 v[144:147], v82 offset:4096\n"
      "ds_read_b128 v[148:151], v82 offset:5120\n"
      "ds_read_b128 v[152:155], v82 offset:6144\n"
      "ds_read_b128 v[156:159], v82 offset:7168\n"
      "v_mfma_f32_16x16x128_f8f6f4 v[176:179], v[104:111], a[0:7], v[176:179] cbsz:1\n"
      "ds_read_b64_tr_b8 v[126:127], v91 offset:25600\n"
      "s_waitcnt lgkmcnt(14)\n"
      "v_mfma_f32_16x16x128_f8f6f4 v[160:163], v[112:119], a[32:39], v[160:163] cbsz:1\n"
      "ds_read_b64_tr_b8 v[96:97], v90 offset:32768\n"
      "v_mfma_f32_16x16x128_f8f6f4 v[164:167], v[112:119], a[40:47], v[164:167] cbsz:1\n"
      "ds_read_b64_tr_b8 v[98:99], v90 offset:33792\n"
      "s_mov_b32 s71, 0\n"
      "s_waitcnt lgkmcnt(11)\n"
      "buffer_store_dwordx4 v[128:131], %[vout], %[osrd], s71 offen offset:0 sc0 sc1\n"
      "v_mfma_f32_16x16x128_f8f6f4 v[168:171], v[112:119], a[48:55], v[168:171] cbsz:1\n"
      "ds_read_b64_tr_b8 v[100:101], v90 offset:40960\n"
      "s_mov_b32 s71, 4194304\n"
      "s_waitcnt lgkmcnt(11)\n"
      "buffer_store_dwordx4 v[132:135], %[vout], %[osrd], s71 offen offset:0 sc0 sc1\n"
      "v_mfma_f32_16x16x128_f8f6f4 v[172:175], v[112:119], a[56:63], v[172:175] cbsz:1\n"
      "ds_read_b64_tr_b8 v[102:103], v90 offset:41984\n"
      "s_mov_b32 s71, 8388608\n"
      "s_waitcnt lgkmcnt(11)\n"
      "buffer_store_dwordx4 v[136:139], %[vout], %[osrd], s71 offen offset:0 sc0 sc1\n"
      "s_waitcnt lgkmcnt(4)\n"
      "v_mfma_f32_16x16x128_f8f6f4 v[188:191], v[120:127], a[56:63], v[188:191] cbsz:1\n"
      "ds_read_b64_tr_b8 v[104:105], v91 offset:32768\n"
      "s_mov_b32 s71, 12582912\n"
      "s_waitcnt lgkmcnt(11)\n"
      "buffer_store_dwordx4 v[140:143], %[vout], %[osrd], s71 offen offset:0 sc0 sc1\n"
      "v_mfma_f32_16x16x128_f8f6f4 v[184:187], v[120:127], a[48:55], v[184:187] cbsz:1\n"
      "ds_read_b64_tr_b8 v[106:107], v91 offset:33792\n"
      "s_mov_b32 s71, 16777216\n"
      "s_waitcnt lgkmcnt(10)\n"
      "buffer_store_dwordx4 v[144:147], %[vout], %[osrd], s71 offen offset:0 sc0 sc1\n"
      "v_mfma_f32_16x16x128_f8f6f4 v[180:183], v[120:127], a[40:47], v[180:183] cbsz:1\n"
      "ds_read_b64_tr_b8 v[108:109], v91 offset:40960\n"
      "s_mov_b32 s71, 20971520\n"
      "s_waitcnt lgkmcnt(10)\n"
      "buffer_store_dwordx4 v[148:151], %[vout], %[osrd], s71 offen offset:0 sc0 sc1\n"
      "v_mfma_f32_16x16x128_f8f6f4 v[176:179], v[120:127], a[32:39], v[176:179] cbsz:1\n"
      "ds_read_b64_tr_b8 v[110:111], v91 offset:41984\n"
      "s_mov_b32 s71, 25165824\n"
      "s_waitcnt lgkmcnt(10)\n"
      "buffer_store_dwordx4 v[152:155], %[vout], %[osrd], s71 offen offset:0 sc0 sc1\n"
      "s_waitcnt lgkmcnt(4)\n"
      "v_mfma_f32_16x16x128_f8f6f4 v[160:163], v[96:103], a[128:135], v[160:163] cbsz:1\n"
      "ds_read_b64_tr_b8 v[112:113], v90 offset:49152\n"
      "s_mov_b32 s71, 29360128\n"
      "s_waitcnt lgkmcnt(10)\n"
      "buffer_store_dwordx4 v[156:159], %[vout], %[osrd], s71 offen offset:0 sc0 sc1\n"
      "v_mfma_f32_16x16x128_f8f6f4 v[164:167], v[96:103], a[136:143], v[164:167] cbsz:1\n"
      "ds_read_b64_tr_b8 v[114:115], v90 offset:50176\n"
      "v_mfma_f32_16x16x128_f8f6f4 v[168:171], v[96:103], a[144:151], v[168:171] cbsz:1\n"
      "ds_read_b64_tr_b8 v[116:117], v90 offset:57344\n"
      "v_mfma_f32_16x16x128_f8f6f4 v[172:175], v[96:103], a[152:159], v[172:175] cbsz:1\n"
      "ds_read_b64_tr_b8 v[118:119], v90 offset:58368\n"
      "s_waitcnt lgkmcnt(4)\n"
      "v_mfma_f32_16x16x128_f8f6f4 v[188:191], v[104:111], a[152:159], v[188:191] cbsz:1\n"
      "ds_read_b64_tr_b8 v[120:121], v91 offset:49152\n"
      "v_mfma_f32_16x16x128_f8f6f4 v[184:187], v[104:111], a[144:151], v[184:187] cbsz:1\n"
      "ds_read_b64_tr_b8 v[122:123], v91 offset:50176\n"
      "v_mfma_f32_16x16x128_f8f6f4 v[180:183], v[104:111], a[136:143], v[180:183] cbsz:1\n"
      "ds_read_b64_tr_b8 v[124:125], v91 offset:57344\n"
      "v_mfma_f32_16x16x128_f8f6f4 v[176:179], v[104:111], a[128:135], v[176:179] cbsz:1\n"
      "ds_read_b64_tr_b8 v[126:127], v91 offset:58368\n"
      "s_waitcnt lgkmcnt(4)\n"
      "v_mfma_f32_16x16x128_f8f6f4 v[160:163], v[112:119], a[160:167], v[160:163] cbsz:1\n"
      "ds_read_b64_tr_b8 v[96:97], v92 offset:0\n"
      "v_mfma_f32_16x16x128_f8f6f4 v[164:167], v[112:119], a[168:175], v[164:167] cbsz:1\n"
      "ds_read_b64_tr_b8 v[98:99], v92 offset:1024\n"
      "v_mfma_f32_16x16x128_f8f6f4 v[168:171], v[112:119], a[176:183], v[168:171] cbsz:1\n"
      "ds_read_b64_tr_b8 v[100:101], v92 offset:8192\n"
      "v_mfma_f32_16x16x128_f8f6f4 v[172:175], v[112:119], a[184:191], v[172:175] cbsz:1\n"
      "ds_read_b64_tr_b8 v[102:103], v92 offset:9216\n"
      "s_waitcnt lgkmcnt(4)\n"
      "v_mfma_f32_16x16x128_f8f6f4 v[188:191], v[120:127], a[184:191], v[188:191] cbsz:1\n"
      "ds_read_b64_tr_b8 v[104:105], v93 offset:0\n"
      "v_mfma_f32_16x16x128_f8f6f4 v[184:187], v[120:127], a[176:183], v[184:187] cbsz:1\n"
      "ds_read_b64_tr_b8 v[106:107], v93 offset:1024\n"
      "v_mul_f32_e32 v160, v61, v160\n"
      "v_mul_f32_e32 v161, v61, v161\n"
      "v_mul_f32_e32 v162, v61, v162\n"
      "v_mul_f32_e32 v163, v61, v163\n"
      "ds_write_b128 v80, v[160:163] offset:0\n"
      "v_mfma_f32_16x16x128_f8f6f4 v[180:183], v[120:127], a[168:175], v[180:183] cbsz:1\n"
      "ds_read_b64_tr_b8 v[108:109], v93 offset:8192\n"
      "v_mul_f32_e32 v164, v61, v164\n"
      "v_mul_f32_e32 v165, v61, v165\n"
      "v_mul_f32_e32 v166, v61, v166\n"
      "v_mul_f32_e32 v167, v61, v167\n"
      "ds_write_b128 v80, v[164:167] offset:2048\n"
      "v_mfma_f32_16x16x128_f8f6f4 v[176:179], v[120:127], a[160:167], v[176:179] cbsz:1\n"
      "ds_read_b64_tr_b8 v[110:111], v93 offset:9216\n"
      "v_mul_f32_e32 v168, v61, v168\n"
      "v_mul_f32_e32 v169, v61, v169\n"
      "v_mul_f32_e32 v170, v61, v170\n"
      "v_mul_f32_e32 v171, v61, v171\n"
      "ds_write_b128 v80, v[168:171] offset:4096\n"
      "s_waitcnt lgkmcnt(7)\n"
      "v_mfma_f32_16x16x128_f8f6f4 v[192:195], v[96:103], a[0:7], v[192:195] cbsz:1\n"
      "ds_read_b64_tr_b8 v[112:113], v92 offset:16384\n"
      "v_mul_f32_e32 v172, v61, v172\n"
      "v_mul_f32_e32 v173, v61, v173\n"
      "v_mul_f32_e32 v174, v61, v174\n"
      "v_mul_f32_e32 v175, v61, v175\n"
      "ds_write_b128 v80, v[172:175] offset:6144\n"
      "v_mfma_f32_16x16x128_f8f6f4 v[196:199], v[96:103], a[8:15], v[196:199] cbsz:1\n"
      "ds_read_b64_tr_b8 v[114:115], v92 offset:17408\n"
      "v_mul_f32_e32 v188, v61, v188\n"
      "v_mul_f32_e32 v189, v61, v189\n"
      "v_mul_f32_e32 v190, v61, v190\n"
      "v_mul_f32_e32 v191, v61, v191\n"
      "ds_write_b128 v81, v[188:191] offset:6144\n"
      "v_mfma_f32_16x16x128_f8f6f4 v[200:203], v[96:103], a[16:23], v[200:203] cbsz:1\n"
      "ds_read_b64_tr_b8 v[116:117], v92 offset:24576\n"
      "v_mul_f32_e32 v184, v61, v184\n"
      "v_mul_f32_e32 v185, v61, v185\n"
      "v_mul_f32_e32 v186, v61, v186\n"
      "v_mul_f32_e32 v187, v61, v187\n"
      "ds_write_b128 v81, v[184:187] offset:4096\n"
      "v_mfma_f32_16x16x128_f8f6f4 v[204:207], v[96:103], a[24:31], v[204:207] cbsz:1\n"
      "ds_read_b64_tr_b8 v[118:119], v92 offset:25600\n"
      "v_mul_f32_e32 v180, v61, v180\n"
      "v_mul_f32_e32 v181, v61, v181\n"
      "v_mul_f32_e32 v182, v61, v182\n"
      "v_mul_f32_e32 v183, v61, v183\n"
      "ds_write_b128 v81, v[180:183] offset:2048\n"
      "s_waitcnt lgkmcnt(9)\n"
      "v_mfma_f32_16x16x128_f8f6f4 v[220:223], v[104:111], a[24:31], v[220:223] cbsz:1\n"
      "ds_read_b64_tr_b8 v[120:121], v93 offset:16384\n"
      "v_mul_f32_e32 v176, v61, v176\n"
      "v_mul_f32_e32 v177, v61, v177\n"
      "v_mul_f32_e32 v178, v61, v178\n"
      "v_mul_f32_e32 v179, v61, v179\n"
      "ds_write_b128 v81, v[176:179] offset:0\n"
      "v_mfma_f32_16x16x128_f8f6f4 v[216:219], v[104:111], a[16:23], v[216:219] cbsz:1\n"
      "ds_read_b64_tr_b8 v[122:123], v93 offset:17408\n"
      "ds_read_b128 v[160:163], v82 offset:0\n"
      "ds_read_b128 v[164:167], v82 offset:1024\n"
      "ds_read_b128 v[168:171], v82 offset:2048\n"
      "ds_read_b128 v[172:175], v82 offset:3072\n"
      "v_mfma_f32_16x16x128_f8f6f4 v[212:215], v[104:111], a[8:15], v[212:215] cbsz:1\n"
      "ds_read_b64_tr_b8 v[124:125], v93 offset:24576\n"
      "ds_read_b128 v[176:179], v82 offset:4096\n"
      "ds_read_b128 v[180:183], v82 offset:5120\n"
      "ds_read_b128 v[184:187], v82 offset:6144\n"
      "ds_read_b128 v[188:191], v82 offset:7168\n"
      "v_mfma_f32_16x16x128_f8f6f4 v[208:211], v[104:111], a[0:7], v[208:211] cbsz:1\n"
      "ds_read_b64_tr_b8 v[126:127], v93 offset:25600\n"
      "s_waitcnt lgkmcnt(14)\n"
      "v_mfma_f32_16x16x128_f8f6f4 v[192:195], v[112:119], a[32:39], v[192:195] cbsz:1\n"
      "ds_read_b64_tr_b8 v[96:97], v92 offset:32768\n"
      "v_mfma_f32_16x16x128_f8f6f4 v[196:199], v[112:119], a[40:47], v[196:199] cbsz:1\n"
      "ds_read_b64_tr_b8 v[98:99], v92 offset:33792\n"
      "s_mov_b32 s71, 0\n"
      "s_waitcnt lgkmcnt(11)\n"
      "buffer_store_dwordx4 v[160:163], %[vout], %[osrd], s71 offen offset:128 sc0 sc1\n"
      "v_mfma_f32_16x16x128_f8f6f4 v[200:203], v[112:119], a[48:55], v[200:203] cbsz:1\n"
      "ds_read_b64_tr_b8 v[100:101], v92 offset:40960\n"
      "v_mfma_f32_16x16x128_f8f6f4 v[204:207], v[112:119], a[56:63], v[204:207] cbsz:1\n"
      "ds_read_b64_tr_b8 v[102:103], v92 offset:41984\n"
      "s_mov_b32 s71, 4194304\n"
      "s_waitcnt lgkmcnt(12)\n"
      "buffer_store_dwordx4 v[164:167], %[vout], %[osrd], s71 offen offset:128 sc0 sc1\n"
      "s_waitcnt lgkmcnt(4)\n"
      "v_mfma_f32_16x16x128_f8f6f4 v[220:223], v[120:127], a[56:63], v[220:223] cbsz:1\n"
      "ds_read_b64_tr_b8 v[104:105], v93 offset:32768\n"
      "v_mfma_f32_16x16x128_f8f6f4 v[216:219], v[120:127], a[48:55], v[216:219] cbsz:1\n"
      "ds_read_b64_tr_b8 v[106:107], v93 offset:33792\n"
      "s_mov_b32 s71, 8388608\n"
      "s_waitcnt lgkmcnt(13)\n"
      "buffer_store_dwordx4 v[168:171], %[vout], %[osrd], s71 offen offset:128 sc0 sc1\n"
      "v_mfma_f32_16x16x128_f8f6f4 v[212:215], v[120:127], a[40:47], v[212:215] cbsz:1\n"
      "ds_read_b64_tr_b8 v[108:109], v93 offset:40960\n"
      "v_mfma_f32_16x16x128_f8f6f4 v[208:211], v[120:127], a[32:39], v[208:211] cbsz:1\n"
      "ds_read_b64_tr_b8 v[110:111], v93 offset:41984\n"
      "s_mov_b32 s71, 12582912\n"
      "s_waitcnt lgkmcnt(14)\n"
      "buffer_store_dwordx4 v[172:175], %[vout], %[osrd], s71 offen offset:128 sc0 sc1\n"
      "s_waitcnt lgkmcnt(4)\n"
      "v_mfma_f32_16x16x128_f8f6f4 v[192:195], v[96:103], a[128:135], v[192:195] cbsz:1\n"
      "ds_read_b64_tr_b8 v[112:113], v92 offset:49152\n"
      "v_mfma_f32_16x16x128_f8f6f4 v[196:199], v[96:103], a[136:143], v[196:199] cbsz:1\n"
      "ds_read_b64_tr_b8 v[114:115], v92 offset:50176\n"
      "s_mov_b32 s71, 16777216\n"
      "s_waitcnt lgkmcnt(14)\n"
      "buffer_store_dwordx4 v[176:179], %[vout], %[osrd], s71 offen offset:128 sc0 sc1\n"
      "v_mfma_f32_16x16x128_f8f6f4 v[200:203], v[96:103], a[144:151], v[200:203] cbsz:1\n"
      "ds_read_b64_tr_b8 v[116:117], v92 offset:57344\n"
      "v_mfma_f32_16x16x128_f8f6f4 v[204:207], v[96:103], a[152:159], v[204:207] cbsz:1\n"
      "ds_read_b64_tr_b8 v[118:119], v92 offset:58368\n"
      "s_mov_b32 s71, 20971520\n"
      "s_waitcnt lgkmcnt(15)\n"
      "buffer_store_dwordx4 v[180:183], %[vout], %[osrd], s71 offen offset:128 sc0 sc1\n"
      "s_waitcnt lgkmcnt(4)\n"
      "v_mfma_f32_16x16x128_f8f6f4 v[220:223], v[104:111], a[152:159], v[220:223] cbsz:1\n"
      "ds_read_b64_tr_b8 v[120:121], v93 offset:49152\n"
      "v_mfma_f32_16x16x128_f8f6f4 v[216:219], v[104:111], a[144:151], v[216:219] cbsz:1\n"
      "ds_read_b64_tr_b8 v[122:123], v93 offset:50176\n"
      "s_mov_b32 s71, 25165824\n"
      "s_waitcnt lgkmcnt(15)\n"
      "buffer_store_dwordx4 v[184:187], %[vout], %[osrd], s71 offen offset:128 sc0 sc1\n"
      "v_mfma_f32_16x16x128_f8f6f4 v[212:215], v[104:111], a[136:143], v[212:215] cbsz:1\n"
      "ds_read_b64_tr_b8 v[124:125], v93 offset:57344\n"
      "v_mfma_f32_16x16x128_f8f6f4 v[208:211], v[104:111], a[128:135], v[208:211] cbsz:1\n"
      "ds_read_b64_tr_b8 v[126:127], v93 offset:58368\n"
      "s_mov_b32 s71, 29360128\n"
      "s_waitcnt lgkmcnt(15)\n"
      "buffer_store_dwordx4 v[188:191], %[vout], %[osrd], s71 offen offset:128 sc0 sc1\n"
      "s_waitcnt lgkmcnt(4)\n"
      "v_mfma_f32_16x16x128_f8f6f4 v[192:195], v[112:119], a[160:167], v[192:195] cbsz:1\n"
      "ds_read_b64_tr_b8 v[96:97], v94 offset:0\n"
      "v_mfma_f32_16x16x128_f8f6f4 v[196:199], v[112:119], a[168:175], v[196:199] cbsz:1\n"
      "ds_read_b64_tr_b8 v[98:99], v94 offset:1024\n"
      "v_mfma_f32_16x16x128_f8f6f4 v[200:203], v[112:119], a[176:183], v[200:203] cbsz:1\n"
      "ds_read_b64_tr_b8 v[100:101], v94 offset:8192\n"
      "v_mfma_f32_16x16x128_f8f6f4 v[204:207], v[112:119], a[184:191], v[204:207] cbsz:1\n"
      "ds_read_b64_tr_b8 v[102:103], v94 offset:9216\n"
      "s_waitcnt lgkmcnt(4)\n"
      "v_mfma_f32_16x16x128_f8f6f4 v[220:223], v[120:127], a[184:191], v[220:223] cbsz:1\n"
      "ds_read_b64_tr_b8 v[104:105], v95 offset:0\n"
      "v_mfma_f32_16x16x128_f8f6f4 v[216:219], v[120:127], a[176:183], v[216:219] cbsz:1\n"
      "ds_read_b64_tr_b8 v[106:107], v95 offset:1024\n"
      "v_mul_f32_e32 v192, v61, v192\n"
      "v_mul_f32_e32 v193, v61, v193\n"
      "v_mul_f32_e32 v194, v61, v194\n"
      "v_mul_f32_e32 v195, v61, v195\n"
      "ds_write_b128 v80, v[192:195] offset:0\n"
      "v_mfma_f32_16x16x128_f8f6f4 v[212:215], v[120:127], a[168:175], v[212:215] cbsz:1\n"
      "ds_read_b64_tr_b8 v[108:109], v95 offset:8192\n"
      "v_mul_f32_e32 v196, v61, v196\n"
      "v_mul_f32_e32 v197, v61, v197\n"
      "v_mul_f32_e32 v198, v61, v198\n"
      "v_mul_f32_e32 v199, v61, v199\n"
      "ds_write_b128 v80, v[196:199] offset:2048\n"
      "v_mfma_f32_16x16x128_f8f6f4 v[208:211], v[120:127], a[160:167], v[208:211] cbsz:1\n"
      "ds_read_b64_tr_b8 v[110:111], v95 offset:9216\n"
      "v_mul_f32_e32 v200, v61, v200\n"
      "v_mul_f32_e32 v201, v61, v201\n"
      "v_mul_f32_e32 v202, v61, v202\n"
      "v_mul_f32_e32 v203, v61, v203\n"
      "ds_write_b128 v80, v[200:203] offset:4096\n"
      "s_waitcnt lgkmcnt(7)\n"
      "v_mfma_f32_16x16x128_f8f6f4 v[224:227], v[96:103], a[0:7], v[224:227] cbsz:1\n"
      "ds_read_b64_tr_b8 v[112:113], v94 offset:16384\n"
      "v_mul_f32_e32 v204, v61, v204\n"
      "v_mul_f32_e32 v205, v61, v205\n"
      "v_mul_f32_e32 v206, v61, v206\n"
      "v_mul_f32_e32 v207, v61, v207\n"
      "ds_write_b128 v80, v[204:207] offset:6144\n"
      "v_mfma_f32_16x16x128_f8f6f4 v[228:231], v[96:103], a[8:15], v[228:231] cbsz:1\n"
      "ds_read_b64_tr_b8 v[114:115], v94 offset:17408\n"
      "v_mul_f32_e32 v220, v61, v220\n"
      "v_mul_f32_e32 v221, v61, v221\n"
      "v_mul_f32_e32 v222, v61, v222\n"
      "v_mul_f32_e32 v223, v61, v223\n"
      "ds_write_b128 v81, v[220:223] offset:6144\n"
      "v_mfma_f32_16x16x128_f8f6f4 v[232:235], v[96:103], a[16:23], v[232:235] cbsz:1\n"
      "ds_read_b64_tr_b8 v[116:117], v94 offset:24576\n"
      "v_mul_f32_e32 v216, v61, v216\n"
      "v_mul_f32_e32 v217, v61, v217\n"
      "v_mul_f32_e32 v218, v61, v218\n"
      "v_mul_f32_e32 v219, v61, v219\n"
      "ds_write_b128 v81, v[216:219] offset:4096\n"
      "v_mfma_f32_16x16x128_f8f6f4 v[236:239], v[96:103], a[24:31], v[236:239] cbsz:1\n"
      "ds_read_b64_tr_b8 v[118:119], v94 offset:25600\n"
      "v_mul_f32_e32 v212, v61, v212\n"
      "v_mul_f32_e32 v213, v61, v213\n"
      "v_mul_f32_e32 v214, v61, v214\n"
      "v_mul_f32_e32 v215, v61, v215\n"
      "ds_write_b128 v81, v[212:215] offset:2048\n"
      "s_waitcnt lgkmcnt(9)\n"
      "v_mfma_f32_16x16x128_f8f6f4 v[252:255], v[104:111], a[24:31], v[252:255] cbsz:1\n"
      "ds_read_b64_tr_b8 v[120:121], v95 offset:16384\n"
      "v_mul_f32_e32 v208, v61, v208\n"
      "v_mul_f32_e32 v209, v61, v209\n"
      "v_mul_f32_e32 v210, v61, v210\n"
      "v_mul_f32_e32 v211, v61, v211\n"
      "ds_write_b128 v81, v[208:211] offset:0\n"
      "v_mfma_f32_16x16x128_f8f6f4 v[248:251], v[104:111], a[16:23], v[248:251] cbsz:1\n"
      "ds_read_b64_tr_b8 v[122:123], v95 offset:17408\n"
      "ds_read_b128 v[192:195], v82 offset:0\n"
      "ds_read_b128 v[196:199], v82 offset:1024\n"
      "ds_read_b128 v[200:203], v82 offset:2048\n"
      "ds_read_b128 v[204:207], v82 offset:3072\n"
      "v_mfma_f32_16x16x128_f8f6f4 v[244:247], v[104:111], a[8:15], v[244:247] cbsz:1\n"
      "ds_read_b64_tr_b8 v[124:125], v95 offset:24576\n"
      "ds_read_b128 v[208:211], v82 offset:4096\n"
      "ds_read_b128 v[212:215], v82 offset:5120\n"
      "ds_read_b128 v[216:219], v82 offset:6144\n"
      "ds_read_b128 v[220:223], v82 offset:7168\n"
      "v_mfma_f32_16x16x128_f8f6f4 v[240:243], v[104:111], a[0:7], v[240:243] cbsz:1\n"
      "ds_read_b64_tr_b8 v[126:127], v95 offset:25600\n"
      "s_waitcnt lgkmcnt(14)\n"
      "v_mfma_f32_16x16x128_f8f6f4 v[224:227], v[112:119], a[32:39], v[224:227] cbsz:1\n"
      "ds_read_b64_tr_b8 v[96:97], v94 offset:32768\n"
      "v_mfma_f32_16x16x128_f8f6f4 v[228:231], v[112:119], a[40:47], v[228:231] cbsz:1\n"
      "ds_read_b64_tr_b8 v[98:99], v94 offset:33792\n"
      "s_mov_b32 s71, 0\n"
      "s_waitcnt lgkmcnt(11)\n"
      "buffer_store_dwordx4 v[192:195], %[vout], %[osrd], s71 offen offset:256 sc0 sc1\n"
      "v_mfma_f32_16x16x128_f8f6f4 v[232:235], v[112:119], a[48:55], v[232:235] cbsz:1\n"
      "ds_read_b64_tr_b8 v[100:101], v94 offset:40960\n"
      "v_mfma_f32_16x16x128_f8f6f4 v[236:239], v[112:119], a[56:63], v[236:239] cbsz:1\n"
      "ds_read_b64_tr_b8 v[102:103], v94 offset:41984\n"
      "s_mov_b32 s71, 4194304\n"
      "s_waitcnt lgkmcnt(12)\n"
      "buffer_store_dwordx4 v[196:199], %[vout], %[osrd], s71 offen offset:256 sc0 sc1\n"
      "s_waitcnt lgkmcnt(4)\n"
      "v_mfma_f32_16x16x128_f8f6f4 v[252:255], v[120:127], a[56:63], v[252:255] cbsz:1\n"
      "ds_read_b64_tr_b8 v[104:105], v95 offset:32768\n"
      "v_mfma_f32_16x16x128_f8f6f4 v[248:251], v[120:127], a[48:55], v[248:251] cbsz:1\n"
      "ds_read_b64_tr_b8 v[106:107], v95 offset:33792\n"
      "s_mov_b32 s71, 8388608\n"
      "s_waitcnt lgkmcnt(13)\n"
      "buffer_store_dwordx4 v[200:203], %[vout], %[osrd], s71 offen offset:256 sc0 sc1\n"
      "v_mfma_f32_16x16x128_f8f6f4 v[244:247], v[120:127], a[40:47], v[244:247] cbsz:1\n"
      "ds_read_b64_tr_b8 v[108:109], v95 offset:40960\n"
      "v_mfma_f32_16x16x128_f8f6f4 v[240:243], v[120:127], a[32:39], v[240:243] cbsz:1\n"
      "ds_read_b64_tr_b8 v[110:111], v95 offset:41984\n"
      "s_mov_b32 s71, 12582912\n"
      "s_waitcnt lgkmcnt(14)\n"
      "buffer_store_dwordx4 v[204:207], %[vout], %[osrd], s71 offen offset:256 sc0 sc1\n"
      "s_waitcnt lgkmcnt(4)\n"
      "v_mfma_f32_16x16x128_f8f6f4 v[224:227], v[96:103], a[128:135], v[224:227] cbsz:1\n"
      "ds_read_b64_tr_b8 v[112:113], v94 offset:49152\n"
      "v_mfma_f32_16x16x128_f8f6f4 v[228:231], v[96:103], a[136:143], v[228:231] cbsz:1\n"
      "ds_read_b64_tr_b8 v[114:115], v94 offset:50176\n"
      "s_mov_b32 s71, 16777216\n"
      "s_waitcnt lgkmcnt(14)\n"
      "buffer_store_dwordx4 v[208:211], %[vout], %[osrd], s71 offen offset:256 sc0 sc1\n"
      "v_mfma_f32_16x16x128_f8f6f4 v[232:235], v[96:103], a[144:151], v[232:235] cbsz:1\n"
      "ds_read_b64_tr_b8 v[116:117], v94 offset:57344\n"
      "v_mfma_f32_16x16x128_f8f6f4 v[236:239], v[96:103], a[152:159], v[236:239] cbsz:1\n"
      "ds_read_b64_tr_b8 v[118:119], v94 offset:58368\n"
      "s_mov_b32 s71, 20971520\n"
      "s_waitcnt lgkmcnt(15)\n"
      "buffer_store_dwordx4 v[212:215], %[vout], %[osrd], s71 offen offset:256 sc0 sc1\n"
      "s_waitcnt lgkmcnt(4)\n"
      "v_mfma_f32_16x16x128_f8f6f4 v[252:255], v[104:111], a[152:159], v[252:255] cbsz:1\n"
      "ds_read_b64_tr_b8 v[120:121], v95 offset:49152\n"
      "v_mfma_f32_16x16x128_f8f6f4 v[248:251], v[104:111], a[144:151], v[248:251] cbsz:1\n"
      "ds_read_b64_tr_b8 v[122:123], v95 offset:50176\n"
      "s_mov_b32 s71, 25165824\n"
      "s_waitcnt lgkmcnt(15)\n"
      "buffer_store_dwordx4 v[216:219], %[vout], %[osrd], s71 offen offset:256 sc0 sc1\n"
      "v_mfma_f32_16x16x128_f8f6f4 v[244:247], v[104:111], a[136:143], v[244:247] cbsz:1\n"
      "ds_read_b64_tr_b8 v[124:125], v95 offset:57344\n"
      "v_mfma_f32_16x16x128_f8f6f4 v[240:243], v[104:111], a[128:135], v[240:243] cbsz:1\n"
      "ds_read_b64_tr_b8 v[126:127], v95 offset:58368\n"
      "s_mov_b32 s71, 29360128\n"
      "s_waitcnt lgkmcnt(15)\n"
      "buffer_store_dwordx4 v[220:223], %[vout], %[osrd], s71 offen offset:256 sc0 sc1\n"
      "s_waitcnt lgkmcnt(4)\n"
      "v_mfma_f32_16x16x128_f8f6f4 v[224:227], v[112:119], a[160:167], v[224:227] cbsz:1\n"
      "v_mfma_f32_16x16x128_f8f6f4 v[228:231], v[112:119], a[168:175], v[228:231] cbsz:1\n"
      "v_mfma_f32_16x16x128_f8f6f4 v[232:235], v[112:119], a[176:183], v[232:235] cbsz:1\n"
      "v_mfma_f32_16x16x128_f8f6f4 v[236:239], v[112:119], a[184:191], v[236:239] cbsz:1\n"
      "s_waitcnt lgkmcnt(0)\n"
      "v_mfma_f32_16x16x128_f8f6f4 v[252:255], v[120:127], a[184:191], v[252:255] cbsz:1\n"
      "v_mfma_f32_16x16x128_f8f6f4 v[248:251], v[120:127], a[176:183], v[248:251] cbsz:1\n"
      "v_mfma_f32_16x16x128_f8f6f4 v[244:247], v[120:127], a[168:175], v[244:247] cbsz:1\n"
      "v_mfma_f32_16x16x128_f8f6f4 v[240:243], v[120:127], a[160:167], v[240:243] cbsz:1\n"
      "s_nop 7\n"
      "v_mul_f32_e32 v224, v61, v224\n"
      "v_mul_f32_e32 v225, v61, v225\n"
      "v_mul_f32_e32 v226, v61, v226\n"
      "v_mul_f32_e32 v227, v61, v227\n"
      "ds_write_b128 v80, v[224:227] offset:0\n"
      "v_mul_f32_e32 v228, v61, v228\n"
      "v_mul_f32_e32 v229, v61, v229\n"
      "v_mul_f32_e32 v230, v61, v230\n"
      "v_mul_f32_e32 v231, v61, v231\n"
      "ds_write_b128 v80, v[228:231] offset:2048\n"
      "v_mul_f32_e32 v232, v61, v232\n"
      "v_mul_f32_e32 v233, v61, v233\n"
      "v_mul_f32_e32 v234, v61, v234\n"
      "v_mul_f32_e32 v235, v61, v235\n"
      "ds_write_b128 v80, v[232:235] offset:4096\n"
      "v_mul_f32_e32 v236, v61, v236\n"
      "v_mul_f32_e32 v237, v61, v237\n"
      "v_mul_f32_e32 v238, v61, v238\n"
      "v_mul_f32_e32 v239, v61, v239\n"
      "ds_write_b128 v80, v[236:239] offset:6144\n"
      "v_mul_f32_e32 v252, v61, v252\n"
      "v_mul_f32_e32 v253, v61, v253\n"
      "v_mul_f32_e32 v254, v61, v254\n"
      "v_mul_f32_e32 v255, v61, v255\n"
      "ds_write_b128 v81, v[252:255] offset:6144\n"
      "v_mul_f32_e32 v248, v61, v248\n"
      "v_mul_f32_e32 v249, v61, v249\n"
      "v_mul_f32_e32 v250, v61, v250\n"
      "v_mul_f32_e32 v251, v61, v251\n"
      "ds_write_b128 v81, v[248:251] offset:4096\n"
      "v_mul_f32_e32 v244, v61, v244\n"
      "v_mul_f32_e32 v245, v61, v245\n"
      "v_mul_f32_e32 v246, v61, v246\n"
      "v_mul_f32_e32 v247, v61, v247\n"
      "ds_write_b128 v81, v[244:247] offset:2048\n"
      "v_mul_f32_e32 v240, v61, v240\n"
      "v_mul_f32_e32 v241, v61, v241\n"
      "v_mul_f32_e32 v242, v61, v242\n"
      "v_mul_f32_e32 v243, v61, v243\n"
      "ds_write_b128 v81, v[240:243] offset:0\n"
      "ds_read_b128 v[224:227], v82 offset:0\n"
      "ds_read_b128 v[228:231], v82 offset:1024\n"
      "ds_read_b128 v[232:235], v82 offset:2048\n"
      "ds_read_b128 v[236:239], v82 offset:3072\n"
      "ds_read_b128 v[240:243], v82 offset:4096\n"
      "ds_read_b128 v[244:247], v82 offset:5120\n"
      "ds_read_b128 v[248:251], v82 offset:6144\n"
      "ds_read_b128 v[252:255], v82 offset:7168\n"
      "s_mov_b32 s71, 0\n"
      "s_waitcnt lgkmcnt(7)\n"
      "buffer_store_dwordx4 v[224:227], %[vout], %[osrd], s71 offen offset:384 sc0 sc1\n"
      "s_mov_b32 s71, 4194304\n"
      "s_waitcnt lgkmcnt(6)\n"
      "buffer_store_dwordx4 v[228:231], %[vout], %[osrd], s71 offen offset:384 sc0 sc1\n"
      "s_mov_b32 s71, 8388608\n"
      "s_waitcnt lgkmcnt(5)\n"
      "buffer_store_dwordx4 v[232:235], %[vout], %[osrd], s71 offen offset:384 sc0 sc1\n"
      "s_mov_b32 s71, 12582912\n"
      "s_waitcnt lgkmcnt(4)\n"
      "buffer_store_dwordx4 v[236:239], %[vout], %[osrd], s71 offen offset:384 sc0 sc1\n"
      "s_mov_b32 s71, 16777216\n"
      "s_waitcnt lgkmcnt(3)\n"
      "buffer_store_dwordx4 v[240:243], %[vout], %[osrd], s71 offen offset:384 sc0 sc1\n"
      "s_mov_b32 s71, 20971520\n"
      "s_waitcnt lgkmcnt(2)\n"
      "buffer_store_dwordx4 v[244:247], %[vout], %[osrd], s71 offen offset:384 sc0 sc1\n"
      "s_mov_b32 s71, 25165824\n"
      "s_waitcnt lgkmcnt(1)\n"
      "buffer_store_dwordx4 v[248:251], %[vout], %[osrd], s71 offen offset:384 sc0 sc1\n"
      "s_mov_b32 s71, 29360128\n"
      "s_waitcnt lgkmcnt(0)\n"
      "buffer_store_dwordx4 v[252:255], %[vout], %[osrd], s71 offen offset:384 sc0 sc1\n"
      "s_cmp_lg_u32 %[sm0], 0\n"
      "s_cbranch_scc1 Lce284_%=\n"
      "s_mov_b64 exec, 1\n"
      "buffer_store_dword v62, off, %[asrd], %[sctr] sc1\n"
      "s_mov_b64 exec, -1\n"
      "Lce284_%=:\n"
      "s_branch Lend_%=\n"
      "Llate_%=:\n"
      "v_mfma_f32_16x16x128_f8f6f4 v[176:179], v[120:127], a[64:71], v[176:179] cbsz:1\n"
      "ds_read_b64_tr_b8 v[110:111], v93 offset:41984\n"
      "v_readlane_b32 s90, v54, 7\n"
      "s_and_b32 s66, s90, 0xfffe0000\n"
      "s_and_b32 m0, s90, 0x1ffff\n"
      "s_nop 0\n"
      "buffer_load_dwordx4 v84, %[wsrd], s66 offen offset:0  lds\n"
      "s_waitcnt lgkmcnt(4)\n"
      "v_mfma_f32_16x16x128_f8f6f4 v[192:195], v[96:103], a[64:71], v[192:195] cbsz:1\n"
      "ds_read_b64_tr_b8 v[112:113], v94 offset:32768\n"
      "v_mfma_f32_16x16x128_f8f6f4 v[196:199], v[96:103], a[72:79], v[196:199] cbsz:1\n"
      "ds_read_b64_tr_b8 v[114:115], v94 offset:33792\n"
      "buffer_load_dwordx4 v85, %[wsrd], s66 offen offset:1024  lds\n"
      "v_mfma_f32_16x16x128_f8f6f4 v[200:203], v[96:103], a[80:87], v[200:203] cbsz:1\n"
      "ds_read_b64_tr_b8 v[116:117], v94 offset:40960\n"
      "v_mfma_f32_16x16x128_f8f6f4 v[204:207], v[96:103], a[88:95], v[204:207] cbsz:1\n"
      "ds_read_b64_tr_b8 v[118:119], v94 offset:41984\n"
      "buffer_load_dwordx4 v86, %[wsrd], s66 offen offset:2048  lds\n"
      "s_waitcnt lgkmcnt(4)\n"
      "v_mfma_f32_16x16x128_f8f6f4 v[220:223], v[104:111], a[88:95], v[220:223] cbsz:1\n"
      "ds_read_b64_tr_b8 v[120:121], v95 offset:32768\n"
      "v_mfma_f32_16x16x128_f8f6f4 v[216:219], v[104:111], a[80:87], v[216:219] cbsz:1\n"
      "ds_read_b64_tr_b8 v[122:123], v95 offset:33792\n"
      "buffer_load_dwordx4 v87, %[wsrd], s66 offen offset:3072  lds\n"
      "v_mfma_f32_16x16x128_f8f6f4 v[212:215], v[104:111], a[72:79], v[212:215] cbsz:1\n"
      "ds_read_b64_tr_b8 v[124:125], v95 offset:40960\n"
      "s_waitcnt vmcnt(12)\n"
      "ds_read_b128 a[96:99], v78 offset:0\n"
      "ds_read_b128 a[100:103], v79 offset:0\n"
      "v_mfma_f32_16x16x128_f8f6f4 v[208:211], v[104:111], a[64:71], v[208:211] cbsz:1\n"
      "ds_read_b64_tr_b8 v[126:127], v95 offset:41984\n"
      "ds_read_b128 a[104:107], v78 offset:2048\n"
      "ds_read_b128 a[108:111], v79 offset:2048\n"
      "s_waitcnt lgkmcnt(4)\n"
      "v_mfma_f32_16x16x128_f8f6f4 v[224:227], v[112:119], a[64:71], v[224:227] cbsz:1\n"
      "ds_read_b64_tr_b8 v[96:97], v88 offset:49152\n"
      "ds_read_b128 a[112:115], v78 offset:4096\n"
      "ds_read_b128 a[116:119], v79 offset:4096\n"
      "v_mfma_f32_16x16x128_f8f6f4 v[228:231], v[112:119], a[72:79], v[228:231] cbsz:1\n"
      "ds_read_b64_tr_b8 v[98:99], v88 offset:50176\n"
      "ds_read_b128 a[120:123], v78 offset:6144\n"
      "ds_read_b128 a[124:127], v79 offset:6144\n"
      "v_mfma_f32_16x16x128_f8f6f4 v[232:235], v[112:119], a[80:87], v[232:235] cbsz:1\n"
      "ds_read_b64_tr_b8 v[100:101], v88 offset:57344\n"
      "v_mfma_f32_16x16x128_f8f6f4 v[236:239], v[112:119], a[88:95], v[236:239] cbsz:1\n"
      "ds_read_b64_tr_b8 v[102:103], v88 offset:58368\n"
      "v_xor_b32_e32 v88, 0x10000, v88\n"
      "s_cmp_eq_u32 %[sps], 5\n"
      "s_cbranch_scc0 Lps223_%=\n"
      "buffer_store_dwordx4 v[64:67], %[vpd], %[psrd], s81 offen sc0 sc1\n"
      "buffer_store_dwordx4 v[68:71], %[vpd], %[psrd], s82 offen sc0 sc1\n"
      "Lps223_%=:\n"
      "s_waitcnt lgkmcnt(4)\n"
      "v_mfma_f32_16x16x128_f8f6f4 v[252:255], v[120:127], a[88:95], v[252:255] cbsz:1\n"
      "ds_read_b64_tr_b8 v[104:105], v89 offset:49152\n"
      "v_mfma_f32_16x16x128_f8f6f4 v[248:251], v[120:127], a[80:87], v[248:251] cbsz:1\n"
      "ds_read_b64_tr_b8 v[106:107], v89 offset:50176\n"
      "v_mfma_f32_16x16x128_f8f6f4 v[244:247], v[120:127], a[72:79], v[244:247] cbsz:1\n"
      "ds_read_b64_tr_b8 v[108:109], v89 offset:57344\n"
      "v_mfma_f32_16x16x128_f8f6f4 v[240:243], v[120:127], a[64:71], v[240:243] cbsz:1\n"
      "ds_read_b64_tr_b8 v[110:111], v89 offset:58368\n"
      "v_xor_b32_e32 v89, 0x10000, v89\n"
      "s_cmp_eq_u32 %[smt], 0\n"
      "s_cbranch_scc1 Ltw224_%=\n"
      "s_waitcnt vmcnt(4)\n"
      "Ltw224_%=:\n"
      "s_waitcnt vmcnt(20)\n"
      "s_barrier\n"
      "s_waitcnt lgkmcnt(4)\n"
      "v_mfma_f32_16x16x128_f8f6f4 v[128:131], v[96:103], a[96:103], v[128:131] cbsz:1\n"
      "ds_read_b64_tr_b8 v[112:113], v90 offset:49152\n"
      "v_mfma_f32_16x16x128_f8f6f4 v[132:135], v[96:103], a[104:111], v[132:135] cbsz:1\n"
      "ds_read_b64_tr_b8 v[114:115], v90 offset:50176\n"
      "v_readlane_b32 s90, v54, 8\n"
      "s_and_b32 s66, s90, 0xfffe0000\n"
      "s_and_b32 m0, s90, 0x1ffff\n"
      "s_nop 0\n"
      "buffer_load_dwordx4 v84, %[wsrd], s66 offen offset:0  lds\n"
      "v_mfma_f32_16x16x128_f8f6f4 v[136:139], v[96:103], a[112:119], v[136:139] cbsz:1\n"
      "ds_read_b64_tr_b8 v[116:117], v90 offset:57344\n"
      "v_mfma_f32_16x16x128_f8f6f4 v[140:143], v[96:103], a[120:127], v[140:143] cbsz:1\n"
      "ds_read_b64_tr_b8 v[118:119], v90 offset:58368\n"
      "v_xor_b32_e32 v90, 0x10000, v90\n"
      "buffer_load_dwordx4 v85, %[wsrd], s66 offen offset:1024  lds\n"
      "s_waitcnt lgkmcnt(4)\n"
      "v_mfma_f32_16x16x128_f8f6f4 v[156:159], v[104:111], a[120:127], v[156:159] cbsz:1\n"
      "ds_read_b64_tr_b8 v[120:121], v91 offset:49152\n"
      "v_mfma_f32_16x16x128_f8f6f4 v[152:155], v[104:111], a[112:119], v[152:155] cbsz:1\n"
      "ds_read_b64_tr_b8 v[122:123], v91 offset:50176\n"
      "buffer_load_dwordx4 v86, %[wsrd], s66 offen offset:2048  lds\n"
      "v_mfma_f32_16x16x128_f8f6f4 v[148:151], v[104:111], a[104:111], v[148:151] cbsz:1\n"
      "ds_read_b64_tr_b8 v[124:125], v91 offset:57344\n"
      "v_mfma_f32_16x16x128_f8f6f4 v[144:147], v[104:111], a[96:103], v[144:147] cbsz:1\n"
      "ds_read_b64_tr_b8 v[126:127], v91 offset:58368\n"
      "v_xor_b32_e32 v91, 0x10000, v91\n"
      "buffer_load_dwordx4 v87, %[wsrd], s66 offen offset:3072  lds\n"
      "s_waitcnt lgkmcnt(4)\n"
      "v_mfma_f32_16x16x128_f8f6f4 v[160:163], v[112:119], a[96:103], v[160:163] cbsz:1\n"
      "ds_read_b64_tr_b8 v[96:97], v92 offset:49152\n"
      "v_mfma_f32_16x16x128_f8f6f4 v[164:167], v[112:119], a[104:111], v[164:167] cbsz:1\n"
      "ds_read_b64_tr_b8 v[98:99], v92 offset:50176\n"
      "v_readlane_b32 s90, v54, 9\n"
      "s_and_b32 s66, s90, 0xfffe0000\n"
      "s_and_b32 m0, s90, 0x1ffff\n"
      "s_nop 0\n"
      "buffer_load_dwordx4 v84, %[wsrd], s66 offen offset:0  lds\n"
      "v_mfma_f32_16x16x128_f8f6f4 v[168:171], v[112:119], a[112:119], v[168:171] cbsz:1\n"
      "ds_read_b64_tr_b8 v[100:101], v92 offset:57344\n"
      "v_mfma_f32_16x16x128_f8f6f4 v[172:175], v[112:119], a[120:127], v[172:175] cbsz:1\n"
      "ds_read_b64_tr_b8 v[102:103], v92 offset:58368\n"
      "v_xor_b32_e32 v92, 0x10000, v92\n"
      "buffer_load_dwordx4 v85, %[wsrd], s66 offen offset:1024  lds\n"
      "s_waitcnt lgkmcnt(4)\n"
      "v_mfma_f32_16x16x128_f8f6f4 v[188:191], v[120:127], a[120:127], v[188:191] cbsz:1\n"
      "ds_read_b64_tr_b8 v[104:105], v93 offset:49152\n"
      "s_cmp_eq_u32 %[sps], 4\n"
      "s_cbranch_scc0 Lpb225_%=\n"
      "s_waitcnt vmcnt(18)\n"
      "s_mov_b64 exec, 1\n"
      "buffer_store_dword v62, off, %[psrd], %[spf] sc0 sc1\n"
      "s_mov_b64 exec, -1\n"
      "Lpb225_%=:\n"
      "v_mfma_f32_16x16x128_f8f6f4 v[184:187], v[120:127], a[112:119], v[184:187] cbsz:1\n"
      "ds_read_b64_tr_b8 v[106:107], v93 offset:50176\n"
      "buffer_load_dwordx4 v86, %[wsrd], s66 offen offset:2048  lds\n"
      "v_mfma_f32_16x16x128_f8f6f4 v[180:183], v[120:127], a[104:111], v[180:183] cbsz:1\n"
      "ds_read_b64_tr_b8 v[108:109], v93 offset:57344\n"
      "v_mfma_f32_16x16x128_f8f6f4 v[176:179], v[120:127], a[96:103], v[176:179] cbsz:1\n"
      "ds_read_b64_tr_b8 v[110:111], v93 offset:58368\n"
      "v_xor_b32_e32 v93, 0x10000, v93\n"
      "buffer_load_dwordx4 v87, %[wsrd], s66 offen offset:3072  lds\n"
      "s_waitcnt lgkmcnt(4)\n"
      "v_mfma_f32_16x16x128_f8f6f4 v[192:195], v[96:103], a[96:103], v[192:195] cbsz:1\n"
      "ds_read_b64_tr_b8 v[112:113], v94 offset:49152\n"
      "v_mfma_f32_16x16x128_f8f6f4 v[196:199], v[96:103], a[104:111], v[196:199] cbsz:1\n"
      "ds_read_b64_tr_b8 v[114:115], v94 offset:50176\n"
      "v_mfma_f32_16x16x128_f8f6f4 v[200:203], v[96:103], a[112:119], v[200:203] cbsz:1\n"
      "ds_read_b64_tr_b8 v[116:117], v94 offset:57344\n"
      "s_cmp_eq_u32 %[sps], 6\n"
      "s_cbranch_scc0 Lps226_%=\n"
      "buffer_store_dwordx4 v[64:67], %[vpd], %[psrd], s81 offen sc0 sc1\n"
      "buffer_store_dwordx4 v[68:71], %[vpd], %[psrd], s82 offen sc0 sc1\n"
      "Lps226_%=:\n"
      "v_mfma_f32_16x16x128_f8f6f4 v[204:207], v[96:103], a[120:127], v[204:207] cbsz:1\n"
      "ds_read_b64_tr_b8 v[118:119], v94 offset:58368\n"
      "v_xor_b32_e32 v94, 0x10000, v94\n"
      "s_waitcnt lgkmcnt(4)\n"
      "v_mfma_f32_16x16x128_f8f6f4 v[220:223], v[104:111], a[120:127], v[220:223] cbsz:1\n"
      "ds_read_b64_tr_b8 v[120:121], v95 offset:49152\n"
      "v_mfma_f32_16x16x128_f8f6f4 v[216:219], v[104:111], a[112:119], v[216:219] cbsz:1\n"
      "ds_read_b64_tr_b8 v[122:123], v95 offset:50176\n"
      "v_mfma_f32_16x16x128_f8f6f4 v[212:215], v[104:111], a[104:111], v[212:215] cbsz:1\n"
      "ds_read_b64_tr_b8 v[124:125], v95 offset:57344\n"
      "s_add_u32 s91, s91, 64\n"
      "s_load_dwordx8 s[56:63], %[pfl], s91\n"
      "v_mfma_f32_16x16x128_f8f6f4 v[208:211], v[104:111], a[96:103], v[208:211] cbsz:1\n"
      "ds_read_b64_tr_b8 v[126:127], v95 offset:58368\n"
      "v_xor_b32_e32 v95, 0x10000, v95\n"
      "s_waitcnt lgkmcnt(4)\n"
      "v_mfma_f32_16x16x128_f8f6f4 v[224:227], v[112:119], a[96:103], v[224:227] cbsz:1\n"
      "ds_read_b64_tr_b8 v[96:97], v88 offset:0\n"
      "v_mfma_f32_16x16x128_f8f6f4 v[228:231], v[112:119], a[104:111], v[228:231] cbsz:1\n"
      "ds_read_b64_tr_b8 v[98:99], v88 offset:1024\n"
      "v_mfma_f32_16x16x128_f8f6f4 v[232:235], v[112:119], a[112:119], v[232:235] cbsz:1\n"
      "ds_read_b64_tr_b8 v[100:101], v88 offset:8192\n"
      "v_mfma_f32_16x16x128_f8f6f4 v[236:239], v[112:119], a[120:127], v[236:239] cbsz:1\n"
      "ds_read_b64_tr_b8 v[102:103], v88 offset:9216\n"
      "s_waitcnt lgkmcnt(4)\n"
      "v_mfma_f32_16x16x128_f8f6f4 v[252:255], v[120:127], a[120:127], v[252:255] cbsz:1\n"
      "ds_read_b64_tr_b8 v[104:105], v89 offset:0\n"
      "v_mfma_f32_16x16x128_f8f6f4 v[248:251], v[120:127], a[112:119], v[248:251] cbsz:1\n"
      "ds_read_b64_tr_b8 v[106:107], v89 offset:1024\n"
      "v_mfma_f32_16x16x128_f8f6f4 v[244:247], v[120:127], a[104:111], v[244:247] cbsz:1\n"
      "ds_read_b64_tr_b8 v[108:109], v89 offset:8192\n"
      "v_mfma_f32_16x16x128_f8f6f4 v[240:243], v[120:127], a[96:103], v[240:243] cbsz:1\n"
      "ds_read_b64_tr_b8 v[110:111], v89 offset:9216\n"
      "s_cmp_eq_u32 %[smt], 1\n"
      "s_cbranch_scc1 Ltw227_%=\n"
      "s_waitcnt vmcnt(12)\n"
      "Ltw227_%=:\n"
      "s_waitcnt vmcnt(28)\n"
      "s_barrier\n"
      "v_readlane_b32 s90, v54, 10\n"
      "s_and_b32 s66, s90, 0xfffe0000\n"
      "s_and_b32 m0, s90, 0x1ffff\n"
      "s_nop 0\n"
      "buffer_load_dwordx4 v84, %[wsrd], s66 offen offset:0  lds\n"
      "buffer_load_dwordx4 v85, %[wsrd], s66 offen offset:1024  lds\n"
      "buffer_load_dwordx4 v86, %[wsrd], s66 offen offset:2048  lds\n"
      "buffer_load_dwordx4 v87, %[wsrd], s66 offen offset:3072  lds\n"
      "s_cmp_eq_u32 %[sps], 5\n"
      "s_cbranch_scc0 Lpb228_%=\n"
      "s_waitcnt vmcnt(12)\n"
      "s_mov_b64 exec, 1\n"
      "buffer_store_dword v62, off, %[psrd], %[spf] sc0 sc1\n"
      "s_mov_b64 exec, -1\n"
      "Lpb228_%=:\n"
      "s_mov_b32 s89, 3\n"
      "Lrp229_%=:\n"
      "s_waitcnt lgkmcnt(0)\n"
      "s_min_u32 s56, s56, s57\n"
      "s_min_u32 s58, s58, s59\n"
      "s_min_u32 s60, s60, s61\n"
      "s_min_u32 s62, s62, s63\n"
      "s_min_u32 s56, s56, s58\n"
      "s_min_u32 s60, s60, s62\n"
      "s_min_u32 s56, s56, s60\n"
      "s_cmp_ge_u32 s56, s68\n"
      "s_cbranch_scc1 Lrq230_%=\n"
      "s_add_u32 s91, s91, 64\n"
      "s_load_dwordx8 s[56:63], %[pfl], s91\n"
      "s_and_b32 s71, s91, 511\n"
      "s_cbranch_scc1 Lrp229_%=\n"
      "s_waitcnt lgkmcnt(0)\n"
      "s_call_b64 s[72:73], Lspin_%=\n"
      "Lrq230_%=:\n"
      "s_mov_b32 s89, 12\n"
      "v_readlane_b32 s65, v63, 4\n"
      "s_nop 4\n"
      "buffer_load_dwordx4 a[0:3], %[va0], %[asrd], s65 offen offset:0 sc0 sc1\n"
      "buffer_load_dwordx4 a[4:7], %[va0], %[asrd], s65 offen offset:1024 sc0 sc1\n"
      "buffer_load_dwordx4 a[8:11], %[va0], %[asrd], s65 offen offset:2048 sc0 sc1\n"
      "buffer_load_dwordx4 a[12:15], %[va0], %[asrd], s65 offen offset:3072 sc0 sc1\n"
      "buffer_load_dwordx4 a[16:19], v59, %[asrd], s65 offen offset:0 sc0 sc1\n"
      "buffer_load_dwordx4 a[20:23], v59, %[asrd], s65 offen offset:1024 sc0 sc1\n"
      "buffer_load_dwordx4 a[24:27], v59, %[asrd], s65 offen offset:2048 sc0 sc1\n"
      "buffer_load_dwordx4 a[28:31], v59, %[asrd], s65 offen offset:3072 sc0 sc1\n"
      "v_readlane_b32 s65, v63, 5\n"
      "s_nop 4\n"
      "buffer_load_dwordx4 a[32:35], %[va0], %[asrd], s65 offen offset:0 sc0 sc1\n"
      "buffer_load_dwordx4 a[36:39], %[va0], %[asrd], s65 offen offset:1024 sc0 sc1\n"
      "buffer_load_dwordx4 a[40:43], %[va0], %[asrd], s65 offen offset:2048 sc0 sc1\n"
      "buffer_load_dwordx4 a[44:47], %[va0], %[asrd], s65 offen offset:3072 sc0 sc1\n"
      "buffer_load_dwordx4 a[48:51], v59, %[asrd], s65 offen offset:0 sc0 sc1\n"
      "buffer_load_dwordx4 a[52:55], v59, %[asrd], s65 offen offset:1024 sc0 sc1\n"
      "buffer_load_dwordx4 a[56:59], v59, %[asrd], s65 offen offset:2048 sc0 sc1\n"
      "buffer_load_dwordx4 a[60:63], v59, %[asrd], s65 offen offset:3072 sc0 sc1\n"
      "s_waitcnt vmcnt(8)\n"
      "s_waitcnt lgkmcnt(4)\n"
      "v_mfma_f32_16x16x128_f8f6f4 v[128:131], v[96:103], a[0:7], v[128:131] cbsz:1\n"
      "ds_read_b64_tr_b8 v[112:113], v90 offset:0\n"
      "v_mfma_f32_16x16x128_f8f6f4 v[132:135], v[96:103], a[8:15], v[132:135] cbsz:1\n"
      "ds_read_b64_tr_b8 v[114:115], v90 offset:1024\n"
      "v_mfma_f32_16x16x128_f8f6f4 v[136:139], v[96:103], a[16:23], v[136:139] cbsz:1\n"
      "ds_read_b64_tr_b8 v[116:117], v90 offset:8192\n"
      "s_load_dwordx4 s[56:59], %[pfl], s70\n"
      "s_load_dwordx4 s[60:63], %[pfl], s70 offset:64\n"
      "v_mfma_f32_16x16x128_f8f6f4 v[140:143], v[96:103], a[24:31], v[140:143] cbsz:1\n"
      "ds_read_b64_tr_b8 v[118:119], v90 offset:9216\n"
      "s_cmp_eq_u32 %[sps], 6\n"
      "s_cbranch_scc0 Lpb231_%=\n"
      "s_waitcnt vmcnt(20)\n"
      "s_mov_b64 exec, 1\n"
      "buffer_store_dword v62, off, %[psrd], %[spf] sc0 sc1\n"
      "s_mov_b64 exec, -1\n"
      "Lpb231_%=:\n"
      "s_waitcnt lgkmcnt(4)\n"
      "v_mfma_f32_16x16x128_f8f6f4 v[156:159], v[104:111], a[24:31], v[156:159] cbsz:1\n"
      "ds_read_b64_tr_b8 v[120:121], v91 offset:0\n"
      "v_mfma_f32_16x16x128_f8f6f4 v[152:155], v[104:111], a[16:23], v[152:155] cbsz:1\n"
      "ds_read_b64_tr_b8 v[122:123], v91 offset:1024\n"
      "v_mfma_f32_16x16x128_f8f6f4 v[148:151], v[104:111], a[8:15], v[148:151] cbsz:1\n"
      "ds_read_b64_tr_b8 v[124:125], v91 offset:8192\n"
      "s_cmp_eq_u32 %[sps], 7\n"
      "s_cbranch_scc0 Lps232_%=\n"
      "buffer_store_dwordx4 v[64:67], %[vpd], %[psrd], s81 offen sc0 sc1\n"
      "buffer_store_dwordx4 v[68:71], %[vpd], %[psrd], s82 offen sc0 sc1\n"
      "Lps232_%=:\n"
      "v_mfma_f32_16x16x128_f8f6f4 v[144:147], v[104:111], a[0:7], v[144:147] cbsz:1\n"
      "ds_read_b64_tr_b8 v[126:127], v91 offset:9216\n"
      "s_waitcnt lgkmcnt(4)\n"
      "v_mfma_f32_16x16x128_f8f6f4 v[160:163], v[112:119], a[0:7], v[160:163] cbsz:1\n"
      "ds_read_b64_tr_b8 v[96:97], v92 offset:0\n"
      "v_mfma_f32_16x16x128_f8f6f4 v[164:167], v[112:119], a[8:15], v[164:167] cbsz:1\n"
      "ds_read_b64_tr_b8 v[98:99], v92 offset:1024\n"
      "v_mfma_f32_16x16x128_f8f6f4 v[168:171], v[112:119], a[16:23], v[168:171] cbsz:1\n"
      "ds_read_b64_tr_b8 v[100:101], v92 offset:8192\n"
      "v_mfma_f32_16x16x128_f8f6f4 v[172:175], v[112:119], a[24:31], v[172:175] cbsz:1\n"
      "ds_read_b64_tr_b8 v[102:103], v92 offset:9216\n"
      "s_waitcnt lgkmcnt(4)\n"
      "v_mfma_f32_16x16x128_f8f6f4 v[188:191], v[120:127], a[24:31], v[188:191] cbsz:1\n"
      "ds_read_b64_tr_b8 v[104:105], v93 offset:0\n"
      "ds_read_b64_tr_b8 v[106:107], v93 offset:1024\n"
      "v_mfma_f32_16x16x128_f8f6f4 v[184:187], v[120:127], a[16:23], v[184:187] cbsz:1\n"
      "ds_read_b64_tr_b8 v[108:109], v93 offset:8192\n"
      "ds_read_b64_tr_b8 v[110:111], v93 offset:9216\n"
      "v_mfma_f32_16x16x128_f8f6f4 v[180:183], v[120:127], a[8:15], v[180:183] cbsz:1\n"
      "v_mfma_f32_16x16x128_f8f6f4 v[176:179], v[120:127], a[0:7], v[176:179] cbsz:1\n"
      "s_waitcnt lgkmcnt(0)\n"
      "v_mfma_f32_16x16x128_f8f6f4 v[192:195], v[96:103], a[0:7], v[192:195] cbsz:1\n"
      "ds_read_b64_tr_b8 v[112:113], v94 offset:0\n"
      "s_min_u32 s56, s56, s57\n"
      "s_min_u32 s58, s58, s59\n"
      "s_min_u32 s60, s60, s61\n"
      "s_min_u32 s62, s62, s63\n"
      "v_mfma_f32_16x16x128_f8f6f4 v[196:199], v[96:103], a[8:15], v[196:199] cbsz:1\n"
      "ds_read_b64_tr_b8 v[114:115], v94 offset:1024\n"
      "s_min_u32 s56, s56, s58\n"
      "s_min_u32 s60, s60, s62\n"
      "s_min_u32 s56, s56, s60\n"
      "s_cmp_ge_u32 s56, s68\n"
      "s_cbranch_scc0 Lsl233_%=\n"
      "Lrt234_%=:\n"
      "v_readlane_b32 s65, v63, 6\n"
      "s_nop 1\n"
      "v_mfma_f32_16x16x128_f8f6f4 v[200:203], v[96:103], a[16:23], v[200:203] cbsz:1\n"
      "ds_read_b64_tr_b8 v[116:117], v94 offset:8192\n"
      "s_add_u32 s70, s70, 128\n"
      "s_min_u32 s70, s70, %[sfl]\n"
      "s_lshl_b32 s89, s89, 2\n"
      "v_mfma_f32_16x16x128_f8f6f4 v[204:207], v[96:103], a[24:31], v[204:207] cbsz:1\n"
      "ds_read_b64_tr_b8 v[118:119], v94 offset:9216\n"
      "buffer_load_dwordx4 a[64:67], %[va0], %[asrd], s65 offen offset:0 sc0 sc1\n"
      "s_waitcnt lgkmcnt(4)\n"
      "v_mfma_f32_16x16x128_f8f6f4 v[220:223], v[104:111], a[24:31], v[220:223] cbsz:1\n"
      "ds_read_b64_tr_b8 v[120:121], v95 offset:0\n"
      "v_mfma_f32_16x16x128_f8f6f4 v[216:219], v[104:111], a[16:23], v[216:219] cbsz:1\n"
      "ds_read_b64_tr_b8 v[122:123], v95 offset:1024\n"
      "buffer_load_dwordx4 a[68:71], %[va0], %[asrd], s65 offen offset:1024 sc0 sc1\n"
      "v_mfma_f32_16x16x128_f8f6f4 v[212:215], v[104:111], a[8:15], v[212:215] cbsz:1\n"
      "ds_read_b64_tr_b8 v[124:125], v95 offset:8192\n"
      "v_mfma_f32_16x16x128_f8f6f4 v[208:211], v[104:111], a[0:7], v[208:211] cbsz:1\n"
      "ds_read_b64_tr_b8 v[126:127], v95 offset:9216\n"
      "buffer_load_dwordx4 a[72:75], %[va0], %[asrd], s65 offen offset:2048 sc0 sc1\n"
      "s_waitcnt lgkmcnt(4)\n"
      "v_mfma_f32_16x16x128_f8f6f4 v[224:227], v[112:119], a[0:7], v[224:227] cbsz:1\n"
      "ds_read_b64_tr_b8 v[96:97], v88 offset:16384\n"
      "v_mfma_f32_16x16x128_f8f6f4 v[228:231], v[112:119], a[8:15], v[228:231] cbsz:1\n"
      "ds_read_b64_tr_b8 v[98:99], v88 offset:17408\n"
      "buffer_load_dwordx4 a[76:79], %[va0], %[asrd], s65 offen offset:3072 sc0 sc1\n"
      "v_mfma_f32_16x16x128_f8f6f4 v[232:235], v[112:119], a[16:23], v[232:235] cbsz:1\n"
      "ds_read_b64_tr_b8 v[100:101], v88 offset:24576\n"
      "v_mfma_f32_16x16x128_f8f6f4 v[236:239], v[112:119], a[24:31], v[236:239] cbsz:1\n"
      "ds_read_b64_tr_b8 v[102:103], v88 offset:25600\n"
      "buffer_load_dwordx4 a[80:83], v59, %[asrd], s65 offen offset:0 sc0 sc1\n"
      "s_waitcnt lgkmcnt(4)\n"
      "v_mfma_f32_16x16x128_f8f6f4 v[252:255], v[120:127], a[24:31], v[252:255] cbsz:1\n"
      "ds_read_b64_tr_b8 v[104:105], v89 offset:16384\n"
      "v_mfma_f32_16x16x128_f8f6f4 v[248:251], v[120:127], a[16:23], v[248:251] cbsz:1\n"
      "ds_read_b64_tr_b8 v[106:107], v89 offset:17408\n"
      "buffer_load_dwordx4 a[84:87], v59, %[asrd], s65 offen offset:1024 sc0 sc1\n"
      "v_mfma_f32_16x16x128_f8f6f4 v[244:247], v[120:127], a[8:15], v[244:247] cbsz:1\n"
      "ds_read_b64_tr_b8 v[108:109], v89 offset:24576\n"
      "v_mfma_f32_16x16x128_f8f6f4 v[240:243], v[120:127], a[0:7], v[240:243] cbsz:1\n"
      "ds_read_b64_tr_b8 v[110:111], v89 offset:25600\n"
      "buffer_load_dwordx4 a[88:91], v59, %[asrd], s65 offen offset:2048 sc0 sc1\n"
      "s_waitcnt vmcnt(7)\n"
      "s_barrier\n"
      "s_waitcnt lgkmcnt(4)\n"
      "v_mfma_f32_16x16x128_f8f6f4 v[128:131], v[96:103], a[32:39], v[128:131] cbsz:1\n"
      "ds_read_b64_tr_b8 v[112:113], v90 offset:16384\n"
      "buffer_load_dwordx4 a[92:95], v59, %[asrd], s65 offen offset:3072 sc0 sc1\n"
      "v_mfma_f32_16x16x128_f8f6f4 v[132:135], v[96:103], a[40:47], v[132:135] cbsz:1\n"
      "ds_read_b64_tr_b8 v[114:115], v90 offset:17408\n"
      "v_readlane_b32 s65, v63, 7\n"
      "s_nop 1\n"
      "v_mfma_f32_16x16x128_f8f6f4 v[136:139], v[96:103], a[48:55], v[136:139] cbsz:1\n"
      "ds_read_b64_tr_b8 v[116:117], v90 offset:24576\n"
      "v_mfma_f32_16x16x128_f8f6f4 v[140:143], v[96:103], a[56:63], v[140:143] cbsz:1\n"
      "ds_read_b64_tr_b8 v[118:119], v90 offset:25600\n"
      "buffer_load_dwordx4 a[96:99], %[va0], %[asrd], s65 offen offset:0 sc0 sc1\n"
      "s_waitcnt lgkmcnt(4)\n"
      "v_mfma_f32_16x16x128_f8f6f4 v[156:159], v[104:111], a[56:63], v[156:159] cbsz:1\n"
      "ds_read_b64_tr_b8 v[120:121], v91 offset:16384\n"
      "s_load_dwordx4 s[56:59], %[pfl], s70\n"
      "s_load_dwordx4 s[60:63], %[pfl], s70 offset:64\n"
      "v_mfma_f32_16x16x128_f8f6f4 v[152:155], v[104:111], a[48:55], v[152:155] cbsz:1\n"
      "ds_read_b64_tr_b8 v[122:123], v91 offset:17408\n"
      "buffer_load_dwordx4 a[100:103], %[va0], %[asrd], s65 offen offset:1024 sc0 sc1\n"
      "v_mfma_f32_16x16x128_f8f6f4 v[148:151], v[104:111], a[40:47], v[148:151] cbsz:1\n"
      "ds_read_b64_tr_b8 v[124:125], v91 offset:24576\n"
      "v_mfma_f32_16x16x128_f8f6f4 v[144:147], v[104:111], a[32:39], v[144:147] cbsz:1\n"
      "ds_read_b64_tr_b8 v[126:127], v91 offset:25600\n"
      "buffer_load_dwordx4 a[104:107], %[va0], %[asrd], s65 offen offset:2048 sc0 sc1\n"
      "s_waitcnt lgkmcnt(4)\n"
      "v_mfma_f32_16x16x128_f8f6f4 v[160:163], v[112:119], a[32:39], v[160:163] cbsz:1\n"
      "ds_read_b64_tr_b8 v[96:97], v92 offset:16384\n"
      "v_mfma_f32_16x16x128_f8f6f4 v[164:167], v[112:119], a[40:47], v[164:167] cbsz:1\n"
      "ds_read_b64_tr_b8 v[98:99], v92 offset:17408\n"
      "buffer_load_dwordx4 a[108:111], %[va0], %[asrd], s65 offen offset:3072 sc0 sc1\n"
      "v_mfma_f32_16x16x128_f8f6f4 v[168:171], v[112:119], a[48:55], v[168:171] cbsz:1\n"
      "ds_read_b64_tr_b8 v[100:101], v92 offset:24576\n"
      "v_mfma_f32_16x16x128_f8f6f4 v[172:175], v[112:119], a[56:63], v[172:175] cbsz:1\n"
      "ds_read_b64_tr_b8 v[102:103], v92 offset:25600\n"
      "buffer_load_dwordx4 a[112:115], v59, %[asrd], s65 offen offset:0 sc0 sc1\n"
      "s_waitcnt lgkmcnt(4)\n"
      "v_mfma_f32_16x16x128_f8f6f4 v[188:191], v[120:127], a[56:63], v[188:191] cbsz:1\n"
      "ds_read_b64_tr_b8 v[104:105], v93 offset:16384\n"
      "v_mfma_f32_16x16x128_f8f6f4 v[184:187], v[120:127], a[48:55], v[184:187] cbsz:1\n"
      "ds_read_b64_tr_b8 v[106:107], v93 offset:17408\n"
      "buffer_load_dwordx4 a[116:119], v59, %[asrd], s65 offen offset:1024 sc0 sc1\n"
      "v_mfma_f32_16x16x128_f8f6f4 v[180:183], v[120:127], a[40:47], v[180:183] cbsz:1\n"
      "ds_read_b64_tr_b8 v[108:109], v93 offset:24576\n"
      "v_mfma_f32_16x16x128_f8f6f4 v[176:179], v[120:127], a[32:39], v[176:179] cbsz:1\n"
      "ds_read_b64_tr_b8 v[110:111], v93 offset:25600\n"
      "buffer_load_dwordx4 a[120:123], v59, %[asrd], s65 offen offset:2048 sc0 sc1\n"
      "s_waitcnt lgkmcnt(4)\n"
      "v_mfma_f32_16x16x128_f8f6f4 v[192:195], v[96:103], a[32:39], v[192:195] cbsz:1\n"
      "ds_read_b64_tr_b8 v[112:113], v94 offset:16384\n"
      "ds_read_b64_tr_b8 v[114:115], v94 offset:17408\n"
      "v_mfma_f32_16x16x128_f8f6f4 v[196:199], v[96:103], a[40:47], v[196:199] cbsz:1\n"
      "ds_read_b64_tr_b8 v[116:117], v94 offset:24576\n"
      "ds_read_b64_tr_b8 v[118:119], v94 offset:25600\n"
      "buffer_load_dwordx4 a[124:127], v59, %[asrd], s65 offen offset:3072 sc0 sc1\n"
      "v_mfma_f32_16x16x128_f8f6f4 v[200:203], v[96:103], a[48:55], v[200:203] cbsz:1\n"
      "v_mfma_f32_16x16x128_f8f6f4 v[204:207], v[96:103], a[56:63], v[204:207] cbsz:1\n"
      "s_waitcnt lgkmcnt(0)\n"
      "v_mfma_f32_16x16x128_f8f6f4 v[220:223], v[104:111], a[56:63], v[220:223] cbsz:1\n"
      "ds_read_b64_tr_b8 v[120:121], v95 offset:16384\n"
      "s_min_u32 s56, s56, s57\n"
      "s_min_u32 s58, s58, s59\n"
      "s_min_u32 s60, s60, s61\n"
      "s_min_u32 s62, s62, s63\n"
      "v_mfma_f32_16x16x128_f8f6f4 v[216:219], v[104:111], a[48:55], v[216:219] cbsz:1\n"
      "ds_read_b64_tr_b8 v[122:123], v95 offset:17408\n"
      "s_min_u32 s56, s56, s58\n"
      "s_min_u32 s60, s60, s62\n"
      "s_min_u32 s56, s56, s60\n"
      "s_cmp_ge_u32 s56, s68\n"
      "s_cbranch_scc0 Lsl235_%=\n"
      "Lrt236_%=:\n"
      "v_mfma_f32_16x16x128_f8f6f4 v[212:215], v[104:111], a[40:47], v[212:215] cbsz:1\n"
      "ds_read_b64_tr_b8 v[124:125], v95 offset:24576\n"
      "s_add_u32 s70, s70, 128\n"
      "s_min_u32 s70, s70, %[sfl]\n"
      "s_lshl_b32 s89, s89, 2\n"
      "s_load_dwordx4 s[56:59], %[pfl], s70\n"
      "s_load_dwordx4 s[60:63], %[pfl], s70 offset:64\n"
      "v_readlane_b32 s65, v63, 8\n"
      "s_nop 1\n"
      "v_mfma_f32_16x16x128_f8f6f4 v[208:211], v[104:111], a[32:39], v[208:211] cbsz:1\n"
      "ds_read_b64_tr_b8 v[126:127], v95 offset:25600\n"
      "s_waitcnt lgkmcnt(4)\n"
      "v_mfma_f32_16x16x128_f8f6f4 v[224:227], v[112:119], a[32:39], v[224:227] cbsz:1\n"
      "ds_read_b64_tr_b8 v[96:97], v88 offset:32768\n"
      "buffer_load_dwordx4 a[0:3], %[va0], %[asrd], s65 offen offset:0 sc0 sc1\n"
      "v_mfma_f32_16x16x128_f8f6f4 v[228:231], v[112:119], a[40:47], v[228:231] cbsz:1\n"
      "ds_read_b64_tr_b8 v[98:99], v88 offset:33792\n"
      "v_mfma_f32_16x16x128_f8f6f4 v[232:235], v[112:119], a[48:55], v[232:235] cbsz:1\n"
      "ds_read_b64_tr_b8 v[100:101], v88 offset:40960\n"
      "buffer_load_dwordx4 a[4:7], %[va0], %[asrd], s65 offen offset:1024 sc0 sc1\n"
      "v_mfma_f32_16x16x128_f8f6f4 v[236:239], v[112:119], a[56:63], v[236:239] cbsz:1\n"
      "ds_read_b64_tr_b8 v[102:103], v88 offset:41984\n"
      "s_waitcnt lgkmcnt(4)\n"
      "v_mfma_f32_16x16x128_f8f6f4 v[252:255], v[120:127], a[56:63], v[252:255] cbsz:1\n"
      "ds_read_b64_tr_b8 v[104:105], v89 offset:32768\n"
      "buffer_load_dwordx4 a[8:11], %[va0], %[asrd], s65 offen offset:2048 sc0 sc1\n"
      "v_mfma_f32_16x16x128_f8f6f4 v[248:251], v[120:127], a[48:55], v[248:251] cbsz:1\n"
      "ds_read_b64_tr_b8 v[106:107], v89 offset:33792\n"
      "v_mfma_f32_16x16x128_f8f6f4 v[244:247], v[120:127], a[40:47], v[244:247] cbsz:1\n"
      "ds_read_b64_tr_b8 v[108:109], v89 offset:40960\n"
      "buffer_load_dwordx4 a[12:15], %[va0], %[asrd], s65 offen offset:3072 sc0 sc1\n"
      "v_mfma_f32_16x16x128_f8f6f4 v[240:243], v[120:127], a[32:39], v[240:243] cbsz:1\n"
      "ds_read_b64_tr_b8 v[110:111], v89 offset:41984\n"
      "s_waitcnt vmcnt(12)\n"
      "s_barrier\n"
      "s_waitcnt lgkmcnt(4)\n"
      "v_mfma_f32_16x16x128_f8f6f4 v[128:131], v[96:103], a[64:71], v[128:131] cbsz:1\n"
      "ds_read_b64_tr_b8 v[112:113], v90 offset:32768\n"
      "v_mfma_f32_16x16x128_f8f6f4 v[132:135], v[96:103], a[72:79], v[132:135] cbsz:1\n"
      "ds_read_b64_tr_b8 v[114:115], v90 offset:33792\n"
      "v_readlane_b32 s90, v54, 11\n"
      "s_and_b32 s66, s90, 0xfffe0000\n"
      "s_and_b32 m0, s90, 0x1ffff\n"
      "s_nop 0\n"
      "buffer_load_dwordx4 v84, %[wsrd], s66 offen offset:0  lds\n"
      "v_mfma_f32_16x16x128_f8f6f4 v[136:139], v[96:103], a[80:87], v[136:139] cbsz:1\n"
      "ds_read_b64_tr_b8 v[116:117], v90 offset:40960\n"
      "v_mfma_f32_16x16x128_f8f6f4 v[140:143], v[96:103], a[88:95], v[140:143] cbsz:1\n"
      "ds_read_b64_tr_b8 v[118:119], v90 offset:41984\n"
      "buffer_load_dwordx4 v85, %[wsrd], s66 offen offset:1024  lds\n"
      "s_waitcnt lgkmcnt(4)\n"
      "v_mfma_f32_16x16x128_f8f6f4 v[156:159], v[104:111], a[88:95], v[156:159] cbsz:1\n"
      "ds_read_b64_tr_b8 v[120:121], v91 offset:32768\n"
      "v_mfma_f32_16x16x128_f8f6f4 v[152:155], v[104:111], a[80:87], v[152:155] cbsz:1\n"
      "ds_read_b64_tr_b8 v[122:123], v91 offset:33792\n"
      "buffer_load_dwordx4 v86, %[wsrd], s66 offen offset:2048  lds\n"
      "v_mfma_f32_16x16x128_f8f6f4 v[148:151], v[104:111], a[72:79], v[148:151] cbsz:1\n"
      "ds_read_b64_tr_b8 v[124:125], v91 offset:40960\n"
      "v_mfma_f32_16x16x128_f8f6f4 v[144:147], v[104:111], a[64:71], v[144:147] cbsz:1\n"
      "ds_read_b64_tr_b8 v[126:127], v91 offset:41984\n"
      "buffer_load_dwordx4 v87, %[wsrd], s66 offen offset:3072  lds\n"
      "s_waitcnt lgkmcnt(4)\n"
      "v_mfma_f32_16x16x128_f8f6f4 v[160:163], v[112:119], a[64:71], v[160:163] cbsz:1\n"
      "ds_read_b64_tr_b8 v[96:97], v92 offset:32768\n"
      "v_mfma_f32_16x16x128_f8f6f4 v[164:167], v[112:119], a[72:79], v[164:167] cbsz:1\n"
      "ds_read_b64_tr_b8 v[98:99], v92 offset:33792\n"
      "buffer_load_dwordx4 a[16:19], v59, %[asrd], s65 offen offset:0 sc0 sc1\n"
      "v_mfma_f32_16x16x128_f8f6f4 v[168:171], v[112:119], a[80:87], v[168:171] cbsz:1\n"
      "ds_read_b64_tr_b8 v[100:101], v92 offset:40960\n"
      "v_mfma_f32_16x16x128_f8f6f4 v[172:175], v[112:119], a[88:95], v[172:175] cbsz:1\n"
      "ds_read_b64_tr_b8 v[102:103], v92 offset:41984\n"
      "buffer_load_dwordx4 a[20:23], v59, %[asrd], s65 offen offset:1024 sc0 sc1\n"
      "s_waitcnt lgkmcnt(4)\n"
      "v_mfma_f32_16x16x128_f8f6f4 v[188:191], v[120:127], a[88:95], v[188:191] cbsz:1\n"
      "ds_read_b64_tr_b8 v[104:105], v93 offset:32768\n"
      "s_cmp_eq_u32 %[sps], 7\n"
      "s_cbranch_scc0 Lpb237_%=\n"
      "s_waitcnt vmcnt(26)\n"
      "s_mov_b64 exec, 1\n"
      "buffer_store_dword v62, off, %[psrd], %[spf] sc0 sc1\n"
      "s_mov_b64 exec, -1\n"
      "Lpb237_%=:\n"
      "v_mfma_f32_16x16x128_f8f6f4 v[184:187], v[120:127], a[80:87], v[184:187] cbsz:1\n"
      "ds_read_b64_tr_b8 v[106:107], v93 offset:33792\n"
      "buffer_load_dwordx4 a[24:27], v59, %[asrd], s65 offen offset:2048 sc0 sc1\n"
      "v_mfma_f32_16x16x128_f8f6f4 v[180:183], v[120:127], a[72:79], v[180:183] cbsz:1\n"
      "ds_read_b64_tr_b8 v[108:109], v93 offset:40960\n"
      "v_mfma_f32_16x16x128_f8f6f4 v[176:179], v[120:127], a[64:71], v[176:179] cbsz:1\n"
      "ds_read_b64_tr_b8 v[110:111], v93 offset:41984\n"
      "buffer_load_dwordx4 a[28:31], v59, %[asrd], s65 offen offset:3072 sc0 sc1\n"
      "s_waitcnt lgkmcnt(4)\n"
      "v_mfma_f32_16x16x128_f8f6f4 v[192:195], v[96:103], a[64:71], v[192:195] cbsz:1\n"
      "ds_read_b64_tr_b8 v[112:113], v94 offset:32768\n"
      "v_readlane_b32 s65, v63, 9\n"
      "s_nop 1\n"
      "v_mfma_f32_16x16x128_f8f6f4 v[196:199], v[96:103], a[72:79], v[196:199] cbsz:1\n"
      "ds_read_b64_tr_b8 v[114:115], v94 offset:33792\n"
      "v_mfma_f32_16x16x128_f8f6f4 v[200:203], v[96:103], a[80:87], v[200:203] cbsz:1\n"
      "ds_read_b64_tr_b8 v[116:117], v94 offset:40960\n"
      "buffer_load_dwordx4 a[32:35], %[va0], %[asrd], s65 offen offset:0 sc0 sc1\n"
      "v_mfma_f32_16x16x128_f8f6f4 v[204:207], v[96:103], a[88:95], v[204:207] cbsz:1\n"
      "ds_read_b64_tr_b8 v[118:119], v94 offset:41984\n"
      "s_waitcnt lgkmcnt(4)\n"
      "v_mfma_f32_16x16x128_f8f6f4 v[220:223], v[104:111], a[88:95], v[220:223] cbsz:1\n"
      "ds_read_b64_tr_b8 v[120:121], v95 offset:32768\n"
      "buffer_load_dwordx4 a[36:39], %[va0], %[asrd], s65 offen offset:1024 sc0 sc1\n"
      "v_mfma_f32_16x16x128_f8f6f4 v[216:219], v[104:111], a[80:87], v[216:219] cbsz:1\n"
      "ds_read_b64_tr_b8 v[122:123], v95 offset:33792\n"
      "v_mfma_f32_16x16x128_f8f6f4 v[212:215], v[104:111], a[72:79], v[212:215] cbsz:1\n"
      "ds_read_b64_tr_b8 v[124:125], v95 offset:40960\n"
      "buffer_load_dwordx4 a[40:43], %[va0], %[asrd], s65 offen offset:2048 sc0 sc1\n"
      "v_mfma_f32_16x16x128_f8f6f4 v[208:211], v[104:111], a[64:71], v[208:211] cbsz:1\n"
      "ds_read_b64_tr_b8 v[126:127], v95 offset:41984\n"
      "s_waitcnt lgkmcnt(4)\n"
      "v_mfma_f32_16x16x128_f8f6f4 v[224:227], v[112:119], a[64:71], v[224:227] cbsz:1\n"
      "ds_read_b64_tr_b8 v[96:97], v88 offset:49152\n"
      "buffer_load_dwordx4 a[44:47], %[va0], %[asrd], s65 offen offset:3072 sc0 sc1\n"
      "v_mfma_f32_16x16x128_f8f6f4 v[228:231], v[112:119], a[72:79], v[228:231] cbsz:1\n"
      "ds_read_b64_tr_b8 v[98:99], v88 offset:50176\n"
      "v_mfma_f32_16x16x128_f8f6f4 v[232:235], v[112:119], a[80:87], v[232:235] cbsz:1\n"
      "ds_read_b64_tr_b8 v[100:101], v88 offset:57344\n"
      "buffer_load_dwordx4 a[48:51], v59, %[asrd], s65 offen offset:0 sc0 sc1\n"
      "v_mfma_f32_16x16x128_f8f6f4 v[236:239], v[112:119], a[88:95], v[236:239] cbsz:1\n"
      "ds_read_b64_tr_b8 v[102:103], v88 offset:58368\n"
      "v_xor_b32_e32 v88, 0x10000, v88\n"
      "s_waitcnt lgkmcnt(4)\n"
      "v_mfma_f32_16x16x128_f8f6f4 v[252:255], v[120:127], a[88:95], v[252:255] cbsz:1\n"
      "ds_read_b64_tr_b8 v[104:105], v89 offset:49152\n"
      "ds_read_b64_tr_b8 v[106:107], v89 offset:50176\n"
      "buffer_load_dwordx4 a[52:55], v59, %[asrd], s65 offen offset:1024 sc0 sc1\n"
      "v_mfma_f32_16x16x128_f8f6f4 v[248:251], v[120:127], a[80:87], v[248:251] cbsz:1\n"
      "ds_read_b64_tr_b8 v[108:109], v89 offset:57344\n"
      "ds_read_b64_tr_b8 v[110:111], v89 offset:58368\n"
      "v_xor_b32_e32 v89, 0x10000, v89\n"
      "v_mfma_f32_16x16x128_f8f6f4 v[244:247], v[120:127], a[72:79], v[244:247] cbsz:1\n"
      "buffer_load_dwordx4 a[56:59], v59, %[asrd], s65 offen offset:2048 sc0 sc1\n"
      "v_mfma_f32_16x16x128_f8f6f4 v[240:243], v[120:127], a[64:71], v[240:243] cbsz:1\n"
      "s_waitcnt vmcnt(19)\n"
      "s_barrier\n"
      "s_waitcnt lgkmcnt(0)\n"
      "v_mfma_f32_16x16x128_f8f6f4 v[128:131], v[96:103], a[96:103], v[128:131] cbsz:1\n"
      "ds_read_b64_tr_b8 v[112:113], v90 offset:49152\n"
      "s_min_u32 s56, s56, s57\n"
      "s_min_u32 s58, s58, s59\n"
      "s_min_u32 s60, s60, s61\n"
      "s_min_u32 s62, s62, s63\n"
      "v_mfma_f32_16x16x128_f8f6f4 v[132:135], v[96:103], a[104:111], v[132:135] cbsz:1\n"
      "ds_read_b64_tr_b8 v[114:115], v90 offset:50176\n"
      "s_min_u32 s56, s56, s58\n"
      "s_min_u32 s60, s60, s62\n"
      "s_min_u32 s56, s56, s60\n"
      "s_cmp_ge_u32 s56, s68\n"
      "s_cbranch_scc0 Lsl238_%=\n"
      "Lrt239_%=:\n"
      "v_readlane_b32 s90, v54, 12\n"
      "s_and_b32 s66, s90, 0xfffe0000\n"
      "s_and_b32 m0, s90, 0x1ffff\n"
      "s_nop 0\n"
      "buffer_load_dwordx4 v84, %[wsrd], s66 offen offset:0  lds\n"
      "v_mfma_f32_16x16x128_f8f6f4 v[136:139], v[96:103], a[112:119], v[136:139] cbsz:1\n"
      "ds_read_b64_tr_b8 v[116:117], v90 offset:57344\n"
      "s_add_u32 s70, s70, 128\n"
      "s_min_u32 s70, s70, %[sfl]\n"
      "s_lshl_b32 s89, s89, 2\n"
      "s_load_dwordx4 s[56:59], %[pfl], s70\n"
      "s_load_dwordx4 s[60:63], %[pfl], s70 offset:64\n"
      "v_mfma_f32_16x16x128_f8f6f4 v[140:143], v[96:103], a[120:127], v[140:143] cbsz:1\n"
      "ds_read_b64_tr_b8 v[118:119], v90 offset:58368\n"
      "v_xor_b32_e32 v90, 0x10000, v90\n"
      "buffer_load_dwordx4 v85, %[wsrd], s66 offen offset:1024  lds\n"
      "s_waitcnt lgkmcnt(4)\n"
      "v_mfma_f32_16x16x128_f8f6f4 v[156:159], v[104:111], a[120:127], v[156:159] cbsz:1\n"
      "ds_read_b64_tr_b8 v[120:121], v91 offset:49152\n"
      "v_mfma_f32_16x16x128_f8f6f4 v[152:155], v[104:111], a[112:119], v[152:155] cbsz:1\n"
      "ds_read_b64_tr_b8 v[122:123], v91 offset:50176\n"
      "buffer_load_dwordx4 v86, %[wsrd], s66 offen offset:2048  lds\n"
      "v_mfma_f32_16x16x128_f8f6f4 v[148:151], v[104:111], a[104:111], v[148:151] cbsz:1\n"
      "ds_read_b64_tr_b8 v[124:125], v91 offset:57344\n"
      "v_mfma_f32_16x16x128_f8f6f4 v[144:147], v[104:111], a[96:103], v[144:147] cbsz:1\n"
      "ds_read_b64_tr_b8 v[126:127], v91 offset:58368\n"
      "v_xor_b32_e32 v91, 0x10000, v91\n"
      "buffer_load_dwordx4 v87, %[wsrd], s66 offen offset:3072  lds\n"
      "s_waitcnt lgkmcnt(4)\n"
      "v_mfma_f32_16x16x128_f8f6f4 v[160:163], v[112:119], a[96:103], v[160:163] cbsz:1\n"
      "ds_read_b64_tr_b8 v[96:97], v92 offset:49152\n"
      "v_mfma_f32_16x16x128_f8f6f4 v[164:167], v[112:119], a[104:111], v[164:167] cbsz:1\n"
      "ds_read_b64_tr_b8 v[98:99], v92 offset:50176\n"
      "buffer_load_dwordx4 a[60:63], v59, %[asrd], s65 offen offset:3072 sc0 sc1\n"
      "v_mfma_f32_16x16x128_f8f6f4 v[168:171], v[112:119], a[112:119], v[168:171] cbsz:1\n"
      "ds_read_b64_tr_b8 v[100:101], v92 offset:57344\n"
      "v_mfma_f32_16x16x128_f8f6f4 v[172:175], v[112:119], a[120:127], v[172:175] cbsz:1\n"
      "ds_read_b64_tr_b8 v[102:103], v92 offset:58368\n"
      "v_xor_b32_e32 v92, 0x10000, v92\n"
      "s_waitcnt lgkmcnt(4)\n"
      "v_mfma_f32_16x16x128_f8f6f4 v[188:191], v[120:127], a[120:127], v[188:191] cbsz:1\n"
      "ds_read_b64_tr_b8 v[104:105], v93 offset:49152\n"
      "v_readlane_b32 s65, v63, 10\n"
      "s_nop 1\n"
      "v_mfma_f32_16x16x128_f8f6f4 v[184:187], v[120:127], a[112:119], v[184:187] cbsz:1\n"
      "ds_read_b64_tr_b8 v[106:107], v93 offset:50176\n"
      "v_mfma_f32_16x16x128_f8f6f4 v[180:183], v[120:127], a[104:111], v[180:183] cbsz:1\n"
      "ds_read_b64_tr_b8 v[108:109], v93 offset:57344\n"
      "buffer_load_dwordx4 a[64:67], %[va0], %[asrd], s65 offen offset:0 sc0 sc1\n"
      "v_mfma_f32_16x16x128_f8f6f4 v[176:179], v[120:127], a[96:103], v[176:179] cbsz:1\n"
      "ds_read_b64_tr_b8 v[110:111], v93 offset:58368\n"
      "v_xor_b32_e32 v93, 0x10000, v93\n"
      "s_waitcnt lgkmcnt(4)\n"
      "v_mfma_f32_16x16x128_f8f6f4 v[192:195], v[96:103], a[96:103], v[192:195] cbsz:1\n"
      "ds_read_b64_tr_b8 v[112:113], v94 offset:49152\n"
      "buffer_load_dwordx4 a[68:71], %[va0], %[asrd], s65 offen offset:1024 sc0 sc1\n"
      "v_mfma_f32_16x16x128_f8f6f4 v[196:199], v[96:103], a[104:111], v[196:199] cbsz:1\n"
      "ds_read_b64_tr_b8 v[114:115], v94 offset:50176\n"
      "v_mfma_f32_16x16x128_f8f6f4 v[200:203], v[96:103], a[112:119], v[200:203] cbsz:1\n"
      "ds_read_b64_tr_b8 v[116:117], v94 offset:57344\n"
      "buffer_load_dwordx4 a[72:75], %[va0], %[asrd], s65 offen offset:2048 sc0 sc1\n"
      "v_mfma_f32_16x16x128_f8f6f4 v[204:207], v[96:103], a[120:127], v[204:207] cbsz:1\n"
      "ds_read_b64_tr_b8 v[118:119], v94 offset:58368\n"
      "v_xor_b32_e32 v94, 0x10000, v94\n"
      "s_waitcnt lgkmcnt(4)\n"
      "v_mfma_f32_16x16x128_f8f6f4 v[220:223], v[104:111], a[120:127], v[220:223] cbsz:1\n"
      "ds_read_b64_tr_b8 v[120:121], v95 offset:49152\n"
      "buffer_load_dwordx4 a[76:79], %[va0], %[asrd], s65 offen offset:3072 sc0 sc1\n"
      "v_mfma_f32_16x16x128_f8f6f4 v[216:219], v[104:111], a[112:119], v[216:219] cbsz:1\n"
      "ds_read_b64_tr_b8 v[122:123], v95 offset:50176\n"
      "v_mfma_f32_16x16x128_f8f6f4 v[212:215], v[104:111], a[104:111], v[212:215] cbsz:1\n"
      "ds_read_b64_tr_b8 v[124:125], v95 offset:57344\n"
      "buffer_load_dwordx4 a[80:83], v59, %[asrd], s65 offen offset:0 sc0 sc1\n"
      "v_mfma_f32_16x16x128_f8f6f4 v[208:211], v[104:111], a[96:103], v[208:211] cbsz:1\n"
      "ds_read_b64_tr_b8 v[126:127], v95 offset:58368\n"
      "v_xor_b32_e32 v95, 0x10000, v95\n"
      "s_waitcnt lgkmcnt(4)\n"
      "v_mfma_f32_16x16x128_f8f6f4 v[224:227], v[112:119], a[96:103], v[224:227] cbsz:1\n"
      "ds_read_b64_tr_b8 v[96:97], v88 offset:0\n"
      "buffer_load_dwordx4 a[84:87], v59, %[asrd], s65 offen offset:1024 sc0 sc1\n"
      "v_mfma_f32_16x16x128_f8f6f4 v[228:231], v[112:119], a[104:111], v[228:231] cbsz:1\n"
      "ds_read_b64_tr_b8 v[98:99], v88 offset:1024\n"
      "v_mfma_f32_16x16x128_f8f6f4 v[232:235], v[112:119], a[112:119], v[232:235] cbsz:1\n"
      "ds_read_b64_tr_b8 v[100:101], v88 offset:8192\n"
      "buffer_load_dwordx4 a[88:91], v59, %[asrd], s65 offen offset:2048 sc0 sc1\n"
      "v_mfma_f32_16x16x128_f8f6f4 v[236:239], v[112:119], a[120:127], v[236:239] cbsz:1\n"
      "ds_read_b64_tr_b8 v[102:103], v88 offset:9216\n"
      "s_waitcnt lgkmcnt(4)\n"
      "v_mfma_f32_16x16x128_f8f6f4 v[252:255], v[120:127], a[120:127], v[252:255] cbsz:1\n"
      "ds_read_b64_tr_b8 v[104:105], v89 offset:0\n"
      "buffer_load_dwordx4 a[92:95], v59, %[asrd], s65 offen offset:3072 sc0 sc1\n"
      "v_mfma_f32_16x16x128_f8f6f4 v[248:251], v[120:127], a[112:119], v[248:251] cbsz:1\n"
      "ds_read_b64_tr_b8 v[106:107], v89 offset:1024\n"
      "v_mfma_f32_16x16x128_f8f6f4 v[244:247], v[120:127], a[104:111], v[244:247] cbsz:1\n"
      "ds_read_b64_tr_b8 v[108:109], v89 offset:8192\n"
      "v_mfma_f32_16x16x128_f8f6f4 v[240:243], v[120:127], a[96:103], v[240:243] cbsz:1\n"
      "ds_read_b64_tr_b8 v[110:111], v89 offset:9216\n"
      "s_mov_b32 s64, 8\n"
      "s_branch Ljoin_%=\n"
      "Lsl233_%=:\n"
      "s_call_b64 s[72:73], Lspin_%=\n"
      "s_branch Lrt234_%=\n"
      "Lsl235_%=:\n"
      "s_call_b64 s[72:73], Lspin_%=\n"
      "s_branch Lrt236_%=\n"
      "Lsl238_%=:\n"
      "s_call_b64 s[72:73], Lspin_%=\n"
      "s_branch Lrt239_%=\n"
      "Lsl256_%=:\n"
      "s_call_b64 s[72:73], Lspin_%=\n"
      "s_branch Lrt257_%=\n"
      "Lsl261_%=:\n"
      "s_call_b64 s[72:73], Lspin_%=\n"
      "s_branch Lrt262_%=\n"
      "Lsl265_%=:\n"
      "s_call_b64 s[72:73], Lspin_%=\n"
      "s_branch Lrt266_%=\n"
      "Lsl269_%=:\n"
      "s_call_b64 s[72:73], Lspin_%=\n"
      "s_branch Lrt270_%=\n"
      "Lsl271_%=:\n"
      "s_call_b64 s[72:73], Lspin_%=\n"
      "s_branch Lrt272_%=\n"
      "Lsl273_%=:\n"
      "s_call_b64 s[72:73], Lspin_%=\n"
      "s_branch Lrt274_%=\n"
      "Lsl275_%=:\n"
      "s_call_b64 s[72:73], Lspin_%=\n"
      "s_branch Lrt276_%=\n"
      "Lsl280_%=:\n"
      "s_call_b64 s[72:73], Lspin_%=\n"
      "s_branch Lrt281_%=\n"
      "Lsl282_%=:\n"
      "s_call_b64 s[72:73], Lspin_%=\n"
      "s_branch Lrt283_%=\n"
      "Lspin_%=:\n"
      "s_andn2_b32 vcc_lo, s89, s67\n"
      "s_cbranch_scc0 Lspin2_%=\n"
      "s_memrealtime s[74:75]\n"
      "Lspin1_%=:\n"
      "buffer_load_dwordx4 v[72:75], %[vfl], %[asrd], 0 offen sc0 sc1\n"
      "buffer_load_dword v58, off, %[asrd], %[stmo] sc0 sc1\n"
      "s_memrealtime s[76:77]\n"
      "s_waitcnt vmcnt(0) lgkmcnt(0)\n"
      "v_min3_u32 v72, v72, v73, v74\n"
      "v_min_u32_e32 v72, v72, v75\n"
      "v_cmp_le_u32_e32 vcc, s68, v72\n"
      "v_readfirstlane_b32 s77, v58\n"
      "s_or_b32 s67, s67, vcc_lo\n"
      "s_andn2_b32 vcc_lo, s89, s67\n"
      "s_cbranch_scc0 Lspin2_%=\n"
      "s_cmp_eq_u32 s77, s68\n"
      "s_cbranch_scc1 Lspin3_%=\n"
      "s_sub_u32 s76, s76, s74\n"
      "s_cmp_gt_u32 s76, 0x2faf080\n"
      "s_cbranch_scc1 Lspin3_%=\n"
      "s_sleep 1\n"
      "s_branch Lspin1_%=\n"
      "Lspin3_%=:\n"
      "s_mov_b64 exec, 1\n"
      "buffer_store_dword v62, off, %[asrd], %[stmo] sc0 sc1\n"
      "s_mov_b64 exec, -1\n"
      "s_mov_b32 s83, 0x7fc00000\n"
      "s_mov_b32 s67, -1\n"
      "Lspin2_%=:\n"
      "s_setpc_b64 s[72:73]\n"
      "Lend_%=:\n"
      ".La2a_end:\n"
      :
      : [wsrd] "s"(wsrd), [xsrd] "s"(xsrd), [osrd] "s"(osrd), [asrd] "s"(asrd), [psrd] "s"(psrd),
        [psx] "s"(psx), [psw] "s"(psw), [pfl] "s"(pfl), [rank] "s"(rank), [sm0] "s"(sm0), [sps] "s"(sps), [spf] "s"(spf),
        [sctr] "s"(sctr), [stmo] "s"(stmo), [sdbg] "s"(sdbg), [sfo] "s"(sfo), [sfl] "s"(sfl), [srep] "s"(srep),
        [smt] "s"(mt), [srot] "s"(srot),
        [vrep] "v"(vrep), [vct] "v"(vct), [vrd] "v"(vrd), [vte] "v"(vte), [vtl] "v"(vtl), [vta] "v"(vta), [va0] "v"(va0),
        [var] "v"(var), [vfl] "v"(vfl), [vout] "v"(vout), [vew] "v"(vew), [ver] "v"(ver), [vpd] "v"(vpd)
      : "v54", "v55", "v56", "v57", "v58", "v59", "v60", "v61", "v62", "v63", "v64", "v65", "v66", "v67", "v68", "v69", "v70", "v71", "v72", "v73", "v74", "v75", "v76", "v77", "v78", "v79", "v80", "v81", "v82", "v83", "v84", "v85", "v86", "v87", "v88", "v89", "v90", "v91", "v92", "v93", "v94", "v95", "v96", "v97", "v98", "v99", "v100", "v101", "v102", "v103", "v104", "v105", "v106", "v107", "v108", "v109", "v110", "v111", "v112", "v113", "v114", "v115", "v116", "v117", "v118", "v119", "v120", "v121", "v122", "v123", "v124", "v125", "v126", "v127", "v128", "v129", "v130", "v131", "v132", "v133", "v134", "v135", "v136", "v137", "v138", "v139", "v140", "v141", "v142", "v143", "v144", "v145", "v146", "v147", "v148", "v149", "v150", "v151", "v152", "v153", "v154", "v155", "v156", "v157", "v158", "v159", "v160", "v161", "v162", "v163", "v164", "v165", "v166", "v167", "v168", "v169", "v170", "v171", "v172", "v173", "v174", "v175", "v176", "v177", "v178", "v179", "v180", "v181", "v182", "v183", "v184", "v185", "v186", "v187", "v188", "v189", "v190", "v191", "v192", "v193", "v194", "v195", "v196", "v197", "v198", "v199", "v200", "v201", "v202", "v203", "v204", "v205", "v206", "v207", "v208", "v209", "v210", "v211", "v212", "v213", "v214", "v215", "v216", "v217", "v218", "v219", "v220", "v221", "v222", "v223", "v224", "v225", "v226", "v227", "v228", "v229", "v230", "v231", "v232", "v233", "v234", "v235", "v236", "v237", "v238", "v239", "v240", "v241", "v242", "v243", "v244", "v245", "v246", "v247", "v248", "v249", "v250", "v251", "v252", "v253", "v254", "v255", "a0", "a1", "a2", "a3", "a4", "a5", "a6", "a7", "a8", "a9", "a10", "a11", "a12", "a13", "a14", "a15", "a16", "a17", "a18", "a19", "a20", "a21", "a22", "a23", "a24", "a25", "a26", "a27", "a28", "a29", "a30", "a31", "a32", "a33", "a34", "a35", "a36", "a37", "a38", "a39", "a40", "a41", "a42", "a43", "a44", "a45", "a46", "a47", "a48", "a49", "a50", "a51", "a52", "a53", "a54", "a55", "a56", "a57", "a58", "a59", "a60", "a61", "a62", "a63", "a64", "a65", "a66", "a67", "a68", "a69", "a70", "a71", "a72", "a73", "a74", "a75", "a76", "a77", "a78", "a79", "a80", "a81", "a82", "a83", "a84", "a85", "a86", "a87", "a88", "a89", "a90", "a91", "a92", "a93", "a94", "a95", "a96", "a97", "a98", "a99", "a100", "a101", "a102", "a103", "a104", "a105", "a106", "a107", "a108", "a109", "a110", "a111", "a112", "a113", "a114", "a115", "a116", "a117", "a118", "a119", "a120", "a121", "a122", "a123", "a124", "a125", "a126", "a127", "a128", "a129", "a130", "a131", "a132", "a133", "a134", "a135", "a136", "a137", "a138", "a139", "a140", "a141", "a142", "a143", "a144", "a145", "a146", "a147", "a148", "a149", "a150", "a151", "a152", "a153", "a154", "a155", "a156", "a157", "a158", "a159", "a160", "a161", "a162", "a163", "a164", "a165", "a166", "a167", "a168", "a169", "a170", "a171", "a172", "a173", "a174", "a175", "a176", "a177", "a178", "a179", "a180", "a181", "a182", "a183", "a184", "a185", "a186", "a187", "a188", "a189", "a190", "a191", "s56", "s57", "s58", "s59", "s60", "s61", "s62", "s63", "s64", "s65", "s66", "s67", "s68", "s69", "s70", "s71", "s72", "s73", "s74", "s75", "s76", "s77", "s78", "s79", "s80", "s81", "s82", "s83", "s84", "s85", "s86", "s87", "s88", "s89", "s90", "s91", "vcc", "scc", "memory");
}

extern "C" void kernel_launch(int rank, int nranks, ncclComm_t comm, hipStream_t stream,
                              void* const* d_in, const int* in_sizes, int n_in,
                              void* d_out, int out_size, const int* in_dtypes, int out_dtype,
                              void* const* d_out_all, void* const* d_ws_all, size_t ws_bytes) {
  (void)comm; (void)in_sizes; (void)n_in; (void)out_size; (void)in_dtypes; (void)out_dtype; (void)d_out_all;
  if (nranks != NRANK || (unsigned)rank >= (unsigned)NRANK || ws_bytes < WS_USED) {
    fprintf(stderr, "kernel_launch: unsupported configuration\n");
    return;
  }
  Params P;
  P.x = make_srd(d_in[0], KDIM * KSH); P.w = make_srd(d_in[1], KDIM * NDIM); P.out = make_srd(d_out, MROWS * NDIM * 4);
  P.sx = (const float*)d_in[2]; P.sw = (const float*)d_in[3];
  for (int d = 0; d < NRANK; ++d) P.wsr[d] = make_srd(d_ws_all[(rank - d) & (NRANK - 1)], WS_USED);
  P.rank = (unsigned)rank;
  hipLaunchKernelGGL(a2a_gemm_kernel, dim3(256), dim3(256), 0, stream, P);
  HIP_CHECK(hipGetLastError());
}
```
